```python
import numpy as np

kernel_name = "gpu_ag_gemm_m4096_k4096_n4096_e4m3e5m2_dequant"


ABSMAX_RTOL = 0.05

COMPUTE_DTYPE = "float32"
EPILOGUE = "dequant"
IN_QUANT = None
OUT_QUANT = None
IN_FP8 = "e4m3e5m2"
IN_DTYPES = {"x": "e4m3", "w_mat": "e5m2",
             "scale_x": "float32", "scale_w": "float32"}
WORLD_SIZE = 8

def _snap_fp8(x, mant_bits, max_abs, min_normal):
    a = np.abs(x.astype(np.float64))
    _m, _e = np.frexp(a)
    q = np.where(a >= min_normal, np.ldexp(1.0, _e - 1 - mant_bits),
                 min_normal * 2.0 ** -mant_bits)
    snapped = np.minimum(np.round(a / q) * q, max_abs)
    return (np.sign(x) * snapped).astype(np.float32)

def setup_inputs():

    x = _snap_fp8(np.random.randn(4096, 4096).astype(np.float32), 3, 448.0, 2.0 ** -6)
    w_mat = _snap_fp8(np.random.randn(4096, 4096).astype(np.float32), 2, 57344.0, 2.0 ** -14)
    scale_x = (0.02 * (1.0 + np.random.rand(1))).astype(np.float32)
    scale_w = (0.02 * (1.0 + np.random.rand(1))).astype(np.float32)
    return {"x": x, "w_mat": w_mat, "scale_x": scale_x, "scale_w": scale_w}

def reference(x, w_mat, scale_x, scale_w):
    acc = (x @ w_mat).astype(np.float32)
    y = (acc * (scale_x * scale_w)).astype(np.float32)
    return y
```

```cpp
#include <hip/hip_runtime.h>
#include <rccl/rccl.h>
#include <cstdio>
#include <cstdint>

namespace {

constexpr int NR = 8;
constexpr int MS = 512;
constexpr int KD = 4096;
constexpr int NL = 512;
constexpr int NKC = 16;
constexpr int NRG = 32;
constexpr int NWG = 256;
constexpr int NWAVE = 12;
constexpr size_t SHARD_BYTES = (size_t)MS * KD;
constexpr size_t PAR_BYTES = NR * SHARD_BYTES;
constexpr size_t FLAG_OFF = 2 * PAR_BYTES;
constexpr size_t CNT_OFF = FLAG_OFF + (size_t)NR * NRG * 128;
constexpr size_t TMO_OFF = CNT_OFF + (size_t)NWG * 128;
constexpr size_t DBG_OFF = TMO_OFF + 4096;
constexpr size_t WS_NEED = DBG_OFF + (1u << 19);
constexpr unsigned long long DEADLINE_TICKS = 50000000ull;

constexpr int SLOT_BYTES = 16384;
constexpr int RING_BYTES = 8 * SLOT_BYTES;
constexpr int SEND_OFF = 4 * SLOT_BYTES;
constexpr int CTL_OFF = RING_BYTES;
constexpr int LDS_BYTES = RING_BYTES + 256;

typedef unsigned u32x4 __attribute__((ext_vector_type(4)));
typedef int i32x2 __attribute__((ext_vector_type(2)));
typedef int i32x8 __attribute__((ext_vector_type(8)));
typedef float f32x4 __attribute__((ext_vector_type(4)));
#define LAS __attribute__((address_space(3)))
#define GAS __attribute__((address_space(1)))

struct Params {
  const unsigned char* x;
  const unsigned char* w;
  const float* sx;
  const float* sw;
  float* out;
  unsigned char* ws[NR];
  int rank;
  int pad;
};

__device__ __forceinline__ void st16_sys(unsigned char* p, u32x4 v) {
  asm volatile("global_store_dwordx4 %0, %1, off sc0 sc1\n\ts_nop 1" ::"v"(p), "v"(v) : "memory");
}
__device__ __forceinline__ void st4_sys(unsigned char* p, unsigned v) {
  asm volatile("global_store_dword %0, %1, off sc0 sc1" ::"v"(p), "v"(v) : "memory");
}
template <int N>
__device__ __forceinline__ void wait_vm() {
  asm volatile("s_waitcnt vmcnt(%0)" ::"i"(N) : "memory");
}
__device__ __forceinline__ unsigned lds_ld(LAS unsigned char* l, int off) {
  return *(volatile LAS unsigned*)(l + off);
}
__device__ __forceinline__ void lds_st(LAS unsigned char* l, int off, unsigned v) {
  *(volatile LAS unsigned*)(l + off) = v;
}
#define CBAR() asm volatile("" ::: "memory")

#define DBG(slot) do {} while (0)

constexpr int CTL_FULL = CTL_OFF;
constexpr int CTL_DONE = CTL_OFF + 16;
constexpr int CTL_XRDY = CTL_OFF + 32;

__global__ void __launch_bounds__(NWAVE * 64, 3) ag_gemm_kernel(Params p) {
  __shared__ __attribute__((aligned(1024))) unsigned char smem[LDS_BYTES];
  LAS unsigned char* lds = (LAS unsigned char*)smem;
  const int wg = blockIdx.x;
  const int lane = threadIdx.x & 63;
  const int wave = __builtin_amdgcn_readfirstlane(threadIdx.x >> 6);
  const int rb = wg & 7;
  const int cb = (wg >> 3) & 3;
  const int src = (p.rank + (wg >> 5)) & 7;
  const bool local = (wg >> 5) == 0;
  unsigned char* my = p.ws[p.rank];

  if (wave > 4 && !local) return;
  DBG(0);

  unsigned call = 0;
  unsigned* cnt = (unsigned*)(my + CNT_OFF + (size_t)wg * 128) + wave;
  if (wave != 4) call = __hip_atomic_load(cnt, __ATOMIC_RELAXED, __HIP_MEMORY_SCOPE_AGENT);
  if (wave == 4 && lane < 16) lds_st(lds, CTL_OFF + lane * 4, 0u);
  asm volatile("s_waitcnt lgkmcnt(0)" ::: "memory");
  __builtin_amdgcn_s_barrier();
  CBAR();

  const int ring_mask = local ? 3 : 7;

  if (wave == 4) {
    const int kk = lane >> 3, c = lane & 7;
    const unsigned char* wb = p.w + (size_t)kk * NL + cb * 128;
    const int f0 = (kk >> 1) & 3;
    const unsigned char* s0 = wb + ((c ^ f0) << 4);
    const unsigned char* s1 = wb + ((c ^ (f0 | 4)) << 4);
    const int R = ring_mask + 1;
    for (int s = 0; s < 32; ++s) {
      if (s >= R) {
        const unsigned need = s - R + 1;
        bool first = true;
        while (true) {
          unsigned d = lds_ld(lds, CTL_DONE + (lane & 3) * 4);
          if (__all(d >= need)) break;
          if (first) {
            wait_vm<0>();
            lds_st(lds, CTL_FULL, s);
            first = false;
          }
          __builtin_amdgcn_s_sleep(2);
        }
        CBAR();
      }
      LAS unsigned char* dst = lds + (s & ring_mask) * SLOT_BYTES;
      const size_t koff = (size_t)s * 128 * NL;
#pragma unroll
      for (int j = 0; j < 16; ++j) {
        const unsigned char* g = (((j >> 1) & 1) ? s1 : s0) + koff + (size_t)j * 8 * NL;
        __builtin_amdgcn_global_load_lds((const GAS void*)g, (LAS void*)(dst + j * 1024), 16, 0, 0);
      }
      wait_vm<16>();
      lds_st(lds, CTL_FULL, s);
    }
    wait_vm<0>();
    lds_st(lds, CTL_FULL, 32);
    return;
  }

  call = __builtin_amdgcn_readfirstlane(call) + 1;
  if (lane == 0) __hip_atomic_store(cnt, call, __ATOMIC_RELAXED, __HIP_MEMORY_SCOPE_AGENT);
  const size_t par = (size_t)(call & 1) * PAR_BYTES;
  const unsigned fbase = (call - 1) * NKC;
  const int g = lane >> 4, r16 = lane & 15;

  if (wave > 4) {
    __builtin_amdgcn_s_setprio(3);
    const int rg = rb * 4 + cb;
    const unsigned peer = (unsigned)(p.rank + wave - 4) & 7u;
    unsigned char* pw = p.ws[peer];
    unsigned char* dst = pw + par + (size_t)p.rank * SHARD_BYTES + (size_t)rg * 4096 + lane * 16;
    unsigned char* flag = pw + FLAG_OFF + ((size_t)p.rank * NRG + rg) * 128;
    wait_vm<0>();
    for (int kc = 0; kc < NKC; ++kc) {
      while (lds_ld(lds, CTL_XRDY + (kc & 3) * 4) <= (unsigned)(kc >> 2)) __builtin_amdgcn_s_sleep(1);
      CBAR();
      const LAS u32x4* sb = (const LAS u32x4*)(lds + SEND_OFF + kc * 4096 + lane * 16);
      u32x4 v0 = sb[0], v1 = sb[64], v2 = sb[128], v3 = sb[192];
      unsigned char* d = dst + (size_t)kc * (NRG * 4096);
      st16_sys(d, v0);
      wait_vm<1>();
      if (kc > 0 && lane == 0) st4_sys(flag, fbase + kc);
      st16_sys(d + 1024, v1);
      wait_vm<1>();
      st16_sys(d + 2048, v2);
      wait_vm<1>();
      st16_sys(d + 3072, v3);
      wait_vm<1>();
      if (kc == 0) DBG(1);
    }
    wait_vm<0>();
    if (lane == 0) st4_sys(flag, fbase + NKC);
    DBG(2);
    return;
  }

  if (local) {
    const int rg = rb * 4 + cb;
    const unsigned char* xs = p.x + (size_t)(rg * 16 + r16) * KD + g * 16;
#pragma unroll
    for (int i = 0; i < 4; ++i) {
      const int kc = i * 4 + wave;
#pragma unroll
      for (int pc = 0; pc < 4; ++pc)
        __builtin_amdgcn_global_load_lds((const GAS void*)(xs + kc * 256 + pc * 64),
                                         (LAS void*)(lds + SEND_OFF + kc * 4096 + pc * 1024), 16, 0, 0);
    }
    wait_vm<12>(); lds_st(lds, CTL_XRDY + wave * 4, 1);
    wait_vm<8>(); lds_st(lds, CTL_XRDY + wave * 4, 2);
    wait_vm<4>(); lds_st(lds, CTL_XRDY + wave * 4, 3);
    wait_vm<0>(); lds_st(lds, CTL_XRDY + wave * 4, 4);
  }

  const int rg = rb * 4 + wave;
  const unsigned char* abase = local ? p.x : my + par + (size_t)src * SHARD_BYTES;
  const __amdgpu_buffer_rsrc_t ars =
      __builtin_amdgcn_make_buffer_rsrc((void*)abase, 0, (int)SHARD_BYTES, 0x00020000);
  const int avoff = local ? (rg * 16 + r16) * KD + g * 16 : rg * 4096 + lane * 16;
  const int akst = local ? 256 : NRG * 4096;
  const int apst = local ? 64 : 1024;
  const unsigned* pollp = (const unsigned*)(lane == 0 ? my + FLAG_OFF + ((size_t)src * NRG + rg) * 128 : my + TMO_OFF);
  unsigned* tmo = (unsigned*)(my + TMO_OFF);

  f32x4 acc[8];
#pragma unroll
  for (int i = 0; i < 8; ++i) acc[i] = f32x4{0.f, 0.f, 0.f, 0.f};

  const int q = r16 >> 1;
  const int wl = (g * 16 + q) * 128 + ((((q >> 1) & 3) | ((g & 1) << 2)) << 4) + (r16 & 1) * 8;

  bool bad = false;
  int ready = local ? NKC : 0;
  u32x4 nx[4];
  unsigned pv = 0;

  auto spin_until = [&](int need) {
    if (ready >= need) return;
    unsigned long long t0 = __builtin_amdgcn_s_memrealtime();
    while (true) {
      unsigned v = __hip_atomic_load(pollp, __ATOMIC_RELAXED, __HIP_MEMORY_SCOPE_SYSTEM);
      int f = (int)(__builtin_amdgcn_readlane(v, 0) - fbase);
      unsigned t = __builtin_amdgcn_readlane(v, 1);
      if (f >= need) { ready = f; break; }
      if (t == call || __builtin_amdgcn_s_memrealtime() - t0 > DEADLINE_TICKS) {
        if (lane == 0) __hip_atomic_store(tmo, call, __ATOMIC_RELAXED, __HIP_MEMORY_SCOPE_AGENT);
        bad = true;
        ready = NKC;
        break;
      }
      __builtin_amdgcn_s_sleep(1);
    }
    CBAR();
  };
  auto issue = [&](int kc) {
#pragma unroll
    for (int pc = 0; pc < 4; ++pc)
      nx[pc] = __builtin_amdgcn_raw_buffer_load_b128(ars, avoff, kc * akst + pc * apst, 17);
    pv = __hip_atomic_load(pollp, __ATOMIC_RELAXED, __HIP_MEMORY_SCOPE_SYSTEM);
  };

  spin_until(1);
  DBG(1);
  issue(0);
  for (int kc = 0; kc < NKC; ++kc) {
    u32x4 cur[4];
#pragma unroll
    for (int pc = 0; pc < 4; ++pc) cur[pc] = nx[pc];
    {
      int f = (int)(__builtin_amdgcn_readlane(pv, 0) - fbase);
      if (f > ready) ready = f;
      if (__builtin_amdgcn_readlane(pv, 1) == call) bad = true;
    }
    CBAR();
    const bool early = (kc + 1 < NKC) && (ready > kc + 1);
    if (early) issue(kc + 1);
#pragma unroll
    for (int s = 0; s < 2; ++s) {
      const int step = kc * 2 + s;
      while (lds_ld(lds, CTL_FULL) <= (unsigned)step) __builtin_amdgcn_s_sleep(1);
      CBAR();
      i32x8 xb;
      xb[0] = cur[2 * s][0]; xb[1] = cur[2 * s][1]; xb[2] = cur[2 * s][2]; xb[3] = cur[2 * s][3];
      xb[4] = cur[2 * s + 1][0]; xb[5] = cur[2 * s + 1][1]; xb[6] = cur[2 * s + 1][2]; xb[7] = cur[2 * s + 1][3];
      LAS unsigned char* sl = lds + (step & ring_mask) * SLOT_BYTES;
#pragma unroll
      for (int nt = 0; nt < 8; ++nt) {
        LAS unsigned char* a = sl + (wl ^ (nt << 4));
        i32x2 t0 = __builtin_amdgcn_ds_read_tr8_b64_v2i32((LAS i32x2*)(a));
        i32x2 t1 = __builtin_amdgcn_ds_read_tr8_b64_v2i32((LAS i32x2*)(a + 1024));
        i32x2 t2 = __builtin_amdgcn_ds_read_tr8_b64_v2i32((LAS i32x2*)(a + 8192));
        i32x2 t3 = __builtin_amdgcn_ds_read_tr8_b64_v2i32((LAS i32x2*)(a + 9216));
        i32x8 wa;
        wa[0] = t0[0]; wa[1] = t0[1]; wa[2] = t1[0]; wa[3] = t1[1];
        wa[4] = t2[0]; wa[5] = t2[1]; wa[6] = t3[0]; wa[7] = t3[1];
        acc[nt] = __builtin_amdgcn_mfma_scale_f32_16x16x128_f8f6f4(wa, xb, acc[nt], 1, 0, 0, 0, 0, 0);
      }
      CBAR();
      lds_st(lds, CTL_DONE + wave * 4, step + 1);
    }
    if (!early && kc + 1 < NKC) {
      spin_until(kc + 2);
      issue(kc + 1);
    }
  }

  const float sc = p.sx[0] * p.sw[0];
  float* o = p.out + (size_t)(src * MS + rg * 16 + r16) * NL + cb * 128 + g * 4;
#pragma unroll
  for (int nt = 0; nt < 8; ++nt) {
    f32x4 v = acc[nt] * sc;
    if (bad) v = f32x4{__builtin_nanf(""), __builtin_nanf(""), __builtin_nanf(""), __builtin_nanf("")};
    *(f32x4*)(o + nt * 16) = v;
  }
}

}

extern "C" void kernel_launch(int rank, int nranks, ncclComm_t comm, hipStream_t stream,
                              void* const* d_in, const int* in_sizes, int n_in, void* d_out,
                              int out_size, const int* in_dtypes, int out_dtype,
                              void* const* d_out_all, void* const* d_ws_all, size_t ws_bytes) {
  (void)comm; (void)in_dtypes; (void)out_dtype; (void)d_out_all;
  if (nranks != NR || n_in != 4 || (unsigned)rank >= (unsigned)NR || ws_bytes < WS_NEED ||
      in_sizes[0] != MS * KD || in_sizes[1] != KD * NL || out_size != NR * MS * NL) {
    fprintf(stderr, "kernel_launch: unexpected problem shape\n");
    return;
  }
  Params p;
  p.x = (const unsigned char*)d_in[0];
  p.w = (const unsigned char*)d_in[1];
  p.sx = (const float*)d_in[2];
  p.sw = (const float*)d_in[3];
  p.out = (float*)d_out;
  for (int i = 0; i < NR; ++i) p.ws[i] = (unsigned char*)d_ws_all[i];
  p.rank = rank;
  p.pad = 0;
  hipLaunchKernelGGL(ag_gemm_kernel, dim3(NWG), dim3(NWAVE * 64), 0, stream, p);
  hipError_t e = hipGetLastError();
  if (e != hipSuccess) fprintf(stderr, "launch error %s\n", hipGetErrorString(e));
}
```

```cpp
#include <hip/hip_runtime.h>
#include <rccl/rccl.h>
#include <cstdio>
#include <cstdint>

namespace {

constexpr int NR = 8;
constexpr int MS = 512;
constexpr int KD = 4096;
constexpr int NL = 512;
constexpr int NKC = 16;
constexpr int NRG = 32;
constexpr int NWG = 256;
constexpr int NWAVE = 12;
constexpr size_t SHARD_BYTES = (size_t)MS * KD;
constexpr size_t PAR_BYTES = NR * SHARD_BYTES;
constexpr size_t FLAG_OFF = 2 * PAR_BYTES;
constexpr size_t FLAG_PAR = (size_t)NR * NRG * 128;
constexpr size_t CNT_OFF = FLAG_OFF + 2 * FLAG_PAR;
constexpr size_t TMO_OFF = CNT_OFF + (size_t)NWG * 128;
constexpr size_t DBG_OFF = TMO_OFF + 4096;
constexpr size_t WS_NEED = DBG_OFF + (1u << 19);
constexpr unsigned long long DEADLINE_TICKS = 50000000ull;

constexpr int LUT_OFF = 0;
constexpr int CTL_OFF = 2048;
constexpr int SCR_OFF = CTL_OFF + 512;
constexpr int SCR_BYTES = 512 + 2048 + 64;
constexpr int BIG_OFF = 16384;
constexpr int SLOT_BYTES = 16384;
constexpr int RING_OFF = BIG_OFF;
constexpr int ABUF_OFF = RING_OFF + 6 * SLOT_BYTES;
constexpr int LDS_BYTES = ABUF_OFF + 4 * 3 * 4096;
static_assert(SCR_OFF + 4 * SCR_BYTES <= BIG_OFF && LDS_BYTES <= 163840, "LDS map");
constexpr int CTL_FULL = CTL_OFF;
constexpr int CTL_DONE = CTL_OFF + 32;
constexpr int CTL_A = CTL_OFF + 64;
constexpr int CTL_META = CTL_OFF;
constexpr int CTL_TAB = CTL_OFF + 128;
constexpr int TAIL = 3072;

typedef unsigned u32x4 __attribute__((ext_vector_type(4)));
typedef unsigned u32x2 __attribute__((ext_vector_type(2)));
typedef int i32x2 __attribute__((ext_vector_type(2)));
typedef int i32x8 __attribute__((ext_vector_type(8)));
typedef float f32x4 __attribute__((ext_vector_type(4)));
typedef unsigned long long u64;
#define LAS __attribute__((address_space(3)))
#define GAS __attribute__((address_space(1)))

struct Params {
  const unsigned char* x;
  const unsigned char* w;
  const float* sx;
  const float* sw;
  float* out;
  unsigned char* ws[NR];
  int rank;
  int pad;
};

__device__ __forceinline__ void st16_sys(unsigned char* p, u32x4 v) {
  asm volatile("global_store_dwordx4 %0, %1, off sc0 sc1\n\ts_nop 1" ::"v"(p), "v"(v) : "memory");
}
__device__ __forceinline__ void st8_sys(unsigned char* p, u32x2 v) {
  asm volatile("global_store_dwordx2 %0, %1, off sc0 sc1" ::"v"(p), "v"(v) : "memory");
}
template <int N>
__device__ __forceinline__ void wait_vm() {
  asm volatile("s_waitcnt vmcnt(%0)" ::"i"(N) : "memory");
}
__device__ __forceinline__ unsigned lds_ld(LAS unsigned char* l, int off) {
  return *(volatile LAS unsigned*)(l + off);
}
__device__ __forceinline__ void lds_st(LAS unsigned char* l, int off, unsigned v) {
  *(volatile LAS unsigned*)(l + off) = v;
}
#define CBAR() asm volatile("" ::: "memory")

__device__ __forceinline__ unsigned wave_scan(unsigned v) {
  v += __builtin_amdgcn_update_dpp(0u, v, 0x111, 0xf, 0xf, true);
  v += __builtin_amdgcn_update_dpp(0u, v, 0x112, 0xf, 0xf, true);
  v += __builtin_amdgcn_update_dpp(0u, v, 0x114, 0xf, 0xf, true);
  v += __builtin_amdgcn_update_dpp(0u, v, 0x118, 0xf, 0xf, true);
  v += __builtin_amdgcn_update_dpp(0u, v, 0x142, 0xa, 0xf, false);
  v += __builtin_amdgcn_update_dpp(0u, v, 0x143, 0xc, 0xf, false);
  return v;
}
__device__ __forceinline__ unsigned esc_count(unsigned c) {
  return __builtin_popcount(c & (c >> 1) & 0x55555555u);
}

#define DBG(slot) do {} while (0)
#define DBG_FLUSH() do {} while (0)

#define SWIN 4
#define TAIL_ROUND 4
#define NRAW 2
#define NRAW_END 2
#define FORCE_RAW 0

#define LAYOUT 1
#define OUT_AUX 16
__device__ __forceinline__ unsigned blk_off(int rg, int kc) {
  if (LAYOUT == 0) return (unsigned)(kc * NRG + rg) * 4096u;
  if (LAYOUT == 1) return (unsigned)(rg * NKC + kc) * 4096u;
  if (LAYOUT == 2) return (unsigned)(kc * NRG + ((rg + 2 * kc) & 31)) * 4096u;
  return (unsigned)(kc * NRG + ((rg + 5 * kc) & 31)) * 4096u;
}
constexpr unsigned SM_MASK = 0x87878787u;

__global__ void __launch_bounds__(NWAVE * 64, 3) ag_gemm_kernel(Params p) {
  __shared__ __attribute__((aligned(1024))) unsigned char smem[LDS_BYTES];
  LAS unsigned char* lds = (LAS unsigned char*)smem;
  const int wg = blockIdx.x;
  const int lane = threadIdx.x & 63;
  const int wave = __builtin_amdgcn_readfirstlane(threadIdx.x >> 6);
  const bool local = (wg >> 5) == 0;
  const bool sendwg = wg < 16;
  unsigned char* my = p.ws[p.rank];
  const int g = lane >> 4, r16 = lane & 15;

  if (wave > 8 && !sendwg) return;
  DBG(0);

  if (sendwg) {
    const int rg0 = wg * 2;
    if (wave < 5) {
      const unsigned char* xs = p.x + (size_t)(rg0 * 16 + r16) * KD + g * 16;
      u32x4 nx[4];
      auto load = [&](int j) {
        const unsigned char* a = xs + (size_t)(j & 1) * (16 * KD) + (j >> 1) * 256;
#pragma unroll
        for (int pc = 0; pc < 4; ++pc) nx[pc] = *(const u32x4*)(a + pc * 64);
      };
      load(wave);
      if (wave == 4) {
        lds_st(lds, CTL_OFF + lane * 4, 0u);
      } else {
        const int idx = wave * 64 + lane;
        unsigned wts = 0, n = 0;
#pragma unroll
        for (int b = 0; b < 4; ++b) {
          const bool e = ((idx >> (2 * b)) & 3) == 3;
          wts |= e ? (1u << (n + 8 * b)) : 0u;
          n += e ? 2u : 0u;
        }
        *(LAS u32x2*)(lds + LUT_OFF + idx * 8) = u32x2{wts, n};
      }
      asm volatile("s_waitcnt lgkmcnt(0)" ::: "memory");
      __builtin_amdgcn_s_barrier();
      CBAR();
#pragma unroll 1
      for (int j = wave; j < 2 * NKC; j += 5) {
        unsigned d[16];
#pragma unroll
        for (int i = 0; i < 16; ++i) d[i] = nx[i >> 2][i & 3];
        LAS unsigned char* blk = lds + BIG_OFF + j * 4096;
        LAS unsigned char* tab = lds + CTL_TAB + (j & 1) * 64;
        bool raw = j < NRAW || j >= 2 * NKC - NRAW_END || FORCE_RAW;
        const bool raw0 = raw;
        unsigned meta = 0x80030000u | 64u;
        if (!raw) {
          while (lds_ld(tab, 32) == 0) __builtin_amdgcn_s_sleep(1);
          CBAR();
          const u32x4 C = *(LAS u32x4*)tab;
          if (j + 5 < 2 * NKC) load(j + 5);
          unsigned c1[4], a2[4], n2[4], a3 = 0, n3 = 0, badv = 0;
#pragma unroll
          for (int q = 0; q < 4; ++q) {
            unsigned d2[4], d3[4];
            u32x2 w1[4], w2[4];
            c1[q] = 0;
#pragma unroll
            for (int i = 0; i < 4; ++i) {
              const unsigned v = d[q * 4 + i];
              const unsigned e7 = (v >> 3) & 0x07070707u;
              const unsigned lo = __builtin_amdgcn_perm(C[1], C[0], e7);
              const unsigned hi = __builtin_amdgcn_perm(C[3], C[2], e7);
              const unsigned code = __builtin_amdgcn_perm(hi, lo, ((v >> 4) & 0x04040404u) | 0x03020100u);
              badv |= code;
              const unsigned p1 = __builtin_amdgcn_udot4(code & 0x03030303u, 0x40100401u, 0u, false);
              c1[q] |= p1 << (8 * i);
              d2[i] = (code >> 2) & 0x03030303u;
              d3[i] = (code >> 4) & 0x03030303u;
              const unsigned p2 = __builtin_amdgcn_udot4(d2[i], 0x40100401u, 0u, false);
              w1[i] = *(LAS u32x2*)(lds + LUT_OFF + p1 * 8);
              w2[i] = *(LAS u32x2*)(lds + LUT_OFF + p2 * 8);
            }
            a2[q] = 0;
            n2[q] = 0;
#pragma unroll
            for (int i = 0; i < 4; ++i) {
              a2[q] |= __builtin_amdgcn_udot4(d2[i], w1[i][0], 0u, false) << n2[q];
              n2[q] += w1[i][1];
              a3 |= __builtin_amdgcn_udot4(d3[i], w2[i][0], 0u, false) << (n3 & 31);
              n3 += w2[i][1];
            }
          }
          const unsigned mine = (n2[0] + n2[1] + n2[2] + n2[3]) | (n3 << 16);
          const unsigned inc = wave_scan(mine);
          const unsigned tot = __builtin_amdgcn_readlane(inc, 63);
          const unsigned t2 = tot & 0xffff, t3 = tot >> 16;
          raw = __any((badv & 0x80808080u) != 0 || n3 > 32) || t2 > 4096 || t3 > 1024;
          if (!raw) {
            *(LAS u32x4*)(blk + 3072 + lane * 16) = u32x4{0u, 0u, 0u, 0u};
            u32x4 s0, s1;
#pragma unroll
            for (int i = 0; i < 4; ++i) {
              s0[i] = (d[2 * i] & SM_MASK) | (__builtin_amdgcn_alignbit(d[2 * i + 1], d[2 * i + 1], 28) & ~SM_MASK);
              s1[i] = (d[8 + 2 * i] & SM_MASK) | (__builtin_amdgcn_alignbit(d[9 + 2 * i], d[9 + 2 * i], 28) & ~SM_MASK);
            }
            *(LAS u32x4*)(blk + lane * 16) = s0;
            *(LAS u32x4*)(blk + 1024 + lane * 16) = s1;
            *(LAS u32x4*)(blk + 2048 + lane * 16) = u32x4{c1[0], c1[1], c1[2], c1[3]};
            CBAR();
            const unsigned exc = inc - mine;
            unsigned pos = exc & 0xffff;
#pragma unroll
            for (int q = 0; q < 4; ++q) {
              const u64 v = (u64)a2[q] << (pos & 31);
              LAS unsigned* o = (LAS unsigned*)(blk + TAIL + (pos >> 5) * 4);
              __hip_atomic_fetch_or(o, (unsigned)v, __ATOMIC_RELAXED, __HIP_MEMORY_SCOPE_WORKGROUP);
              __hip_atomic_fetch_or(o + 1, (unsigned)(v >> 32), __ATOMIC_RELAXED, __HIP_MEMORY_SCOPE_WORKGROUP);
              pos += n2[q];
            }
            const unsigned b3 = (t2 + 31) & ~31u;
            {
              const unsigned p3 = b3 + (exc >> 16);
              const u64 v = (u64)a3 << (p3 & 31);
              LAS unsigned* o = (LAS unsigned*)(blk + TAIL + (p3 >> 5) * 4);
              __hip_atomic_fetch_or(o, (unsigned)v, __ATOMIC_RELAXED, __HIP_MEMORY_SCOPE_WORKGROUP);
              __hip_atomic_fetch_or(o + 1, (unsigned)(v >> 32), __ATOMIC_RELAXED, __HIP_MEMORY_SCOPE_WORKGROUP);
            }
            unsigned nl = ((b3 + t3 + TAIL_ROUND * 128 - 1) / (TAIL_ROUND * 128)) * TAIL_ROUND;
            if (nl == 0) nl = TAIL_ROUND;
            meta = 0x80000000u | ((nl <= 20 ? 0u : nl <= 24 ? 1u : 2u) << 16) | nl;
          }
        }
        if (raw) {
#pragma unroll
          for (int pc = 0; pc < 4; ++pc)
            *(LAS u32x4*)(blk + pc * 1024 + lane * 16) = u32x4{d[pc * 4], d[pc * 4 + 1], d[pc * 4 + 2], d[pc * 4 + 3]};
        }
        CBAR();
        lds_st(lds, CTL_META + j * 4, meta);
        if (j < 5) DBG(10);
        if (raw0 && j + 5 < 2 * NKC) load(j + 5);
        if (j < 2) {
          u64 h0 = 0, h1 = 0, h2 = 0;
#pragma unroll
          for (int i = 0; i < 9; ++i) {
#pragma unroll
            for (int b = 0; b < 4; ++b) {
              const u64 one = 1ull << (((d[i] >> (8 * b + 3)) & 15u) * 4u);
              if (i < 3) h0 += one; else if (i < 6) h1 += one; else h2 += one;
            }
          }
          const u64 M = 0x0f0f0f0f0f0f0f0full;
          const u64 ev = (h0 & M) + (h1 & M) + (h2 & M);
          const u64 od = ((h0 >> 4) & M) + ((h1 >> 4) & M) + ((h2 >> 4) & M);
          unsigned hw[8];
          hw[0] = (unsigned)ev & 0x00ff00ffu;
          hw[1] = (unsigned)od & 0x00ff00ffu;
          hw[2] = ((unsigned)ev >> 8) & 0x00ff00ffu;
          hw[3] = ((unsigned)od >> 8) & 0x00ff00ffu;
          hw[4] = (unsigned)(ev >> 32) & 0x00ff00ffu;
          hw[5] = (unsigned)(od >> 32) & 0x00ff00ffu;
          hw[6] = (unsigned)(ev >> 40) & 0x00ff00ffu;
          hw[7] = (unsigned)(od >> 40) & 0x00ff00ffu;
          unsigned cnt = 0;
#pragma unroll
          for (int t = 0; t < 8; ++t) {
            const unsigned s = __builtin_amdgcn_readlane(wave_scan(hw[t]), 63);
            const int e0 = (t & 3) + (t >> 2) * 8;
            cnt = r16 == e0 ? (s & 0xffff) : cnt;
            cnt = r16 == e0 + 4 ? (s >> 16) : cnt;
          }
          const unsigned nb = __builtin_amdgcn_update_dpp(0u, cnt, 0x111, 0xf, 0xf, true) +
                              __builtin_amdgcn_update_dpp(0u, cnt, 0x101, 0xf, 0xf, true);
          const unsigned key = (cnt << 17) | ((nb < 8191u ? nb : 8191u) << 4) | (unsigned)r16;
          unsigned rank = 0;
#pragma unroll
          for (int t = 0; t < 16; ++t) rank += __builtin_amdgcn_readlane(key, t) > key ? 1u : 0u;
          const unsigned code = rank < 3 ? rank : rank < 6 ? (3u | ((rank - 3) << 2)) : rank < 10 ? (15u | ((rank - 6) << 4)) : 0x80u;
          const unsigned slot = rank < 3 ? rank : rank < 6 ? rank + 1 : rank + 2;
          if (lane < 16) {
            *(volatile LAS unsigned char*)(tab + lane) = (unsigned char)code;
            if (rank < 10) *(volatile LAS unsigned char*)(tab + 16 + slot) = (unsigned char)(lane << 3);
          }
          CBAR();
          lds_st(tab, 32, 1u);
        }
        DBG(1 + j / 5);
      }
      DBG_FLUSH();
      return;
    }
    unsigned* cnt = (unsigned*)(my + CNT_OFF + (size_t)wg * 128) + wave;
    unsigned call = __hip_atomic_load(cnt, __ATOMIC_RELAXED, __HIP_MEMORY_SCOPE_AGENT);
    __builtin_amdgcn_s_barrier();
    CBAR();
    __builtin_amdgcn_s_setprio(3);
    call = __builtin_amdgcn_readfirstlane(call) + 1;
    const unsigned fbase = (call - 1) * NKC;
    DBG(1);
    const unsigned peer = (unsigned)(p.rank + wave - 4) & 7u;
    unsigned char* pw = p.ws[peer];
    unsigned char* dst = pw + (size_t)(call & 1) * PAR_BYTES + (size_t)p.rank * SHARD_BYTES + lane * 16;
    unsigned char* flag = pw + FLAG_OFF + (size_t)(call & 1) * FLAG_PAR + ((size_t)p.rank * NRG + rg0) * 128;
    constexpr int LAG = (SWIN + 4) / 5;
    static_assert(SWIN <= 9, "an older flag store to the same word must have completed");
    u64 cls = 0;
#pragma unroll 1
    for (int j = 0; j < 2 * NKC; ++j) {
      unsigned meta;
      while ((meta = __builtin_amdgcn_readfirstlane(lds_ld(lds, CTL_META + j * 4))) == 0) __builtin_amdgcn_s_sleep(1);
      CBAR();
      const LAS u32x4* sb = (const LAS u32x4*)(lds + BIG_OFF + j * 4096 + lane * 16);
      const u32x4 v0 = sb[0], v1 = sb[64], v2 = sb[128], v3 = sb[192];
      unsigned char* d = dst + blk_off(rg0 + (j & 1), j >> 1);
      cls |= (u64)((meta >> 16) & 3u) << ((j >> 1) * 2 + (j & 1) * 32);
      if (j == NRAW || j == NRAW + 1) {
        const u32x2 tv = *(const LAS u32x2*)(lds + CTL_TAB + (j & 1) * 64 + 16 + (lane & 1) * 8);
        if (lane < 2) st8_sys(flag + (j & 1) * 128 + 8 + lane * 8, tv);
      }
      st16_sys(d, v0);
      wait_vm<SWIN>();
      st16_sys(d + 1024, v1);
      wait_vm<SWIN>();
      st16_sys(d + 2048, v2);
      wait_vm<SWIN>();
      if ((unsigned)lane < (meta & 0xff)) st16_sys(d + 3072, v3);
      wait_vm<SWIN>();
      const int jf = j + 2 - LAG;
      if (lane == 0) st8_sys(flag + (jf & 1) * 128, u32x2{fbase + (jf >> 1), (unsigned)(cls >> ((jf & 1) * 32))});
      DBG(2 + (j >> 1));
    }
    if (LAG == 2) {
      wait_vm<5>();
      if (lane == 0) st8_sys(flag, u32x2{fbase + NKC, (unsigned)cls});
    }
    wait_vm<1>();
    if (lane == 0) st8_sys(flag + 128, u32x2{fbase + NKC, (unsigned)(cls >> 32)});
    if (lane == 0) __hip_atomic_store(cnt, call, __ATOMIC_RELAXED, __HIP_MEMORY_SCOPE_AGENT);
    DBG(18);
    DBG_FLUSH();
    return;
  }

  const int cb = local ? (wg & 3) : ((wg >> 3) & 3);
  const int src = (p.rank + (wg >> 5)) & 7;

  if (wave == 4) {
    lds_st(lds, CTL_OFF + lane * 4, (!local && lane >= 12 && lane < 16) ? 1000u : 0u);
    asm volatile("s_waitcnt lgkmcnt(0)" ::: "memory");
    __builtin_amdgcn_s_barrier();
    const int kk = lane >> 3, c = lane & 7;
    const unsigned char* wb = p.w + (size_t)kk * NL + cb * 128;
    const int f0 = (kk >> 1) & 3;
    const unsigned char* s0 = wb + ((c ^ f0) << 4);
    const unsigned char* s1 = wb + ((c ^ (f0 | 4)) << 4);
    int slot = 0;
#pragma unroll 1
    for (int s = 0; s < 32; ++s) {
      if (s >= 6) {
        const unsigned need = s - 5;
        bool first = true;
        while (true) {
          unsigned d = lds_ld(lds, CTL_DONE + (lane & 7) * 4);
          if (__all(d >= need)) break;
          if (first) {
            wait_vm<0>();
            lds_st(lds, CTL_FULL, s);
            first = false;
          }
          __builtin_amdgcn_s_sleep(2);
        }
        CBAR();
      }
      LAS unsigned char* dst = lds + RING_OFF + slot * SLOT_BYTES;
      slot = slot == 5 ? 0 : slot + 1;
      const size_t koff = (size_t)s * 128 * NL;
#pragma unroll
      for (int j = 0; j < 16; ++j) {
        const unsigned char* gp = (((j >> 1) & 1) ? s1 : s0) + koff + (size_t)j * 8 * NL;
        __builtin_amdgcn_global_load_lds((const GAS void*)gp, (LAS void*)(dst + j * 1024), 16, 0, 0);
      }
      wait_vm<16>();
      lds_st(lds, CTL_FULL, s);
    }
    wait_vm<0>();
    lds_st(lds, CTL_FULL, 32);
    return;
  }

  if (!local && wave > 4) {
    const int gw = wave - 5;
    unsigned* cnt = (unsigned*)(my + CNT_OFF + (size_t)wg * 128) + wave;
    unsigned call = __hip_atomic_load(cnt, __ATOMIC_RELAXED, __HIP_MEMORY_SCOPE_AGENT);
    __builtin_amdgcn_s_barrier();
    CBAR();
    call = __builtin_amdgcn_readfirstlane(call) + 1;
    if (lane == 0) __hip_atomic_store(cnt, call, __ATOMIC_RELAXED, __HIP_MEMORY_SCOPE_AGENT);
    const unsigned fbase = (call - 1) * NKC;
    const int rg = (wg & 7) * 4 + gw;
    const unsigned char* ap = my + (size_t)(call & 1) * PAR_BYTES + (size_t)src * SHARD_BYTES + lane * 16;
    const u64* pollp = (const u64*)(lane < 3 ? my + FLAG_OFF + (size_t)(call & 1) * FLAG_PAR + ((size_t)src * NRG + rg) * 128 + lane * 8
                                             : my + TMO_OFF);
    unsigned* tmo = (unsigned*)(my + TMO_OFF);
    LAS unsigned char* ctl = lds + CTL_A + gw * 32;
    u64 pv = __hip_atomic_load(pollp, __ATOMIC_RELAXED, __HIP_MEMORY_SCOPE_SYSTEM);
    int slot = 0;
#pragma unroll 1
    for (int kc = 0; kc < NKC; ++kc) {
      bool bad = false;
      const unsigned long long t0 = __builtin_amdgcn_s_memrealtime();
      while ((int)(__builtin_amdgcn_readlane((unsigned)pv, 0) - fbase) <= kc) {
        if (__builtin_amdgcn_readlane((unsigned)pv, 3) == call || __builtin_amdgcn_s_memrealtime() - t0 > DEADLINE_TICKS) {
          bad = true;
          break;
        }
        pv = __hip_atomic_load(pollp, __ATOMIC_RELAXED, __HIP_MEMORY_SCOPE_SYSTEM);
      }
      CBAR();
      if (kc == 15) DBG(17);
      if (!bad) {
        const unsigned cls = __builtin_amdgcn_readlane((unsigned)(pv >> 32), 0);
        const unsigned c = (cls >> (2 * kc)) & 3u;
        const unsigned nl = c == 0 ? 20u : c == 1 ? 24u : c == 2 ? 40u : 64u;
        while (lds_ld(ctl, 24) + 3 <= (unsigned)kc) __builtin_amdgcn_s_sleep(1);
        CBAR();
        LAS unsigned char* dst = lds + ABUF_OFF + (gw * 3 + slot) * 4096;
        slot = slot == 2 ? 0 : slot + 1;
        const unsigned char* a = ap + blk_off(rg, kc);
        __builtin_amdgcn_global_load_lds((const GAS void*)(a + 2048), (LAS void*)(dst + 2048), 16, 0, 17);
        if ((unsigned)lane < nl) __builtin_amdgcn_global_load_lds((const GAS void*)(a + 3072), (LAS void*)(dst + 3072), 16, 0, 17);
        __builtin_amdgcn_global_load_lds((const GAS void*)(a), (LAS void*)(dst), 16, 0, 17);
        __builtin_amdgcn_global_load_lds((const GAS void*)(a + 1024), (LAS void*)(dst + 1024), 16, 0, 17);
        pv = __hip_atomic_load(pollp, __ATOMIC_RELAXED, __HIP_MEMORY_SCOPE_SYSTEM);
        wait_vm<0>();
        bad = __builtin_amdgcn_readlane((unsigned)pv, 3) == call;
        if (lane == 1 || lane == 2) *(volatile LAS u64*)(ctl + lane * 8) = pv;
        CBAR();
        if (!bad && lane == 0) *(volatile LAS u64*)(ctl) = ((u64)cls << 32) | (unsigned)(kc + 1);
      }
      if (bad) {
        if (lane == 0) {
          __hip_atomic_store(tmo, call, __ATOMIC_RELAXED, __HIP_MEMORY_SCOPE_AGENT);
          *(volatile LAS u64*)(ctl) = 0xffffffff80000000ull;
        }
        break;
      }
      DBG(1 + kc);
    }
    DBG_FLUSH();
    return;
  }

  const int gw = wave < 4 ? wave : wave - 1;
  const float sc = p.sx[0] * p.sw[0];
  const int rg = local ? ((wg >> 2) & 3) * 8 + gw : (wg & 7) * 4 + gw;
  const int q = r16 >> 1;
  const int wl = RING_OFF + (g * 16 + q) * 128 + ((((q >> 1) & 3) | ((g & 1) << 2)) << 4) + (r16 & 1) * 8;

  f32x4 acc[8];
#pragma unroll
  for (int i = 0; i < 8; ++i) acc[i] = f32x4{0.f, 0.f, 0.f, 0.f};

  auto mma = [&](int kc, int slot, const unsigned* xv) {
    i32x8 wa[8];
    auto read_w = [&](LAS unsigned char* sl, int nt) {
      LAS unsigned char* a = sl + (wl ^ (nt << 4));
      i32x2 t0 = __builtin_amdgcn_ds_read_tr8_b64_v2i32((LAS i32x2*)(a));
      i32x2 t1 = __builtin_amdgcn_ds_read_tr8_b64_v2i32((LAS i32x2*)(a + 1024));
      i32x2 t2 = __builtin_amdgcn_ds_read_tr8_b64_v2i32((LAS i32x2*)(a + 8192));
      i32x2 t3 = __builtin_amdgcn_ds_read_tr8_b64_v2i32((LAS i32x2*)(a + 9216));
      wa[nt][0] = t0[0]; wa[nt][1] = t0[1]; wa[nt][2] = t1[0]; wa[nt][3] = t1[1];
      wa[nt][4] = t2[0]; wa[nt][5] = t2[1]; wa[nt][6] = t3[0]; wa[nt][7] = t3[1];
    };
    LAS unsigned char* sl = lds + slot * (2 * SLOT_BYTES);
#pragma unroll
    for (int nt = 0; nt < 8; ++nt) read_w(sl, nt);
    __builtin_amdgcn_sched_barrier(0);
    i32x8 xb;
#pragma unroll
    for (int i = 0; i < 8; ++i) xb[i] = xv[i];
#pragma unroll
    for (int nt = 0; nt < 8; ++nt) {
      acc[nt] = __builtin_amdgcn_mfma_scale_f32_16x16x128_f8f6f4(wa[nt], xb, acc[nt], 1, 0, 0, 0, 0, 0);
      __builtin_amdgcn_sched_barrier(0);
      read_w(sl + SLOT_BYTES, nt);
      __builtin_amdgcn_sched_barrier(0);
    }
    CBAR();
    lds_st(lds, CTL_DONE + gw * 4, kc * 2 + 2);
#pragma unroll
    for (int i = 0; i < 8; ++i) xb[i] = xv[8 + i];
#pragma unroll
    for (int nt = 0; nt < 8; ++nt)
      acc[nt] = __builtin_amdgcn_mfma_scale_f32_16x16x128_f8f6f4(wa[nt], xb, acc[nt], 1, 0, 0, 0, 0, 0);
  };

  bool bad = false;
  if (local) {
    const unsigned char* xs = p.x + (size_t)(rg * 16 + r16) * KD + g * 16;
    u32x4 n0[4];
#pragma unroll
    for (int pc = 0; pc < 4; ++pc) n0[pc] = *(const u32x4*)(xs + pc * 64);
    __builtin_amdgcn_s_barrier();
    CBAR();
    int slot = 0;
#pragma unroll 1
    for (int kc = 0; kc < NKC; ++kc) {
      unsigned xv[16];
#pragma unroll
      for (int i = 0; i < 16; ++i) xv[i] = n0[i >> 2][i & 3];
      const int kn = kc + 1 < NKC ? kc + 1 : NKC - 1;
#pragma unroll
      for (int pc = 0; pc < 4; ++pc) n0[pc] = *(const u32x4*)(xs + kn * 256 + pc * 64);
      while (lds_ld(lds, CTL_FULL) <= (unsigned)(kc * 2 + 1)) __builtin_amdgcn_s_sleep(1);
      CBAR();
      mma(kc, slot, xv);
      slot = slot == 2 ? 0 : slot + 1;
      DBG(2 + kc);
    }
  } else {
    {
      const int idx = wave * 64 + lane;
      unsigned sel = 0, n = 0;
#pragma unroll
      for (int b = 0; b < 4; ++b) {
        const unsigned dg = (idx >> (2 * b)) & 3;
        sel |= (dg == 3 ? 4u + n : dg) << (8 * b);
        n += dg == 3 ? 1u : 0u;
      }
      *(LAS u32x2*)(lds + LUT_OFF + idx * 8) = u32x2{sel, n};
    }
    asm volatile("s_waitcnt lgkmcnt(0)" ::: "memory");
    __builtin_amdgcn_s_barrier();
    CBAR();
    LAS unsigned char* ctl = lds + CTL_A + gw * 32;
    const unsigned e3 = SCR_OFF + gw * SCR_BYTES, e2 = e3 + 512;
    int slot = 0;
#pragma unroll 1
    for (int kc = 0; kc < NKC; ++kc) {
      while (lds_ld(lds, CTL_FULL) <= (unsigned)(kc * 2 + 1)) __builtin_amdgcn_s_sleep(1);
      u64 st;
      while ((unsigned)(st = *(volatile LAS u64*)ctl) <= (unsigned)kc) __builtin_amdgcn_s_sleep(1);
      CBAR();
      if (kc == 15) DBG(1);
      bad = (int)__builtin_amdgcn_readfirstlane((unsigned)st) < 0;
      const bool israw = ((__builtin_amdgcn_readfirstlane((unsigned)(st >> 32)) >> (2 * kc)) & 3u) == 3u;
      LAS unsigned char* blk = lds + ABUF_OFF + (gw * 3 + slot) * 4096;
      unsigned xv[16];
      if (israw) {
#pragma unroll
        for (int pc = 0; pc < 4; ++pc) {
          const u32x4 v = *(LAS u32x4*)(blk + pc * 1024 + lane * 16);
          xv[pc * 4] = v[0]; xv[pc * 4 + 1] = v[1]; xv[pc * 4 + 2] = v[2]; xv[pc * 4 + 3] = v[3];
        }
        CBAR();
        lds_st(ctl, 24, kc + 1);
      } else {
        const u32x4 c1 = *(LAS u32x4*)(blk + 2048 + lane * 16);
        const unsigned l2a = *(LAS unsigned*)(blk + TAIL + lane * 4);
        const u32x2 T12 = *(LAS u32x2*)(ctl + 8);
        const unsigned T3 = *(LAS unsigned*)(ctl + 16);
        auto lut2 = [&](unsigned c, int i) { return *(LAS u32x2*)(lds + LUT_OFF + ((c >> (8 * i)) & 255u) * 8); };
        auto pick = [&](unsigned off, unsigned T, unsigned sel) {
          const LAS unsigned* w = (const LAS unsigned*)(lds + (off & ~3u));
          return __builtin_amdgcn_perm(__builtin_amdgcn_alignbyte(w[1], w[0], off), T, sel);
        };
        const unsigned mine = (esc_count(c1[0]) + esc_count(c1[1]) + esc_count(c1[2]) + esc_count(c1[3])) | (esc_count(l2a) << 16);
        const unsigned inc = wave_scan(mine);
        const unsigned exc = inc - mine;
        const unsigned tot = __builtin_amdgcn_readlane(inc, 63);
        const unsigned n2 = tot & 0xffff;
        const unsigned l3 = *(LAS unsigned short*)(blk + TAIL + ((n2 + 15) >> 4) * 4 + lane * 2);
        u32x2 s2[4], s1[16];
#pragma unroll
        for (int i = 0; i < 4; ++i) s2[i] = lut2(l2a, i);
        {
          const unsigned T3x = __builtin_amdgcn_perm(T3, T3, 0x03030303u);
          const unsigned sa = lut2(l3, 0)[0], sb = lut2(l3, 1)[0];
          *(LAS u32x2*)(lds + e3 + lane * 8) = u32x2{__builtin_amdgcn_perm(T3x, T3, sa), __builtin_amdgcn_perm(T3x, T3, sb)};
        }
        CBAR();
        {
          u32x4 o;
          unsigned off = e3 + (exc >> 16);
#pragma unroll
          for (int i = 0; i < 4; ++i) { o[i] = pick(off, T12[1], s2[i][0]); off += s2[i][1]; }
#pragma unroll
          for (int i = 0; i < 16; ++i) s1[i] = lut2(c1[i >> 2], i & 3);
          *(LAS u32x4*)(lds + e2 + lane * 16) = o;
          if (n2 > 1024) {
            const unsigned l2b = *(LAS unsigned*)(blk + TAIL + 256 + lane * 4);
            const unsigned m2 = esc_count(l2b);
            off = e3 + (tot >> 16) + wave_scan(m2) - m2;
#pragma unroll
            for (int i = 0; i < 4; ++i) s2[i] = lut2(l2b, i);
#pragma unroll
            for (int i = 0; i < 4; ++i) { o[i] = pick(off, T12[1], s2[i][0]); off += s2[i][1]; }
            *(LAS u32x4*)(lds + e2 + 1024 + lane * 16) = o;
          }
        }
        CBAR();
        const u32x4 sm0 = *(LAS u32x4*)(blk + lane * 16), sm1 = *(LAS u32x4*)(blk + 1024 + lane * 16);
        CBAR();
        lds_st(ctl, 24, kc + 1);
        unsigned off = e2 + (exc & 0xffff);
#pragma unroll
        for (int i = 0; i < 16; ++i) {
          const unsigned s = i < 8 ? sm0[i >> 1] : sm1[(i >> 1) & 3];
          xv[i] = (((i & 1) ? __builtin_amdgcn_alignbit(s, s, 4) : s) & SM_MASK) | pick(off, T12[0], s1[i][0]);
          off += s1[i][1];
        }
        CBAR();
      }
      mma(kc, slot, xv);
      slot = slot == 2 ? 0 : slot + 1;
      DBG(2 + kc);
    }
  }
  {
    const __amdgpu_buffer_rsrc_t ors =
        __builtin_amdgcn_make_buffer_rsrc((void*)p.out, 0, NR * MS * NL * 4, 0x00020000);
    const int ooff = ((src * MS + rg * 16 + r16) * NL + cb * 128 + g * 4) * 4;
    const float s = bad ? __builtin_nanf("") : sc;
#pragma unroll
    for (int nt = 0; nt < 8; ++nt) {
      const f32x4 v = acc[nt] * s;
      __builtin_amdgcn_raw_buffer_store_b128(__builtin_bit_cast(u32x4, v), ors, ooff + nt * 64, 0, OUT_AUX);
    }
  }
}

}

extern "C" void kernel_launch(int rank, int nranks, ncclComm_t comm, hipStream_t stream,
                              void* const* d_in, const int* in_sizes, int n_in, void* d_out,
                              int out_size, const int* in_dtypes, int out_dtype,
                              void* const* d_out_all, void* const* d_ws_all, size_t ws_bytes) {
  (void)comm; (void)in_dtypes; (void)out_dtype; (void)d_out_all;
  if (nranks != NR || n_in != 4 || (unsigned)rank >= (unsigned)NR || ws_bytes < WS_NEED ||
      in_sizes[0] != MS * KD || in_sizes[1] != KD * NL || out_size != NR * MS * NL) {
    fprintf(stderr, "kernel_launch: unexpected problem shape\n");
    return;
  }
  Params p;
  p.x = (const unsigned char*)d_in[0];
  p.w = (const unsigned char*)d_in[1];
  p.sx = (const float*)d_in[2];
  p.sw = (const float*)d_in[3];
  p.out = (float*)d_out;
  for (int i = 0; i < NR; ++i) p.ws[i] = (unsigned char*)d_ws_all[i];
  p.rank = rank;
  p.pad = 0;
  hipLaunchKernelGGL(ag_gemm_kernel, dim3(NWG), dim3(NWAVE * 64), 0, stream, p);
  hipError_t e = hipGetLastError();
  if (e != hipSuccess) fprintf(stderr, "launch error %s\n", hipGetErrorString(e));
}
```

```cpp
#include <hip/hip_runtime.h>
#include <rccl/rccl.h>
#include <cstdio>
#include <cstdint>

namespace {

constexpr int NR = 8;
constexpr int MS = 512;
constexpr int KD = 4096;
constexpr int NL = 512;
constexpr int NKC = 16;
constexpr int NRG = 32;
constexpr int NWG = 256;
constexpr int NWAVE = 10;
constexpr size_t SHARD_BYTES = (size_t)MS * KD;
constexpr size_t PAR_BYTES = NR * SHARD_BYTES;
constexpr size_t FLAG_OFF = 2 * PAR_BYTES;
constexpr size_t FLAG_PAR = (size_t)NR * (NRG / 2) * 128;
constexpr size_t CNT_OFF = FLAG_OFF + 2 * FLAG_PAR;
constexpr size_t TMO_OFF = CNT_OFF + (size_t)NWG * 128;
constexpr size_t TICK_OFF = TMO_OFF + 128;
constexpr size_t DBG_OFF = TMO_OFF + 4096;
constexpr size_t WS_NEED = DBG_OFF + (1u << 19);
constexpr unsigned long long DEADLINE_TICKS = 50000000ull;

constexpr int LUT_OFF = 0;
constexpr int CTL_OFF = 2048;
constexpr int SCR_OFF = CTL_OFF + 512;
constexpr int SCR_BYTES = 512 + 2048 + 64;
constexpr int BIG_OFF = 16384;
constexpr int SLOT_BYTES = 16384;
constexpr int RING_OFF = BIG_OFF;
constexpr int ABUF_OFF = RING_OFF + 6 * SLOT_BYTES;
constexpr int LDS_BYTES = ABUF_OFF + 4 * 3 * 4096;
static_assert(SCR_OFF + 4 * SCR_BYTES <= BIG_OFF && LDS_BYTES <= 163840, "LDS map");
constexpr int CTL_FULL = CTL_OFF;
constexpr int CTL_DONE = CTL_OFF + 32;
constexpr int CTL_A = CTL_OFF + 64;
constexpr int CTL_META = CTL_OFF;
constexpr int CTL_PRDY = CTL_OFF + 128;
constexpr int CTL_PROG = CTL_OFF + 256;
constexpr int TAB_OFF = SCR_OFF;
constexpr int QUE_OFF = SCR_OFF + 1024;
constexpr int NSTAT = 2 * NKC - 2;
constexpr unsigned TICKETS = NRG + 16;
constexpr int TAIL = 3072;

typedef unsigned u32x4 __attribute__((ext_vector_type(4)));
typedef unsigned u32x3 __attribute__((ext_vector_type(3)));
typedef unsigned u32x2 __attribute__((ext_vector_type(2)));
typedef int i32x2 __attribute__((ext_vector_type(2)));
typedef int i32x8 __attribute__((ext_vector_type(8)));
typedef float f32x4 __attribute__((ext_vector_type(4)));
typedef unsigned long long u64;
#define LAS __attribute__((address_space(3)))
#define GAS __attribute__((address_space(1)))

struct Params {
  const unsigned char* x;
  const unsigned char* w;
  const float* sx;
  const float* sw;
  float* out;
  unsigned char* ws[NR];
  int rank;
  int pad;
};

__device__ __forceinline__ void st16_sys(unsigned char* p, u32x4 v) {
  asm volatile("global_store_dwordx4 %0, %1, off sc0 sc1\n\ts_nop 1" ::"v"(p), "v"(v) : "memory");
}
__device__ __forceinline__ void st8_sys(unsigned char* p, u32x2 v) {
  asm volatile("global_store_dwordx2 %0, %1, off sc0 sc1" ::"v"(p), "v"(v) : "memory");
}
template <int N>
__device__ __forceinline__ void wait_vm() {
  asm volatile("s_waitcnt vmcnt(%0)" ::"i"(N) : "memory");
}
__device__ __forceinline__ unsigned lds_ld(LAS unsigned char* l, int off) {
  return *(volatile LAS unsigned*)(l + off);
}
__device__ __forceinline__ void lds_st(LAS unsigned char* l, int off, unsigned v) {
  *(volatile LAS unsigned*)(l + off) = v;
}
#define CBAR() asm volatile("" ::: "memory")

__device__ __forceinline__ unsigned wave_scan(unsigned v) {
  v += __builtin_amdgcn_update_dpp(0u, v, 0x111, 0xf, 0xf, true);
  v += __builtin_amdgcn_update_dpp(0u, v, 0x112, 0xf, 0xf, true);
  v += __builtin_amdgcn_update_dpp(0u, v, 0x114, 0xf, 0xf, true);
  v += __builtin_amdgcn_update_dpp(0u, v, 0x118, 0xf, 0xf, true);
  v += __builtin_amdgcn_update_dpp(0u, v, 0x142, 0xa, 0xf, false);
  v += __builtin_amdgcn_update_dpp(0u, v, 0x143, 0xc, 0xf, false);
  return v;
}
__device__ __forceinline__ unsigned esc_count(unsigned c) {
  return __builtin_popcount(c & (c >> 1) & 0x55555555u);
}

#define DBG(slot) do {} while (0)
#define DBG_FLUSH() do {} while (0)

#define LEAD 3
#define QUIET 0
#define OUT_AUX 18
__device__ __forceinline__ unsigned blk_off(int rg, int kc) { return (unsigned)(rg * NKC + kc) * 4096u; }
__device__ __forceinline__ unsigned tail_lanes(unsigned c) { return c == 0 ? 20u : c == 1 ? 24u : c == 2 ? 40u : 64u; }
constexpr unsigned SM_MASK = 0x87878787u;

__global__ void __launch_bounds__(NWAVE * 64, 3) ag_gemm_kernel(Params p) {
  __shared__ __attribute__((aligned(1024))) unsigned char smem[LDS_BYTES];
  LAS unsigned char* lds = (LAS unsigned char*)smem;
  const int wg = blockIdx.x;
  const int lane = threadIdx.x & 63;
  const int wave = __builtin_amdgcn_readfirstlane(threadIdx.x >> 6);
  const int grp = wg >> 5;
  unsigned char* my = p.ws[p.rank];
  const int g = lane >> 4, r16 = lane & 15;

  if (wave == 9 && (grp == 0 || grp > 4)) return;
  DBG(0);

  if (grp == 0) {
    const int pair = wg & 15, rg0 = pair * 2;
    const int peer0 = (wg >> 4) * 4, nsend = 4 - (wg >> 4);
    if (wave == 0) {
      const unsigned char* xs = p.x + (size_t)r16 * KD + g * 16;
#pragma unroll
      for (int i = 0; i < 8; ++i)
        __builtin_amdgcn_global_load_lds((const GAS void*)(xs + (size_t)(rg0 + (i >> 2)) * (16 * KD) + (i & 3) * 64),
                                         (LAS void*)(lds + BIG_OFF + i * 1024), 16, 0, 0);
      __builtin_amdgcn_s_barrier();
      wait_vm<4>();
      lds_st(lds, CTL_META, 0x80030000u | 64u);
      wait_vm<0>();
      lds_st(lds, CTL_META + 4, 0x80030000u | 64u);
      DBG(1);
      unsigned* cnt = (unsigned*)(my + CNT_OFF + (size_t)wg * 128);
      const unsigned call = __builtin_amdgcn_readfirstlane(__hip_atomic_load(cnt, __ATOMIC_RELAXED, __HIP_MEMORY_SCOPE_AGENT)) + 1;
      if (lane == 0) __hip_atomic_store(cnt, call, __ATOMIC_RELAXED, __HIP_MEMORY_SCOPE_AGENT);
      unsigned* tick = (unsigned*)(my + TICK_OFF + (peer0 + (lane & 3)) * 128);
      unsigned got = 0;
      bool live = lane < nsend;
      while (__any(live)) {
        if (live && lds_ld(lds, CTL_PROG + lane * 4) + LEAD > NSTAT + got) {
          const unsigned t = __hip_atomic_fetch_add(tick, 1u, __ATOMIC_RELAXED, __HIP_MEMORY_SCOPE_AGENT) - (call - 1) * TICKETS;
          lds_st(lds, QUE_OFF + lane * 256 + got * 4, 1u + (t < NRG ? t : NRG));
          live = t < NRG;
          ++got;
        }
        __builtin_amdgcn_s_sleep(2);
      }
      DBG_FLUSH();
      return;
    }
    if (wave < 5) {
      const unsigned char* xs = p.x + (size_t)(rg0 * 16 + r16) * KD + g * 16;
      const int j0 = 1 + wave;
      __builtin_amdgcn_s_barrier();
      __builtin_amdgcn_sched_barrier(0);
      CBAR();
      const u32x3 smp = *(const u32x3*)(p.x + (size_t)(rg0 * 16 + (lane & 31)) * KD + (lane >> 5) * 16);
      u32x4 nx[4];
      auto load = [&](int j) {
        const unsigned char* a = xs + (size_t)(j & 1) * (16 * KD) + (j >> 1) * 256;
#pragma unroll
        for (int pc = 0; pc < 4; ++pc) nx[pc] = *(const u32x4*)(a + pc * 64);
      };
      load(j0);
      CBAR();
      __builtin_amdgcn_sched_barrier(0);
      u32x4 C;
      {
        LAS unsigned char* tab = lds + TAB_OFF + wave * 128;
        u64 h = 0;
#pragma unroll
        for (int i = 0; i < 3; ++i) {
#pragma unroll
          for (int b = 0; b < 4; ++b) h += 1ull << (((smp[i] >> (8 * b + 3)) & 15u) * 4u);
        }
        u32x4 hb = {(unsigned)h & 0x0f0f0f0fu, (unsigned)(h >> 32) & 0x0f0f0f0fu,
                    ((unsigned)h >> 4) & 0x0f0f0f0fu, (unsigned)(h >> 36) & 0x0f0f0f0fu};
#pragma unroll
        for (int i = 0; i < 4; ++i) {
          hb[i] += __builtin_amdgcn_update_dpp(0u, hb[i], 0x121, 0xf, 0xf, false);
          hb[i] += __builtin_amdgcn_update_dpp(0u, hb[i], 0x122, 0xf, 0xf, false);
          hb[i] += __builtin_amdgcn_update_dpp(0u, hb[i], 0x124, 0xf, 0xf, false);
          hb[i] += __builtin_amdgcn_update_dpp(0u, hb[i], 0x128, 0xf, 0xf, false);
        }
        const unsigned w01 = (r16 & 8) ? hb[1] : hb[0], w23 = (r16 & 8) ? hb[3] : hb[2];
        unsigned cnt = (((r16 & 1) ? w23 : w01) >> ((r16 & 6) * 4)) & 255u;
        cnt += __shfl_xor(cnt, 16);
        cnt += __shfl_xor(cnt, 32);
        const unsigned nb = __builtin_amdgcn_update_dpp(0u, cnt, 0x111, 0xf, 0xf, true) +
                            __builtin_amdgcn_update_dpp(0u, cnt, 0x101, 0xf, 0xf, true);
        const unsigned key = (cnt << 16) | (nb << 4) | (unsigned)r16;
        unsigned rank = 0;
#define RK(t) rank += __builtin_amdgcn_update_dpp(0u, key, 0x120 + t, 0xf, 0xf, false) > key ? 1u : 0u;
        RK(1) RK(2) RK(3) RK(4) RK(5) RK(6) RK(7) RK(8) RK(9) RK(10) RK(11) RK(12) RK(13) RK(14) RK(15)
#undef RK
        const unsigned code = rank < 3 ? rank : rank < 6 ? (3u | ((rank - 3) << 2)) : rank < 10 ? (15u | ((rank - 6) << 4)) : 0x80u;
        const unsigned slot = rank < 3 ? rank : rank < 6 ? rank + 1 : rank + 2;
        *(volatile LAS unsigned char*)(tab + r16) = (unsigned char)code;
        *(volatile LAS unsigned char*)(tab + 16 + slot) = (unsigned char)(r16 << 3);
        CBAR();
        C = *(LAS u32x4*)tab;
      }
      DBG(10);
#pragma unroll 1
      for (int j = j0; j < NSTAT; j += 4) {
        unsigned d[16];
#pragma unroll
        for (int i = 0; i < 16; ++i) d[i] = nx[i >> 2][i & 3];
        if (j + 4 < NSTAT) load(j + 4);
        LAS unsigned char* blk = lds + BIG_OFF + j * 4096;
        unsigned meta = 0x80030000u | 64u;
        unsigned c1[4], a2[4], n2[4], a3 = 0, n3 = 0, badv = 0;
#pragma unroll
        for (int hf = 0; hf < 2; ++hf) {
          unsigned d2[8], d3[8];
          u32x2 w1[8], w2[8];
          c1[2 * hf] = 0;
          c1[2 * hf + 1] = 0;
#pragma unroll
          for (int i = 0; i < 8; ++i) {
            const unsigned v = d[hf * 8 + i];
            const unsigned e7 = (v >> 3) & 0x07070707u;
            const unsigned lo = __builtin_amdgcn_perm(C[1], C[0], e7);
            const unsigned hi = __builtin_amdgcn_perm(C[3], C[2], e7);
            const unsigned code = __builtin_amdgcn_perm(hi, lo, ((v >> 4) & 0x04040404u) | 0x03020100u);
            badv |= code;
            const unsigned p1 = __builtin_amdgcn_udot4(code & 0x03030303u, 0x40100401u, 0u, false);
            c1[2 * hf + (i >> 2)] |= p1 << (8 * (i & 3));
            d2[i] = (code >> 2) & 0x03030303u;
            d3[i] = (code >> 4) & 0x03030303u;
            const unsigned p2 = __builtin_amdgcn_udot4(d2[i], 0x40100401u, 0u, false);
            w1[i] = *(LAS u32x2*)(lds + LUT_OFF + p1 * 8);
            w2[i] = *(LAS u32x2*)(lds + LUT_OFF + p2 * 8);
          }
          __builtin_amdgcn_sched_barrier(0);
#pragma unroll
          for (int i = 0; i < 8; ++i) {
            const int q = 2 * hf + (i >> 2);
            if ((i & 3) == 0) { a2[q] = 0; n2[q] = 0; }
            a2[q] |= __builtin_amdgcn_udot4(d2[i], w1[i][0], 0u, false) << n2[q];
            n2[q] += w1[i][1];
            a3 |= __builtin_amdgcn_udot4(d3[i], w2[i][0], 0u, false) << (n3 & 31);
            n3 += w2[i][1];
          }
        }
        const unsigned mine = (n2[0] + n2[1] + n2[2] + n2[3]) | (n3 << 16);
        const unsigned inc = wave_scan(mine);
        const unsigned tot = __builtin_amdgcn_readlane(inc, 63);
        const unsigned t2 = tot & 0xffff, t3 = tot >> 16;
        const bool raw = __any((badv & 0x80808080u) != 0 || n3 > 32) || t2 > 4096 || t3 > 1024;
        if (!raw) {
          *(LAS u32x4*)(blk + TAIL + lane * 16) = u32x4{0u, 0u, 0u, 0u};
          CBAR();
          const unsigned exc = inc - mine;
          unsigned pos = exc & 0xffff;
#pragma unroll
          for (int q = 0; q < 4; ++q) {
            const u64 v = (u64)a2[q] << (pos & 31);
            LAS unsigned* o = (LAS unsigned*)(blk + TAIL + (pos >> 5) * 4);
            __hip_atomic_fetch_or(o, (unsigned)v, __ATOMIC_RELAXED, __HIP_MEMORY_SCOPE_WORKGROUP);
            __hip_atomic_fetch_or(o + 1, (unsigned)(v >> 32), __ATOMIC_RELAXED, __HIP_MEMORY_SCOPE_WORKGROUP);
            pos += n2[q];
          }
          const unsigned tw = ((t2 + 31) >> 5) + ((t3 + 31) >> 5);
          const unsigned cl = tw <= 80 ? 0u : tw <= 96 ? 1u : 2u;
          const unsigned nl = tail_lanes(cl);
          {
            const unsigned p3 = exc >> 16;
            const u64 v = (u64)a3 << (p3 & 31);
            LAS unsigned* o = (LAS unsigned*)(blk + TAIL + nl * 16 - 8 - (p3 >> 5) * 4);
            __hip_atomic_fetch_or(o + 1, (unsigned)v, __ATOMIC_RELAXED, __HIP_MEMORY_SCOPE_WORKGROUP);
            __hip_atomic_fetch_or(o, (unsigned)(v >> 32), __ATOMIC_RELAXED, __HIP_MEMORY_SCOPE_WORKGROUP);
          }
          u32x4 s0, s1;
#pragma unroll
          for (int i = 0; i < 4; ++i) {
            s0[i] = (d[2 * i] & SM_MASK) | (__builtin_amdgcn_alignbit(d[2 * i + 1], d[2 * i + 1], 28) & ~SM_MASK);
            s1[i] = (d[8 + 2 * i] & SM_MASK) | (__builtin_amdgcn_alignbit(d[9 + 2 * i], d[9 + 2 * i], 28) & ~SM_MASK);
          }
          *(LAS u32x4*)(blk + lane * 16) = s0;
          *(LAS u32x4*)(blk + 1024 + lane * 16) = s1;
          *(LAS u32x4*)(blk + 2048 + lane * 16) = u32x4{c1[0], c1[1], c1[2], c1[3]};
          meta = 0x80000000u | (cl << 16) | nl;
        } else {
#pragma unroll
          for (int pc = 0; pc < 4; ++pc)
            *(LAS u32x4*)(blk + pc * 1024 + lane * 16) = u32x4{d[pc * 4], d[pc * 4 + 1], d[pc * 4 + 2], d[pc * 4 + 3]};
        }
        CBAR();
        lds_st(lds, CTL_META + j * 4, meta);
        DBG(1 + (j - 2) / 4);
      }
      const unsigned char* xl = p.x + (size_t)r16 * KD + (NKC - 1) * 256 + g * 16;
#pragma unroll 1
      for (int t = wave - 1; t < NRG; t += 4) {
        const int slot = t < 6 ? NSTAT + t : t - 6;
        if (t >= 6) {
          while (!__all(lds_ld(lds, CTL_PROG + (lane & 3) * 4) > (unsigned)slot || (lane & 3) >= nsend)) __builtin_amdgcn_s_sleep(4);
          CBAR();
        }
#pragma unroll
        for (int pc = 0; pc < 4; ++pc)
          __builtin_amdgcn_global_load_lds((const GAS void*)(xl + (size_t)t * (16 * KD) + pc * 64),
                                           (LAS void*)(lds + BIG_OFF + slot * 4096 + pc * 1024), 16, 0, 0);
        wait_vm<0>();
        lds_st(lds, CTL_PRDY + t * 4, 1u);
      }
      DBG_FLUSH();
      return;
    }
    unsigned* cnt = (unsigned*)(my + CNT_OFF + (size_t)wg * 128) + wave;
    unsigned call = __hip_atomic_load(cnt, __ATOMIC_RELAXED, __HIP_MEMORY_SCOPE_AGENT);
    const unsigned peer = (unsigned)(p.rank + 1 + peer0 + wave - 5) & 7u;
    unsigned char* pw = p.ws[peer];
    {
      lds_st(lds, QUE_OFF + (wave - 5) * 256 + lane * 4, 0u);
      if (wave < 7) lds_st(lds, CTL_OFF + (wave - 5) * 256 + lane * 4, 0u);
      const int idx = (wave - 5) * 64 + lane;
      unsigned wts = 0, n = 0;
#pragma unroll
      for (int b = 0; b < 4; ++b) {
        const bool e = ((idx >> (2 * b)) & 3) == 3;
        wts |= e ? (1u << (n + 8 * b)) : 0u;
        n += e ? 2u : 0u;
      }
      *(LAS u32x2*)(lds + LUT_OFF + idx * 8) = u32x2{wts, n};
    }
    asm volatile("s_waitcnt lgkmcnt(0)" ::: "memory");
    __builtin_amdgcn_s_barrier();
    CBAR();
    __builtin_amdgcn_s_setprio(3);
    if (wave - 5 >= nsend) return;
    call = __builtin_amdgcn_readfirstlane(call) + 1;
    const unsigned fbase = (call - 1) * NKC;
    DBG(1);
    unsigned char* dst = pw + (size_t)(call & 1) * PAR_BYTES + (size_t)p.rank * SHARD_BYTES + lane * 16;
    unsigned char* flags = pw + FLAG_OFF + (size_t)(call & 1) * FLAG_PAR + (size_t)p.rank * (NRG / 2) * 128 + (lane & 3) * 8;
    u64 cls = 0;
    unsigned tprev = 0;
#pragma unroll 1
    for (int j = 0;; ++j) {
      unsigned nl = 64, t = 0, off = 0;
      int slot = j;
      if (j < NSTAT) {
        unsigned meta;
        while ((meta = __builtin_amdgcn_readfirstlane(lds_ld(lds, CTL_META + j * 4))) == 0) __builtin_amdgcn_s_sleep(1);
        off = blk_off(rg0 + (j & 1), j >> 1);
        nl = meta & 0xff;
        cls |= (u64)((meta >> 16) & 3u) << ((j >> 1) * 2 + (j & 1) * 32);
      } else {
        while ((t = __builtin_amdgcn_readfirstlane(lds_ld(lds, QUE_OFF + (wave - 5) * 256 + (j - NSTAT) * 4))) == 0)
          __builtin_amdgcn_s_sleep(1);
        t -= 1;
        if (t < NRG) {
          while (lds_ld(lds, CTL_PRDY + t * 4) == 0) __builtin_amdgcn_s_sleep(1);
          off = blk_off(t, NKC - 1);
          slot = t < 6 ? NSTAT + t : t - 6;
        }
      }
      CBAR();
      if (t < NRG) {
        const LAS u32x4* sb = (const LAS u32x4*)(lds + BIG_OFF + slot * 4096 + lane * 16);
        const u32x4 v0 = sb[0], v1 = sb[64], v2 = sb[128], v3 = sb[192];
        CBAR();
        lds_st(lds, CTL_PROG + (wave - 5) * 4, j + 1);
        unsigned char* d = dst + off;
        st16_sys(d, v0);
        wait_vm<4>();
        st16_sys(d + 1024, v1);
        wait_vm<4>();
        st16_sys(d + 2048, v2);
        wait_vm<4>();
        if ((unsigned)lane < nl) st16_sys(d + 3072, v3);
        wait_vm<4>();
      } else {
        wait_vm<0>();
      }
      if (j <= NSTAT) {
        if (j >= 2 && (!(j & 1) || j == NSTAT - 1)) {
          u32x2 fv = *(const LAS u32x2*)(lds + TAB_OFF + 128 + (lane & 3) * 8);
          if (lane < 2) fv = u32x2{fbase + ((j + 1 - lane) >> 1), (unsigned)(cls >> (lane * 32))};
          if (lane < 4) st8_sys(flags + pair * 128, fv);
        }
      } else if (lane < 4) {
        st8_sys(flags + (tprev >> 1) * 128 + 64 + (tprev & 1) * 32, u32x2{call, call});
      }
      tprev = t;
      if (t >= NRG) {
        break;
      }
      DBG(2 + (j >> 1 < 15 ? j >> 1 : 15));
    }
    if (lane == 0) __hip_atomic_store(cnt, call, __ATOMIC_RELAXED, __HIP_MEMORY_SCOPE_AGENT);
    DBG(18);
    DBG_FLUSH();
    return;
  }

  const int cb = (wg >> 3) & 3;
  const bool local = wave == 9;
  const int src = local ? p.rank : (p.rank + grp) & 7;

  if (wave == 4) {
    lds_st(lds, CTL_OFF + lane * 4, (lane >= (grp > 4 ? 12 : 13) && lane < 16) ? 1000u : 0u);
    asm volatile("s_waitcnt lgkmcnt(0)" ::: "memory");
    __builtin_amdgcn_s_barrier();
    const int kk = lane >> 3, c = lane & 7;
    const unsigned char* wb = p.w + (size_t)kk * NL + cb * 128;
    const int f0 = (kk >> 1) & 3;
    const unsigned char* s0 = wb + ((c ^ f0) << 4);
    const unsigned char* s1 = wb + ((c ^ (f0 | 4)) << 4);
    int slot = 0;
#pragma unroll 1
    for (int s = 0; s < 32; ++s) {
      if (s >= 6) {
        const unsigned need = s - 5;
        bool first = true;
        while (true) {
          unsigned d = lds_ld(lds, CTL_DONE + (lane & 7) * 4);
          if (__all(d >= need)) break;
          if (first) {
            wait_vm<0>();
            lds_st(lds, CTL_FULL, s);
            first = false;
          }
          __builtin_amdgcn_s_sleep(2);
        }
        CBAR();
      }
      LAS unsigned char* dst = lds + RING_OFF + slot * SLOT_BYTES;
      slot = slot == 5 ? 0 : slot + 1;
      const size_t koff = (size_t)s * 128 * NL;
#pragma unroll
      for (int j = 0; j < 16; ++j) {
        const unsigned char* gp = (((j >> 1) & 1) ? s1 : s0) + koff + (size_t)j * 8 * NL;
        __builtin_amdgcn_global_load_lds((const GAS void*)gp, (LAS void*)(dst + j * 1024), 16, 0, 0);
      }
      wait_vm<16>();
      lds_st(lds, CTL_FULL, s);
    }
    wait_vm<0>();
    lds_st(lds, CTL_FULL, 32);
    return;
  }

  if (!local && wave > 4) {
    const int gw = wave - 5;
    unsigned* cnt = (unsigned*)(my + CNT_OFF + (size_t)wg * 128) + wave;
    unsigned call = __hip_atomic_load(cnt, __ATOMIC_RELAXED, __HIP_MEMORY_SCOPE_AGENT);
    __builtin_amdgcn_s_barrier();
    CBAR();
    call = __builtin_amdgcn_readfirstlane(call) + 1;
    if (lane == 0) __hip_atomic_store(cnt, call, __ATOMIC_RELAXED, __HIP_MEMORY_SCOPE_AGENT);
    const unsigned fbase = (call - 1) * NKC;
    const int rg = (wg & 7) * 4 + gw;
    const unsigned char* ap = my + (size_t)(call & 1) * PAR_BYTES + (size_t)src * SHARD_BYTES + lane * 16;
    const unsigned char* line = my + FLAG_OFF + (size_t)(call & 1) * FLAG_PAR + ((size_t)src * (NRG / 2) + (rg >> 1)) * 128;
    const u64* pollp = (const u64*)(lane == 0 ? line + (rg & 1) * 8 : lane < 3 ? line + 8 + lane * 8
                                    : lane == 4 ? line + 64 + (rg & 1) * 32 : my + TMO_OFF);
    unsigned* tmo = (unsigned*)(my + TMO_OFF);
    LAS unsigned char* ctl = lds + CTL_A + gw * 32;
    u64 pv = __hip_atomic_load(pollp, __ATOMIC_RELAXED, __HIP_MEMORY_SCOPE_SYSTEM);
    int slot = 0;
    unsigned swap = 0;
#pragma unroll 1
    for (int turn = 0; turn < NKC; ++turn) {
      bool bad = false;
      const unsigned long long t0 = __builtin_amdgcn_s_memrealtime();
      int kc = turn ^ (int)swap;
      while (true) {
        const bool last = (int)(__builtin_amdgcn_readlane((unsigned)pv, 4) - call) >= 0;
        if (kc < NKC - 1 ? (int)(__builtin_amdgcn_readlane((unsigned)pv, 0) - fbase) > kc : last) break;
        if (turn == NKC - 2 && last) {
          swap = 1;
          kc = NKC - 1;
          break;
        }
        if (__builtin_amdgcn_readlane((unsigned)pv, 3) == call || __builtin_amdgcn_s_memrealtime() - t0 > DEADLINE_TICKS) {
          bad = true;
          break;
        }
        pv = __hip_atomic_load(pollp, __ATOMIC_RELAXED, __HIP_MEMORY_SCOPE_SYSTEM);
      }
      CBAR();
      if (turn == 15) DBG(17);
      if (!bad) {
        const unsigned cls = __builtin_amdgcn_readlane((unsigned)(pv >> 32), 0) | 0xc0000000u;
        const unsigned nl = tail_lanes((cls >> (2 * kc)) & 3u);
        while (lds_ld(ctl, 24) + 3 <= (unsigned)turn) __builtin_amdgcn_s_sleep(1);
        CBAR();
        LAS unsigned char* dst = lds + ABUF_OFF + (gw * 3 + slot) * 4096;
        slot = slot == 2 ? 0 : slot + 1;
        const unsigned char* a = ap + blk_off(rg, kc);
        __builtin_amdgcn_global_load_lds((const GAS void*)(a + 2048), (LAS void*)(dst + 2048), 16, 0, 17);
        if ((unsigned)lane < nl) __builtin_amdgcn_global_load_lds((const GAS void*)(a + 3072), (LAS void*)(dst + 3072), 16, 0, 17);
        __builtin_amdgcn_global_load_lds((const GAS void*)(a), (LAS void*)(dst), 16, 0, 17);
        __builtin_amdgcn_global_load_lds((const GAS void*)(a + 1024), (LAS void*)(dst + 1024), 16, 0, 17);
        pv = __hip_atomic_load(pollp, __ATOMIC_RELAXED, __HIP_MEMORY_SCOPE_SYSTEM);
        wait_vm<0>();
        bad = __builtin_amdgcn_readlane((unsigned)pv, 3) == call;
        if (lane == 1 || lane == 2) *(volatile LAS u64*)(ctl + lane * 8) = pv;
        CBAR();
        if (!bad && lane == 0) *(volatile LAS u64*)(ctl) = ((u64)cls << 32) | (unsigned)(2 * turn + 2 + swap);
      }
      if (bad) {
        if (lane == 0) {
          __hip_atomic_store(tmo, call, __ATOMIC_RELAXED, __HIP_MEMORY_SCOPE_AGENT);
          *(volatile LAS u64*)(ctl) = 0xffffffff80000000ull;
        }
        break;
      }
      DBG(1 + turn);
    }
    DBG_FLUSH();
    return;
  }

  const int gw = local ? 4 : wave;
  const float sc = p.sx[0] * p.sw[0];
  const int rg = (wg & 7) * 4 + (local ? grp - 1 : gw);
  const int q = r16 >> 1;
  const int wl = RING_OFF + (g * 16 + q) * 128 + ((((q >> 1) & 3) | ((g & 1) << 2)) << 4) + (r16 & 1) * 8;

  f32x4 acc[8];
#pragma unroll
  for (int i = 0; i < 8; ++i) acc[i] = f32x4{0.f, 0.f, 0.f, 0.f};

  i32x8 wa[8];
  auto read_w = [&](LAS unsigned char* sl, int nt) {
    LAS unsigned char* a = sl + (wl ^ (nt << 4));
    i32x2 t0 = __builtin_amdgcn_ds_read_tr8_b64_v2i32((LAS i32x2*)(a));
    i32x2 t1 = __builtin_amdgcn_ds_read_tr8_b64_v2i32((LAS i32x2*)(a + 1024));
    i32x2 t2 = __builtin_amdgcn_ds_read_tr8_b64_v2i32((LAS i32x2*)(a + 8192));
    i32x2 t3 = __builtin_amdgcn_ds_read_tr8_b64_v2i32((LAS i32x2*)(a + 9216));
    wa[nt][0] = t0[0]; wa[nt][1] = t0[1]; wa[nt][2] = t1[0]; wa[nt][3] = t1[1];
    wa[nt][4] = t2[0]; wa[nt][5] = t2[1]; wa[nt][6] = t3[0]; wa[nt][7] = t3[1];
  };
  auto mma = [&](int turn, LAS unsigned char* sl, const unsigned* xv) {
    __builtin_amdgcn_sched_barrier(0);
    i32x8 xb;
#pragma unroll
    for (int i = 0; i < 8; ++i) xb[i] = xv[i];
#pragma unroll
    for (int nt = 0; nt < 8; ++nt) {
      acc[nt] = __builtin_amdgcn_mfma_scale_f32_16x16x128_f8f6f4(wa[nt], xb, acc[nt], 1, 0, 0, 0, 0, 0);
      __builtin_amdgcn_sched_barrier(0);
      read_w(sl + SLOT_BYTES, nt);
      __builtin_amdgcn_sched_barrier(0);
    }
    CBAR();
    lds_st(lds, CTL_DONE + gw * 4, turn * 2 + 2);
#pragma unroll
    for (int i = 0; i < 8; ++i) xb[i] = xv[8 + i];
#pragma unroll
    for (int nt = 0; nt < 8; ++nt)
      acc[nt] = __builtin_amdgcn_mfma_scale_f32_16x16x128_f8f6f4(wa[nt], xb, acc[nt], 1, 0, 0, 0, 0, 0);
  };

  bool bad = false;
  if (local) {
    const unsigned char* xs = p.x + (size_t)(rg * 16 + r16) * KD + g * 16;
    u32x4 n0[4];
#pragma unroll
    for (int pc = 0; pc < 4; ++pc) n0[pc] = *(const u32x4*)(xs + pc * 64);
    __builtin_amdgcn_s_barrier();
    CBAR();
    int slot = 0;
#pragma unroll 1
    for (int kc = 0; kc < NKC; ++kc) {
      unsigned xv[16];
#pragma unroll
      for (int i = 0; i < 16; ++i) xv[i] = n0[i >> 2][i & 3];
      const int kn = kc + 1 < NKC ? kc + 1 : NKC - 1;
#pragma unroll
      for (int pc = 0; pc < 4; ++pc) n0[pc] = *(const u32x4*)(xs + kn * 256 + pc * 64);
      while (lds_ld(lds, CTL_FULL) <= (unsigned)(kc * 2 + 1)) __builtin_amdgcn_s_sleep(1);
      CBAR();
      LAS unsigned char* sl = lds + slot * (2 * SLOT_BYTES);
#pragma unroll
      for (int nt = 0; nt < 8; ++nt) read_w(sl, nt);
      mma(kc, sl, xv);
      slot = slot == 2 ? 0 : slot + 1;
      DBG(2 + kc);
    }
  } else {
    {
      const int idx = wave * 64 + lane;
      unsigned sel = 0, n = 0;
#pragma unroll
      for (int b = 0; b < 4; ++b) {
        const unsigned dg = (idx >> (2 * b)) & 3;
        sel |= (dg == 3 ? 4u + n : dg) << (8 * b);
        n += dg == 3 ? 1u : 0u;
      }
      *(LAS u32x2*)(lds + LUT_OFF + idx * 8) = u32x2{sel, n};
    }
    asm volatile("s_waitcnt lgkmcnt(0)" ::: "memory");
    __builtin_amdgcn_s_barrier();
    CBAR();
    LAS unsigned char* ctl = lds + CTL_A + gw * 32;
    const unsigned e3 = SCR_OFF + gw * SCR_BYTES, e2 = e3 + 512;
    int slot = 0;
    unsigned swap = 0;
#define SB() __builtin_amdgcn_sched_barrier(0)
#pragma unroll 1
    for (int turn = 0; turn < NKC; ++turn) {
      int kc = turn ^ (int)swap;
      LAS unsigned char* sl = lds + (swap ? 2 : slot) * (2 * SLOT_BYTES);
      while (lds_ld(lds, CTL_FULL) <= (unsigned)(kc * 2 + 1)) __builtin_amdgcn_s_sleep(1);
      CBAR();
#pragma unroll
      for (int nt = 0; nt < 4; ++nt) read_w(sl, nt);
      u64 st;
      while ((unsigned)(st = *(volatile LAS u64*)ctl) < (unsigned)(2 * turn + 2)) __builtin_amdgcn_s_sleep(1);
      CBAR();
      if (turn == 15) DBG(1);
      if (turn == 8) DBG(20);
      const unsigned st0 = __builtin_amdgcn_readfirstlane((unsigned)st);
      bad = (int)st0 < 0;
      if (turn == NKC - 2 && (st0 & 1)) {
        swap = 1;
        kc = NKC - 1;
        sl = lds;
        while (lds_ld(lds, CTL_FULL) < 2 * NKC) __builtin_amdgcn_s_sleep(1);
        CBAR();
#pragma unroll
        for (int nt = 0; nt < 4; ++nt) read_w(sl, nt);
      }
      const unsigned cl = (__builtin_amdgcn_readfirstlane((unsigned)(st >> 32)) >> (2 * kc)) & 3u;
      LAS unsigned char* blk = lds + ABUF_OFF + (gw * 3 + slot) * 4096;
      unsigned xv[16];
      if (cl == 3) {
#pragma unroll
        for (int pc = 0; pc < 4; ++pc) {
          const u32x4 v = *(LAS u32x4*)(blk + pc * 1024 + lane * 16);
          xv[pc * 4] = v[0]; xv[pc * 4 + 1] = v[1]; xv[pc * 4 + 2] = v[2]; xv[pc * 4 + 3] = v[3];
        }
        CBAR();
        lds_st(ctl, 24, turn + 1);
#pragma unroll
        for (int nt = 4; nt < 8; ++nt) read_w(sl, nt);
      } else {
        const unsigned l3 = *(LAS unsigned short*)(blk + TAIL + tail_lanes(cl) * 16 - 4 - (lane >> 1) * 4 + (lane & 1) * 2);
        const unsigned T3 = *(LAS unsigned*)(ctl + 16);
        const unsigned l2a = *(LAS unsigned*)(blk + TAIL + lane * 4);
        const u32x4 c1 = *(LAS u32x4*)(blk + 2048 + lane * 16);
        const u32x2 T12 = *(LAS u32x2*)(ctl + 8);
        const u32x4 sm0 = *(LAS u32x4*)(blk + lane * 16), sm1 = *(LAS u32x4*)(blk + 1024 + lane * 16);
        SB();
        auto lut2 = [&](unsigned c, int i) { return *(LAS u32x2*)(lds + LUT_OFF + ((c >> (8 * i)) & 255u) * 8); };
        auto window = [&](unsigned off) {
          const LAS unsigned* w = (const LAS unsigned*)(lds + (off & ~3u));
          return u32x2{w[0], w[1]};
        };
        auto pick = [&](u32x2 w, unsigned off, unsigned T, unsigned sel) {
          return __builtin_amdgcn_perm(__builtin_amdgcn_alignbyte(w[1], w[0], off), T, sel);
        };
        {
          const unsigned t = l3 | (l3 << 12);
          const unsigned u = t & 0x000f000fu, v = (t >> 8) & 0x000f000fu;
          *(LAS u32x2*)(lds + e3 + lane * 8) = u32x2{__builtin_amdgcn_perm(0u, T3, (u | (u << 6)) & 0x03030303u),
                                                     __builtin_amdgcn_perm(0u, T3, (v | (v << 6)) & 0x03030303u)};
        }
        u32x2 s2[4], s1[16];
#pragma unroll
        for (int i = 0; i < 4; ++i) s2[i] = lut2(l2a, i);
        SB();
        const unsigned mine = (esc_count(c1[0]) + esc_count(c1[1]) + esc_count(c1[2]) + esc_count(c1[3])) | (esc_count(l2a) << 16);
#pragma unroll
        for (int i = 0; i < 8; ++i) s1[i] = lut2(c1[i >> 2], i & 3);
        const unsigned inc = wave_scan(mine);
        const unsigned exc = inc - mine;
        const unsigned tot = __builtin_amdgcn_readlane(inc, 63);
        CBAR();
        SB();
        unsigned o2[4], o1[16];
        u32x2 w2[4], w1[16];
        o2[0] = e3 + (exc >> 16);
#pragma unroll
        for (int i = 1; i < 4; ++i) o2[i] = o2[i - 1] + s2[i - 1][1];
#pragma unroll
        for (int i = 0; i < 4; ++i) w2[i] = window(o2[i]);
        SB();
#pragma unroll
        for (int i = 8; i < 16; ++i) s1[i] = lut2(c1[i >> 2], i & 3);
        SB();
        {
          u32x4 o;
#pragma unroll
          for (int i = 0; i < 4; ++i) o[i] = pick(w2[i], o2[i], T12[1], s2[i][0]);
          *(LAS u32x4*)(lds + e2 + lane * 16) = o;
          if ((tot & 0xffff) > 1024) {
            const unsigned l2b = *(LAS unsigned*)(blk + TAIL + 256 + lane * 4);
            const unsigned m2 = esc_count(l2b);
            unsigned off = e3 + (tot >> 16) + wave_scan(m2) - m2;
#pragma unroll
            for (int i = 0; i < 4; ++i) s2[i] = lut2(l2b, i);
#pragma unroll
            for (int i = 0; i < 4; ++i) { o[i] = pick(window(off), off, T12[1], s2[i][0]); off += s2[i][1]; }
            *(LAS u32x4*)(lds + e2 + 1024 + lane * 16) = o;
          }
        }
        CBAR();
        lds_st(ctl, 24, turn + 1);
        o1[0] = e2 + (exc & 0xffff);
#pragma unroll
        for (int i = 0; i < 16; ++i) {
          if (i) o1[i] = o1[i - 1] + s1[i - 1][1];
          w1[i] = window(o1[i]);
        }
        SB();
#pragma unroll
        for (int i = 0; i < 16; ++i) {
          const unsigned s = i < 8 ? sm0[i >> 1] : sm1[(i >> 1) & 3];
          xv[i] = (((i & 1) ? __builtin_amdgcn_alignbit(s, s, 4) : s) & SM_MASK) | pick(w1[i], o1[i], T12[0], s1[i][0]);
          if (i == 7) {
            SB();
#pragma unroll
            for (int nt = 4; nt < 8; ++nt) read_w(sl, nt);
            SB();
          }
        }
        CBAR();
      }
      mma(turn, sl, xv);
      slot = slot == 2 ? 0 : slot + 1;
      DBG(2 + turn);
    }
  }
  {
    const __amdgpu_buffer_rsrc_t ors =
        __builtin_amdgcn_make_buffer_rsrc((void*)p.out, 0, NR * MS * NL * 4, 0x00020000);
    const float s = bad ? __builtin_nanf("") : sc;
    const int ooff = ((src * MS + rg * 16 + (r16 & 7)) * NL + cb * 128 + (r16 >> 3) * 16 + g * 4) * 4;
#pragma unroll
    for (int t = 0; t < 4; ++t) {
      const u32x4 a = __builtin_bit_cast(u32x4, acc[2 * t] * s), b = __builtin_bit_cast(u32x4, acc[2 * t + 1] * s);
      u32x4 lo, hi;
#pragma unroll
      for (int i = 0; i < 4; ++i) {
        lo[i] = __builtin_amdgcn_update_dpp(a[i], b[i], 0x128, 0xf, 0xc, false);
        hi[i] = __builtin_amdgcn_update_dpp(b[i], a[i], 0x128, 0xf, 0x3, false);
      }
      __builtin_amdgcn_raw_buffer_store_b128(lo, ors, ooff + t * 128, 0, OUT_AUX);
      __builtin_amdgcn_raw_buffer_store_b128(hi, ors, ooff + t * 128 + 8 * NL * 4, 0, OUT_AUX);
    }
  }
}

}

extern "C" void kernel_launch(int rank, int nranks, ncclComm_t comm, hipStream_t stream,
                              void* const* d_in, const int* in_sizes, int n_in, void* d_out,
                              int out_size, const int* in_dtypes, int out_dtype,
                              void* const* d_out_all, void* const* d_ws_all, size_t ws_bytes) {
  (void)comm; (void)in_dtypes; (void)out_dtype; (void)d_out_all;
  if (nranks != NR || n_in != 4 || (unsigned)rank >= (unsigned)NR || ws_bytes < WS_NEED ||
      in_sizes[0] != MS * KD || in_sizes[1] != KD * NL || out_size != NR * MS * NL) {
    fprintf(stderr, "kernel_launch: unexpected problem shape\n");
    return;
  }
  Params p;
  p.x = (const unsigned char*)d_in[0];
  p.w = (const unsigned char*)d_in[1];
  p.sx = (const float*)d_in[2];
  p.sw = (const float*)d_in[3];
  p.out = (float*)d_out;
  for (int i = 0; i < NR; ++i) p.ws[i] = (unsigned char*)d_ws_all[i];
  p.rank = rank;
  p.pad = 0;
  hipLaunchKernelGGL(ag_gemm_kernel, dim3(NWG), dim3(NWAVE * 64), 0, stream, p);
  hipError_t e = hipGetLastError();
  if (e != hipSuccess) fprintf(stderr, "launch error %s\n", hipGetErrorString(e));
}
```

```cpp
#include <hip/hip_runtime.h>
#include <rccl/rccl.h>
#include <cstdio>
#include <cstdint>
#include <type_traits>

namespace {

constexpr int NR = 8;
constexpr int MS = 512;
constexpr int KD = 4096;
constexpr int NL = 512;
constexpr int NKC = 16;
constexpr int NRG = 32;
constexpr int NWG = 256;
constexpr int NWAVE = 12;
#define NPUSH 6
constexpr int NP = NPUSH;
constexpr int NPART = (NKC - NP + 3) / 4;
constexpr size_t SHARD_BYTES = (size_t)MS * KD;
constexpr size_t PAR_BYTES = NR * SHARD_BYTES;
constexpr size_t FLAG_OFF = 2 * PAR_BYTES;
constexpr size_t FLAG_PAR = (size_t)NR * (NRG / 2) * 128;
constexpr size_t CNT_OFF = FLAG_OFF + 2 * FLAG_PAR;
constexpr size_t TMO_OFF = CNT_OFF + (size_t)NWG * 128;
constexpr size_t DBG_OFF = TMO_OFF + 4096;
constexpr size_t WS_NEED = DBG_OFF + (1u << 19);
constexpr unsigned long long DEADLINE_TICKS = 50000000ull;

constexpr int LUT_OFF = 0;
constexpr int CTL_OFF = 2048;
constexpr int SCR_OFF = CTL_OFF + 512;
constexpr int SCR_BYTES = 512;
constexpr int TAB_OFF = SCR_OFF + 2048;
constexpr int ELUT_OFF = TAB_OFF + 1536;
constexpr int ETAIL_OFF = ELUT_OFF + 2048;
constexpr int BIG_OFF = 16384;
#define WCH 2
constexpr int NWS = 2 * WCH;
constexpr int NAS = 9 - 2 * WCH;
constexpr int SLOT_BYTES = 16384;
constexpr int RING_OFF = BIG_OFF;
constexpr int ABUF_OFF = RING_OFF + NWS * SLOT_BYTES;
constexpr int LDS_BYTES = ABUF_OFF + 4 * NAS * 4096;
static_assert(ETAIL_OFF + 4096 <= 14336 && NPART <= 3 && NP >= 4 && LDS_BYTES <= 163840, "LDS map");
constexpr int CTL_FULL = CTL_OFF;
constexpr int CTL_DONE = CTL_OFF + 32;
constexpr int CTL_A = CTL_OFF + 64;
constexpr int CTL_T = CTL_OFF + 192;
constexpr int CTL_E = CTL_OFF + 224;
constexpr int CTL_META = CTL_OFF;
constexpr int CTL_CALL = CTL_OFF + 128;
constexpr int TAIL = 3072;

typedef unsigned u32x4 __attribute__((ext_vector_type(4)));
typedef unsigned u32x3 __attribute__((ext_vector_type(3)));
typedef unsigned u32x2 __attribute__((ext_vector_type(2)));
typedef int i32x2 __attribute__((ext_vector_type(2)));
typedef int i32x8 __attribute__((ext_vector_type(8)));
typedef float f32x4 __attribute__((ext_vector_type(4)));
typedef unsigned long long u64;
#define LAS __attribute__((address_space(3)))
#define GAS __attribute__((address_space(1)))

struct Params {
  const unsigned char* x;
  unsigned char* my;
  const unsigned char* w;
  const float* sx;
  const float* sw;
  float* out;
  unsigned char* ws[NR];
  int rank;
  int pad;
};

__device__ __forceinline__ void st16_sys(unsigned char* p, u32x4 v) {
  asm volatile("global_store_dwordx4 %0, %1, off sc0 sc1\n\ts_nop 1" ::"v"(p), "v"(v) : "memory");
}
__device__ __forceinline__ void st8_sys(unsigned char* p, u32x2 v) {
  asm volatile("global_store_dwordx2 %0, %1, off sc0 sc1" ::"v"(p), "v"(v) : "memory");
}
template <int N>
__device__ __forceinline__ void wait_vm() {
  asm volatile("s_waitcnt vmcnt(%0)" ::"i"(N) : "memory");
}
__device__ __forceinline__ unsigned lds_ld(LAS unsigned char* l, int off) {
  return *(volatile LAS unsigned*)(l + off);
}
__device__ __forceinline__ void lds_st(LAS unsigned char* l, int off, unsigned v) {
  *(volatile LAS unsigned*)(l + off) = v;
}
#define CBAR() asm volatile("" ::: "memory")

__device__ __forceinline__ unsigned wave_scan(unsigned v) {
  v += __builtin_amdgcn_update_dpp(0u, v, 0x111, 0xf, 0xf, true);
  v += __builtin_amdgcn_update_dpp(0u, v, 0x112, 0xf, 0xf, true);
  v += __builtin_amdgcn_update_dpp(0u, v, 0x114, 0xf, 0xf, true);
  v += __builtin_amdgcn_update_dpp(0u, v, 0x118, 0xf, 0xf, true);
  v += __builtin_amdgcn_update_dpp(0u, v, 0x142, 0xa, 0xf, false);
  v += __builtin_amdgcn_update_dpp(0u, v, 0x143, 0xc, 0xf, false);
  return v;
}
__device__ __forceinline__ unsigned esc_count(unsigned c) {
  return __builtin_popcount(c & (c >> 1) & 0x55555555u);
}

#define DBG(slot) do {} while (0)
#define DBG_FLUSH() do {} while (0)

#define ENC_DELAY 4
#define HWIN 12
#define XMAP 0
#define RAW_END 2
#define UPRIO 2
#define GSLP 2
#define WWIN 16
#define TSUM 8
#define EBLK 0
#define EWIN 12
#define PWIN 6
#define QUIET 0
#define OUT_AUX 18
__device__ __forceinline__ unsigned blk_off(int rg, int kc) { return (unsigned)(rg * NKC + kc) * 4096u; }
#define ORD 0x1302
__device__ __forceinline__ int chunk_of(int turn) {
  const unsigned d = turn - (NP - 2);
  return d < 4 ? NP - 2 + ((ORD >> (4 * d)) & 3) : turn;
}
__device__ __forceinline__ int turn_of(int kc) {
  constexpr unsigned INV = (0u << (4 * (ORD & 3))) | (1u << (4 * ((ORD >> 4) & 3))) | (2u << (4 * ((ORD >> 8) & 3))) | (3u << (4 * ((ORD >> 12) & 3)));
  const unsigned d = kc - (NP - 2);
  return d < 4 ? NP - 2 + ((INV >> (4 * d)) & 3) : kc;
}
__device__ __forceinline__ unsigned tail_lanes(unsigned c) { return c == 0 ? 20u : c == 1 ? 24u : c == 2 ? 40u : 64u; }
constexpr unsigned SM_MASK = 0x87878787u;

__global__ void __launch_bounds__(NWAVE * 64, 3) ag_gemm_kernel(Params p) {
  __shared__ __attribute__((aligned(1024))) unsigned char smem[LDS_BYTES];
  LAS unsigned char* lds = (LAS unsigned char*)smem;
  const int wg = blockIdx.x;
  const int lane = threadIdx.x & 63;
  const int wave = __builtin_amdgcn_readfirstlane(threadIdx.x >> 6);
  const int grp = wg >> 5;
  unsigned char* my = p.my;
  const int g = lane >> 4, r16 = lane & 15;

  DBG(0);

  if (__builtin_expect(grp == 0 && wave == 0, 1)) {
    unsigned* cnt = (unsigned*)(my + CNT_OFF + (size_t)wg * 128);
    unsigned call;
    asm volatile("global_load_dword %0, %1, off sc1" : "=v"(call) : "v"(cnt) : "memory");
    const unsigned char* xs = p.x + (size_t)((wg & 15) * 32 + r16) * KD + g * 16;
#pragma unroll
    for (int i = 0; i < 8; ++i)
      __builtin_amdgcn_global_load_lds((const GAS void*)(xs + (size_t)(i >> 2) * (16 * KD) + (i & 3) * 64),
                                       (LAS void*)(lds + BIG_OFF + i * 1024), 16, 0, 0);
    __builtin_amdgcn_s_barrier();
    asm volatile("s_waitcnt vmcnt(4)" : "+v"(call)::"memory");
    call = __builtin_amdgcn_readfirstlane(call) + 1;
    lds_st(lds, CTL_CALL, call);
    lds_st(lds, CTL_META, 0x80030000u | 64u);
    wait_vm<0>();
    lds_st(lds, CTL_META + 4, 0x80030000u | 64u);
    DBG(1);
    if (lane == 0) __hip_atomic_store(cnt, call, __ATOMIC_RELAXED, __HIP_MEMORY_SCOPE_AGENT);
    DBG(18);
    DBG_FLUSH();
    return;
  }
  if (grp == 0 ? wave > 8 : (wave == 9 && grp > 4)) return;

  auto build_lut = [&](bool pack) {
    const int idx = (wave & 3) * 64 + lane;
    unsigned v = 0, n = 0;
#pragma unroll
    for (int b = 0; b < 4; ++b) {
      const unsigned dg = (idx >> (2 * b)) & 3;
      v |= (pack ? (dg == 3 ? 1u << (2 * n) : 0u) : (dg == 3 ? 4u + n : dg)) << (8 * b);
      n += dg == 3 ? 1u : 0u;
    }
    *(LAS u32x2*)(lds + (pack ? ELUT_OFF : LUT_OFF) + idx * 8) = u32x2{v, pack ? 2 * n : n};
    asm volatile("s_waitcnt lgkmcnt(0)" ::: "memory");
  };

  const int pkc = NP + (grp - 5) * 4 + wave - 5;
  if (grp == 0 ? wave < 5 : (grp > 4 && wave > 4 && wave < 9 && pkc < NKC)) {
    {
      const bool home = grp != 0;
      const int rg0 = home ? wg & 30 : (wg & 15) * 2;
      const int j0 = home ? pkc * 2 + (wg & 1) : 1 + wave, jend = home ? j0 + 1 : 2 * NP;
      unsigned* cnt = (unsigned*)(my + CNT_OFF + (size_t)wg * 128) + wave;
      const unsigned char* xs = p.x + (size_t)(rg0 * 16 + r16) * KD + g * 16;
      u32x4 nx[4];
      auto load = [&](int j) {
        const unsigned char* a = xs + (size_t)(j & 1) * (16 * KD) + (j >> 1) * 256;
#pragma unroll
        for (int pc = 0; pc < 4; ++pc) nx[pc] = *(const u32x4*)(a + pc * 64);
      };
      if (wave > 2) __builtin_amdgcn_s_sleep(ENC_DELAY);
      const u32x3 smp = *(const u32x3*)(p.x + (size_t)(rg0 * 16 + (lane & 31)) * KD + (lane >> 5) * 16);
      load(j0);
      unsigned call = __hip_atomic_load(cnt, __ATOMIC_RELAXED, __HIP_MEMORY_SCOPE_AGENT);
      CBAR();
      __builtin_amdgcn_sched_barrier(0);
      if (home) build_lut(false);
      __builtin_amdgcn_s_barrier();
      __builtin_amdgcn_sched_barrier(0);
      CBAR();
      u32x4 C;
      {
        LAS unsigned char* tab = lds + TAB_OFF + wave * 128;
        u64 h = 0;
#pragma unroll
        for (int i = 0; i < 3; ++i) {
#pragma unroll
          for (int b = 0; b < 4; ++b) h += 1ull << (((smp[i] >> (8 * b + 3)) & 15u) * 4u);
        }
        u32x4 hb = {(unsigned)h & 0x0f0f0f0fu, (unsigned)(h >> 32) & 0x0f0f0f0fu,
                    ((unsigned)h >> 4) & 0x0f0f0f0fu, (unsigned)(h >> 36) & 0x0f0f0f0fu};
#pragma unroll
        for (int i = 0; i < 4; ++i) {
          hb[i] += __builtin_amdgcn_update_dpp(0u, hb[i], 0x121, 0xf, 0xf, false);
          hb[i] += __builtin_amdgcn_update_dpp(0u, hb[i], 0x122, 0xf, 0xf, false);
          hb[i] += __builtin_amdgcn_update_dpp(0u, hb[i], 0x124, 0xf, 0xf, false);
          hb[i] += __builtin_amdgcn_update_dpp(0u, hb[i], 0x128, 0xf, 0xf, false);
        }
        const unsigned w01 = (r16 & 8) ? hb[1] : hb[0], w23 = (r16 & 8) ? hb[3] : hb[2];
        unsigned cnt = (((r16 & 1) ? w23 : w01) >> ((r16 & 6) * 4)) & 255u;
        cnt += __shfl_xor(cnt, 16);
        cnt += __shfl_xor(cnt, 32);
        const unsigned nb = __builtin_amdgcn_update_dpp(0u, cnt, 0x111, 0xf, 0xf, true) +
                            __builtin_amdgcn_update_dpp(0u, cnt, 0x101, 0xf, 0xf, true);
        const unsigned key = (cnt << 16) | (nb << 4) | (unsigned)r16;
        unsigned rank = 0;
#define RK(t) rank += __builtin_amdgcn_update_dpp(0u, key, 0x120 + t, 0xf, 0xf, false) > key ? 1u : 0u;
        RK(1) RK(2) RK(3) RK(4) RK(5) RK(6) RK(7) RK(8) RK(9) RK(10) RK(11) RK(12) RK(13) RK(14) RK(15)
#undef RK
        const unsigned code = rank < 3 ? rank : rank < 6 ? (3u | ((rank - 3) << 2)) : rank < 10 ? (15u | ((rank - 6) << 4)) : 0x80u;
        const unsigned slot = rank < 3 ? rank : rank < 6 ? rank + 1 : rank + 2;
        *(volatile LAS unsigned char*)(tab + r16) = (unsigned char)code;
        *(volatile LAS unsigned char*)(tab + 16 + slot) = (unsigned char)(r16 << 3);
        CBAR();
        C = *(LAS u32x4*)tab;
      }
      if (!home) DBG(10);
#pragma unroll 1
      for (int j = j0; j < jend; j += 4) {
        unsigned d[16];
#pragma unroll
        for (int i = 0; i < 16; ++i) d[i] = nx[i >> 2][i & 3];
        if (j + 4 < jend) load(j + 4);
        LAS unsigned char* blk = lds + (home ? ETAIL_OFF + (wave - 5) * 1024 - TAIL : BIG_OFF + j * 4096);
        unsigned meta = 0x80030000u | 64u;
        u32x4 o0, o1, o2;
        unsigned c1[4], a2[4], n2[4], a3 = 0, n3 = 0, badv = 0;
#pragma unroll
        for (int hf = 0; hf < 2; ++hf) {
          unsigned d2[8], d3[8];
          u32x2 w1[8], w2[8];
          c1[2 * hf] = 0;
          c1[2 * hf + 1] = 0;
#pragma unroll
          for (int i = 0; i < 8; ++i) {
            const unsigned v = d[hf * 8 + i];
            const unsigned e7 = (v >> 3) & 0x07070707u;
            const unsigned lo = __builtin_amdgcn_perm(C[1], C[0], e7);
            const unsigned hi = __builtin_amdgcn_perm(C[3], C[2], e7);
            const unsigned code = __builtin_amdgcn_perm(hi, lo, ((v >> 4) & 0x04040404u) | 0x03020100u);
            badv |= code;
            const unsigned p1 = __builtin_amdgcn_udot4(code & 0x03030303u, 0x40100401u, 0u, false);
            c1[2 * hf + (i >> 2)] |= p1 << (8 * (i & 3));
            d2[i] = (code >> 2) & 0x03030303u;
            d3[i] = (code >> 4) & 0x03030303u;
            const unsigned p2 = __builtin_amdgcn_udot4(d2[i], 0x40100401u, 0u, false);
            w1[i] = *(LAS u32x2*)(lds + ELUT_OFF + p1 * 8);
            w2[i] = *(LAS u32x2*)(lds + ELUT_OFF + p2 * 8);
          }
          __builtin_amdgcn_sched_barrier(0);
#pragma unroll
          for (int i = 0; i < 8; ++i) {
            const int q = 2 * hf + (i >> 2);
            if ((i & 3) == 0) { a2[q] = 0; n2[q] = 0; }
            a2[q] |= __builtin_amdgcn_udot4(d2[i], w1[i][0], 0u, false) << n2[q];
            n2[q] += w1[i][1];
            a3 |= __builtin_amdgcn_udot4(d3[i], w2[i][0], 0u, false) << (n3 & 31);
            n3 += w2[i][1];
          }
          asm volatile("" : "+v"(a2[2 * hf]), "+v"(a2[2 * hf + 1]), "+v"(a3), "+v"(c1[2 * hf]), "+v"(c1[2 * hf + 1]));
        }
        const unsigned mine = (n2[0] + n2[1] + n2[2] + n2[3]) | (n3 << 16);
        const unsigned inc = wave_scan(mine);
        const unsigned tot = __builtin_amdgcn_readlane(inc, 63);
        const unsigned t2 = tot & 0xffff, t3 = tot >> 16;
        const bool raw = __any((badv & 0x80808080u) != 0 || n3 > 32) || t2 > 4096 || t3 > 1024 || j >= 2 * NKC - RAW_END;
        if (!raw) {
          *(LAS u32x4*)(blk + TAIL + lane * 16) = u32x4{0u, 0u, 0u, 0u};
          CBAR();
          const unsigned exc = inc - mine;
          unsigned pos = exc & 0xffff;
#pragma unroll
          for (int q = 0; q < 4; ++q) {
            const u64 v = (u64)a2[q] << (pos & 31);
            LAS unsigned* o = (LAS unsigned*)(blk + TAIL + (pos >> 5) * 4);
            __hip_atomic_fetch_or(o, (unsigned)v, __ATOMIC_RELAXED, __HIP_MEMORY_SCOPE_WORKGROUP);
            __hip_atomic_fetch_or(o + 1, (unsigned)(v >> 32), __ATOMIC_RELAXED, __HIP_MEMORY_SCOPE_WORKGROUP);
            pos += n2[q];
          }
          const unsigned tw = ((t2 + 31) >> 5) + ((t3 + 31) >> 5);
          const unsigned cl = tw <= 80 ? 0u : tw <= 96 ? 1u : 2u;
          const unsigned nl = tail_lanes(cl);
          {
            const unsigned p3 = exc >> 16;
            const u64 v = (u64)a3 << (p3 & 31);
            LAS unsigned* o = (LAS unsigned*)(blk + TAIL + nl * 16 - 8 - (p3 >> 5) * 4);
            __hip_atomic_fetch_or(o + 1, (unsigned)v, __ATOMIC_RELAXED, __HIP_MEMORY_SCOPE_WORKGROUP);
            __hip_atomic_fetch_or(o, (unsigned)(v >> 32), __ATOMIC_RELAXED, __HIP_MEMORY_SCOPE_WORKGROUP);
          }
#pragma unroll
          for (int i = 0; i < 4; ++i) {
            o0[i] = (d[2 * i] & SM_MASK) | (__builtin_amdgcn_alignbit(d[2 * i + 1], d[2 * i + 1], 28) & ~SM_MASK);
            o1[i] = (d[8 + 2 * i] & SM_MASK) | (__builtin_amdgcn_alignbit(d[9 + 2 * i], d[9 + 2 * i], 28) & ~SM_MASK);
          }
          o2 = u32x4{c1[0], c1[1], c1[2], c1[3]};
          meta = 0x80000000u | (cl << 16) | nl;
        } else {
          o0 = u32x4{d[0], d[1], d[2], d[3]};
          o1 = u32x4{d[4], d[5], d[6], d[7]};
          o2 = u32x4{d[8], d[9], d[10], d[11]};
          *(LAS u32x4*)(blk + TAIL + lane * 16) = u32x4{d[12], d[13], d[14], d[15]};
        }
        if (!home) {
          *(LAS u32x4*)(blk + lane * 16) = o0;
          *(LAS u32x4*)(blk + 1024 + lane * 16) = o1;
          *(LAS u32x4*)(blk + 2048 + lane * 16) = o2;
          CBAR();
          lds_st(lds, CTL_META + j * 4, meta);
          DBG(1 + (j - 2) / 4);
        } else {
          CBAR();
          const u32x4 o3 = *(const LAS u32x4*)(blk + TAIL + lane * 16);
          call = __builtin_amdgcn_readfirstlane(call) + 1;
          unsigned char* dst = my + (size_t)(call & 1) * PAR_BYTES + (size_t)p.rank * SHARD_BYTES + blk_off(wg & 31, pkc) + lane * 16;
          st16_sys(dst, o0);
          st16_sys(dst + 1024, o1);
          st16_sys(dst + 2048, o2);
          if ((unsigned)lane < (meta & 0xff)) st16_sys(dst + 3072, o3);
          if (lane == 0) __hip_atomic_store(cnt, call, __ATOMIC_RELAXED, __HIP_MEMORY_SCOPE_AGENT);
          unsigned char* pf = p.ws[lane & 7] + FLAG_OFF + (size_t)(call & 1) * FLAG_PAR +
                              ((size_t)p.rank * (NRG / 2) + ((wg & 31) >> 1)) * 128 + 64 + (wg & 1) * 32 + (grp - 5) * 8;
          wait_vm<0>();
          unsigned prev = 0;
          if (lane == 0) {
            __hip_atomic_fetch_or((LAS unsigned*)(lds + CTL_E + 4), ((meta >> 16) & 3u) << (2 * pkc), __ATOMIC_RELAXED, __HIP_MEMORY_SCOPE_WORKGROUP);
            prev = __hip_atomic_fetch_add((LAS unsigned*)(lds + CTL_E), 1u, __ATOMIC_RELAXED, __HIP_MEMORY_SCOPE_WORKGROUP);
          }
          const int left = NKC - NP - (grp - 5) * 4;
          if ((int)__builtin_amdgcn_readfirstlane(prev) + 1 == (left < 4 ? left : 4)) {
            const unsigned cls = lds_ld(lds, CTL_E + 4);
            if (lane < NR && lane != p.rank) st8_sys(pf, u32x2{call, cls});
          }
          DBG(22);
        }
      }
      if (!home) {
        DBG_FLUSH();
        return;
      }
    }
  } else if (grp != 0 && wave > 4 && wave < 9) {
    build_lut(false);
    __builtin_amdgcn_s_barrier();
  }

  if (__builtin_expect(grp == 0, 1)) {
    const int pair = wg & 15, rg0 = pair * 2;
    const unsigned peer = (unsigned)(p.rank + 1 + (wg >> 4) * 4 + wave - 5) & 7u;
    unsigned char* pw = p.ws[peer];
    if (wave == 5) lds_st(lds, CTL_OFF + lane * 4, 0u);
    build_lut(true);
    __builtin_amdgcn_s_barrier();
    CBAR();
    if (peer == (unsigned)p.rank) return;
    __builtin_amdgcn_s_setprio(3);
    unsigned call;
    while ((call = __builtin_amdgcn_readfirstlane(lds_ld(lds, CTL_CALL))) == 0) {}
    const unsigned fbase = (call - 1) * NKC;
    DBG(1);
    unsigned char* dst = pw + (size_t)(call & 1) * PAR_BYTES + (size_t)p.rank * SHARD_BYTES + lane * 16;
    const size_t foff = FLAG_OFF + (size_t)(call & 1) * FLAG_PAR + ((size_t)p.rank * (NRG / 2) + pair) * 128;
    unsigned char* flag = pw + foff + (lane & 3) * 8;
    u64 cls = 0;
    const int jend = 2 * NP;
#pragma unroll 1
    for (int j = 0; j <= jend; ++j) {
      if (j < jend) {
        unsigned meta;
        while ((meta = __builtin_amdgcn_readfirstlane(lds_ld(lds, CTL_META + j * 4))) == 0) __builtin_amdgcn_s_sleep(1);
        CBAR();
        cls |= (u64)((meta >> 16) & 3u) << ((j >> 1) * 2 + (j & 1) * 32);
        const LAS u32x4* sb = (const LAS u32x4*)(lds + BIG_OFF + j * 4096 + lane * 16);
        const u32x4 v0 = sb[0], v1 = sb[64], v2 = sb[128], v3 = sb[192];
        unsigned char* d = dst + blk_off(rg0 + (j & 1), j >> 1);
        st16_sys(d, v0);
        wait_vm<4>();
        st16_sys(d + 1024, v1);
        wait_vm<4>();
        st16_sys(d + 2048, v2);
        wait_vm<4>();
        if ((unsigned)lane < (meta & 0xff)) st16_sys(d + 3072, v3);
        wait_vm<4>();
      } else {
        wait_vm<0>();
      }
      if (j >= 2 && !(j & 1)) {
        u32x2 fv = *(const LAS u32x2*)(lds + TAB_OFF + 128 + (lane & 3) * 8);
        if (lane < 2) fv = u32x2{fbase + (j >> 1), (unsigned)(cls >> (lane * 32))};
        if (lane < 4) st8_sys(flag, fv);
      }
      DBG(2 + (j >> 1 < 15 ? j >> 1 : 15));
    }
    DBG(18);
    DBG_FLUSH();
    return;
  }

  const int cb = (wg >> 3) & 3;
  const bool local = wave == 9;
  const int src = local ? p.rank : (p.rank + grp) & 7;

  if (wave == 4) {
    lds_st(lds, CTL_OFF + lane * 4, (lane >= (grp > 4 ? 12 : 13) && lane < 16) ? 1000u : 0u);
    asm volatile("s_waitcnt lgkmcnt(0)" ::: "memory");
    __builtin_amdgcn_s_barrier();
    const int kk = lane >> 3, c = lane & 7;
    const unsigned char* wb = p.w + (size_t)kk * NL + cb * 128;
    const int f0 = (kk >> 1) & 3;
    const unsigned char* s0 = wb + ((c ^ f0) << 4);
    const unsigned char* s1 = wb + ((c ^ (f0 | 4)) << 4);
    int slot = 0, full = 0;
#pragma unroll 1
    for (int s = 0; s < 32; ++s) {
      if (s >= NWS) {
        const unsigned need = s - (NWS - 1);
        bool first = true;
        while (true) {
          unsigned d = lds_ld(lds, CTL_DONE + (lane & 7) * 4);
          if (__all(d >= need)) break;
          if (first) {
            if (WWIN > 16) {
              wait_vm<16>();
              lds_st(lds, CTL_FULL, full > s - 1 ? full : s - 1);
            }
            wait_vm<0>();
            full = s;
            lds_st(lds, CTL_FULL, s);
            first = false;
          }
          __builtin_amdgcn_s_sleep(2);
        }
        CBAR();
      }
      LAS unsigned char* dst = lds + RING_OFF + slot * SLOT_BYTES;
      slot = slot == NWS - 1 ? 0 : slot + 1;
      const size_t koff = (size_t)(chunk_of(s >> 1) * 2 + (s & 1)) * 128 * NL;
#pragma unroll
      for (int j = 0; j < 16; ++j) {
        const unsigned char* gp = (((j >> 1) & 1) ? s1 : s0) + koff + (size_t)j * 8 * NL;
        __builtin_amdgcn_global_load_lds((const GAS void*)gp, (LAS void*)(dst + j * 1024), 16, 0, 0);
      }
      wait_vm<WWIN>();
      full = full > s + 1 - WWIN / 16 ? full : s + 1 - WWIN / 16;
      lds_st(lds, CTL_FULL, full);
    }
    wait_vm<0>();
    lds_st(lds, CTL_FULL, 32);
    return;
  }

  if (wave > 9) {
    unsigned* cnt = (unsigned*)(my + CNT_OFF + (size_t)wg * 128) + wave;
    unsigned call = __hip_atomic_load(cnt, __ATOMIC_RELAXED, __HIP_MEMORY_SCOPE_AGENT);
    __builtin_amdgcn_s_barrier();
    CBAR();
    call = __builtin_amdgcn_readfirstlane(call) + 1;
    if (lane == 0) __hip_atomic_store(cnt, call, __ATOMIC_RELAXED, __HIP_MEMORY_SCOPE_AGENT);
    const unsigned fbase = (call - 1) * NKC;
    const int rg0 = (wg & 7) * 4;
    const size_t aoff = (size_t)(call & 1) * PAR_BYTES + (size_t)src * SHARD_BYTES + lane * 16;
    const unsigned char* lines = my + FLAG_OFF + (size_t)(call & 1) * FLAG_PAR + (size_t)src * (NRG / 2) * 128;
    LAS unsigned char* ctl = lds + CTL_A;
    auto slot_busy = [&](int r, int kc) { return __any(lds_ld(ctl, r * 32 + 24 + (lane & 1) * 4) + NAS <= (unsigned)turn_of(kc)); };
    bool bad = false;
    unsigned long long t0 = __builtin_amdgcn_s_memrealtime();
    if (wave == 10) {
      const u64* pollp = (const u64*)(lane < 8 ? lines + ((wg & 7) * 2 + (lane >> 2)) * 128 + (lane & 3) * 8 : my + TMO_OFF);
      int fk[2] = {0, 0};
      u64 pv = __hip_atomic_load(pollp, __ATOMIC_RELAXED, __HIP_MEMORY_SCOPE_SYSTEM);
      while (fk[0] < NP || fk[1] < NP) {
        const unsigned cls = (unsigned)(pv >> 32);
        bool any = false;
#pragma unroll
        for (int pr = 0; pr < 2; ++pr) {
          const int kc = fk[pr];
          if (kc < NP && __all((lane >> 1) != pr * 2 || (int)((unsigned)pv - fbase) > kc) && !slot_busy(pr * 2, kc) &&
              !slot_busy(pr * 2 + 1, kc)) {
            CBAR();
#pragma unroll
            for (int r = pr * 2; r < pr * 2 + 2; ++r) {
              const unsigned nl = tail_lanes(((unsigned)__builtin_amdgcn_readlane(cls, pr * 2 + r) >> (2 * kc)) & 3u);
              LAS unsigned char* dst = lds + ABUF_OFF + (r * NAS + turn_of(kc) % NAS) * 4096;
              const unsigned char* a = my + aoff + blk_off(rg0 + r, kc);
#pragma unroll
              for (int pc = 0; pc < 3; ++pc)
                __builtin_amdgcn_global_load_lds((const GAS void*)(a + pc * 1024), (LAS void*)(dst + pc * 1024), 16, 0, 17);
              if ((unsigned)lane < nl) __builtin_amdgcn_global_load_lds((const GAS void*)(a + 3072), (LAS void*)(dst + 3072), 16, 0, 17);
            }
            ++fk[pr];
            any = true;
          }
        }
        if (any) {
          t0 = __builtin_amdgcn_s_memrealtime();
        } else if ((unsigned)__builtin_amdgcn_readlane((unsigned)pv, 8) == call || __builtin_amdgcn_s_memrealtime() - t0 > DEADLINE_TICKS) {
          bad = true;
          break;
        }
        pv = __hip_atomic_load(pollp, __ATOMIC_RELAXED, __HIP_MEMORY_SCOPE_SYSTEM);
        wait_vm<0>();
        if (any) {
          if (lane < 8 && (lane & 2)) *(volatile LAS u64*)(lds + CTL_T + (lane >> 2) * 16 + (lane & 1) * 8) = pv;
          CBAR();
          if (lane < 8 && !(lane & 2))
            *(volatile LAS u64*)(ctl + ((lane >> 1) + (lane & 1)) * 32) = ((u64)cls << 32) | (unsigned)(lane < 4 ? fk[0] : fk[1]);
          DBG(fk[0] < fk[1] ? fk[0] : fk[1]);
        }
      }
    } else {
      const bool stg = lane >= 16 && lane < 16 + 4 * NPART;
      const u64* pollp = (const u64*)(lane < 16 ? lines + lane * 128
                                      : stg ? lines + ((wg & 7) * 2 + ((lane >> 1) & 1)) * 128 + 64 + (lane & 1) * 32 + ((lane >> 2) & 3) * 8
                                            : my + TMO_OFF);
      bool acquired = false;
      unsigned cls = 0;
      while (true) {
        u64 pv = fbase;
        if (acquired || lane >= 16) pv = __hip_atomic_load(pollp, __ATOMIC_RELAXED, __HIP_MEMORY_SCOPE_SYSTEM);
        if (acquired) {
          const int f = (int)((unsigned)pv - fbase);
          if ((int)__builtin_amdgcn_readlane(wave_scan(lane < 16 && f > 0 ? f : 0), 15) >= 16 * (NP - 2) + TSUM) break;
        } else if (__all(!stg || (int)((unsigned)pv - call) >= 0)) {
          DBG(1);
          cls = stg ? (unsigned)(pv >> 32) : 0u;
          cls |= __builtin_amdgcn_update_dpp(0u, cls, 0x124, 0xf, 0xf, false);
          cls |= __builtin_amdgcn_update_dpp(0u, cls, 0x128, 0xf, 0xf, false);
          __builtin_amdgcn_fence(__ATOMIC_ACQUIRE, "");
          wait_vm<0>();
          acquired = true;
          DBG(2);
          continue;
        }
        if ((unsigned)__builtin_amdgcn_readlane((unsigned)pv, 32) == call || __builtin_amdgcn_s_memrealtime() - t0 > DEADLINE_TICKS) {
          bad = true;
          break;
        }
        __builtin_amdgcn_s_sleep(4);
      }
      CBAR();
      DBG(3);
      __builtin_amdgcn_s_setprio(3);
      constexpr int NB = 4 * (NKC - NP);
      const unsigned char* rp = p.ws[src] + aoff + blk_off(rg0, 0);
      int pub = 0;
      auto landed = [&](int n) {
        pub = n > pub ? n : pub;
        if ((lane >> 2) == 4)
          *(volatile LAS u64*)(ctl + (lane & 3) * 32 + 8) = ((u64)cls << 32) | (unsigned)(NP + ((pub + 3 - (lane & 3)) >> 2));
      };
      auto used = [&]() { return lds_ld(ctl, ((lane >> 1) & 3) * 32 + 24 + (lane & 1) * 4); };
      unsigned us = used();
      int slot = turn_of(NP) % NAS;
      auto turn = [&](int i, auto win) {
        constexpr int W = decltype(win)::value;
        const int kc = NP + (i >> 2), r = i & 3;
        if (__any(((lane >> 1) & 3) == r && us + NAS <= (unsigned)turn_of(kc)) && slot_busy(r, kc)) {
          wait_vm<0>();
          landed(i);
          while (slot_busy(r, kc)) __builtin_amdgcn_s_sleep(1);
        }
        CBAR();
        us = used();
        const unsigned nl = tail_lanes(((unsigned)__builtin_amdgcn_readlane(cls, 16 + r) >> (2 * kc)) & 3u);
        LAS unsigned char* dst = lds + ABUF_OFF + (r * NAS + slot) * 4096;
        const unsigned char* a = rp + blk_off(r, kc);
#pragma unroll
        for (int pc = 0; pc < 4; ++pc) {
          if (pc < 3 || (unsigned)lane < nl)
            __builtin_amdgcn_global_load_lds((const GAS void*)(a + pc * 1024), (LAS void*)(dst + pc * 1024), 16, 0, 0);
          wait_vm<W>();
          if (pc == ((W - 1) & 3)) landed(i - (W - 1) / 4);
        }
        if (r == 3) {
          slot = turn_of(kc + 1) % NAS;
          DBG(kc);
        }
      };
      if (!bad) {
#pragma unroll 1
        for (int i = 0; i < NB - EBLK; ++i) turn(i, std::integral_constant<int, PWIN>{});
#pragma unroll 1
        for (int i = NB - EBLK; i < NB; ++i) turn(i, std::integral_constant<int, EWIN>{});
        wait_vm<12>();
        landed(NB - 3);
        wait_vm<8>();
        landed(NB - 2);
        wait_vm<4>();
        landed(NB - 1);
        wait_vm<0>();
        landed(NB);
      }
    }
    if (bad) {
      if (lane == 0) __hip_atomic_store((unsigned*)(my + TMO_OFF), call, __ATOMIC_RELAXED, __HIP_MEMORY_SCOPE_AGENT);
      if (lane < 4) *(volatile LAS u64*)(ctl + lane * 32 + (wave - 10) * 8) = 0xffffffff80000000ull;
    }
    DBG(18);
    DBG_FLUSH();
    return;
  }

#define SB() __builtin_amdgcn_sched_barrier(0)
#define PH(i) do {} while (0)
  if (!local && wave > 4) {
    const int gw = wave - 5;
    CBAR();
    LAS unsigned char* ctl = lds + CTL_A + gw * 32;
    LAS unsigned char* tab = lds + CTL_T + (gw >> 1) * 16;
    const unsigned e3 = SCR_OFF + gw * SCR_BYTES;
    __builtin_amdgcn_s_setprio(UPRIO);
    int k0 = 0, k1 = NP;
#pragma unroll 1
    while (k0 < NP || k1 < NKC) {
      const u32x4 st = *(volatile LAS u32x4*)ctl;
      const bool a0 = k0 < NP && (unsigned)__builtin_amdgcn_readfirstlane(st[0]) > (unsigned)k0;
      if (!a0 && !(k1 < NKC && k0 >= 2 && (unsigned)__builtin_amdgcn_readfirstlane(st[2]) > (unsigned)k1)) {
        __builtin_amdgcn_s_sleep(1);
        continue;
      }
      CBAR();
      const int kc = a0 ? k0++ : k1++;
      if (kc == 12) DBG(20);
      const unsigned cl = ((unsigned)__builtin_amdgcn_readfirstlane(a0 ? st[1] : st[3]) >> (2 * kc)) & 3u;
      const unsigned boff = ABUF_OFF + (gw * NAS + turn_of(kc) % NAS) * 4096;
      LAS unsigned char* blk = lds + boff;
      if (cl == 3) continue;
      PH(0);
      {
        const unsigned l3 = *(LAS unsigned short*)(blk + TAIL + tail_lanes(cl) * 16 - 4 - (lane >> 1) * 4 + (lane & 1) * 2);
        const unsigned T3 = *(LAS unsigned*)(tab + 8);
        const unsigned l2a = *(LAS unsigned*)(blk + TAIL + lane * 4);
        const u32x4 c1 = *(LAS u32x4*)(blk + 2048 + lane * 16);
        const u32x2 T12 = *(LAS u32x2*)(tab);
        const u32x4 sm0 = *(LAS u32x4*)(blk + lane * 16), sm1 = *(LAS u32x4*)(blk + 1024 + lane * 16);
        const unsigned l2b = *(LAS unsigned*)(blk + TAIL + 256 + lane * 4);
        SB();
        PH(1);
        auto lut2 = [&](unsigned c, int i) { return *(LAS u32x2*)(lds + LUT_OFF + ((c >> (8 * i)) & 255u) * 8); };
        auto window = [&](unsigned off) {
          const LAS unsigned* w = (const LAS unsigned*)(lds + (off & ~3u));
          return u32x2{w[0], w[1]};
        };
        auto pick = [&](u32x2 w, unsigned off, unsigned T, unsigned sel) {
          return __builtin_amdgcn_perm(__builtin_amdgcn_alignbyte(w[1], w[0], off), T, sel);
        };
        {
          const unsigned t = l3 | (l3 << 12);
          const unsigned u = t & 0x000f000fu, v = (t >> 8) & 0x000f000fu;
          *(LAS u32x2*)(lds + e3 + lane * 8) = u32x2{__builtin_amdgcn_perm(0u, T3, (u | (u << 6)) & 0x03030303u),
                                                     __builtin_amdgcn_perm(0u, T3, (v | (v << 6)) & 0x03030303u)};
        }
        u32x2 s2[4], s1[16];
#pragma unroll
        for (int i = 0; i < 4; ++i) s2[i] = lut2(l2a, i);
        SB();
        const unsigned mine = (esc_count(c1[0]) + esc_count(c1[1]) + esc_count(c1[2]) + esc_count(c1[3])) | (esc_count(l2a) << 16);
#pragma unroll
        for (int i = 0; i < 8; ++i) s1[i] = lut2(c1[i >> 2], i & 3);
        const unsigned inc = wave_scan(mine);
        const unsigned exc = inc - mine;
        const unsigned tot = __builtin_amdgcn_readlane(inc, 63);
        CBAR();
        SB();
        PH(2);
        unsigned o2[4], o1[16];
        u32x2 w2[4], w1[16];
        o2[0] = e3 + (exc >> 16);
#pragma unroll
        for (int i = 1; i < 4; ++i) o2[i] = o2[i - 1] + s2[i - 1][1];
#pragma unroll
        for (int i = 0; i < 4; ++i) w2[i] = window(o2[i]);
        SB();
#pragma unroll
        for (int i = 8; i < 16; ++i) s1[i] = lut2(c1[i >> 2], i & 3);
        SB();
        PH(3);
        {
          auto put = [&](const u32x4& o, LAS unsigned char* to) {
            const unsigned nxt = __builtin_amdgcn_update_dpp(0u, o[0], 0x130, 0xf, 0xf, true);
            *(LAS u32x4*)(to + lane * 32) = u32x4{o[0], o[1], o[1], o[2]};
            *(LAS u32x4*)(to + lane * 32 + 16) = u32x4{o[2], o[3], o[3], nxt};
          };
          u32x4 o;
#pragma unroll
          for (int i = 0; i < 4; ++i) o[i] = pick(w2[i], o2[i], T12[1], s2[i][0]);
          put(o, blk);
          if ((tot & 0xffff) > 1024) {
            const unsigned m2 = esc_count(l2b);
            unsigned off = e3 + (tot >> 16) + wave_scan(m2) - m2;
#pragma unroll
            for (int i = 0; i < 4; ++i) s2[i] = lut2(l2b, i);
#pragma unroll
            for (int i = 0; i < 4; ++i) { o[i] = pick(window(off), off, T12[1], s2[i][0]); off += s2[i][1]; }
            put(o, blk + 2048);
            if (lane == 0) *(LAS unsigned*)(blk + 2044) = o[0];
          }
        }
        CBAR();
        PH(4);
        o1[0] = boff / 2 - 4 + (exc & 0xffff);
#pragma unroll
        for (int i = 0; i < 16; ++i) {
          if (i) o1[i] = o1[i - 1] + s1[i - 1][1];
          w1[i] = *(const LAS u32x2*)(lds + ((o1[i] << 1) | 7u) + 1);
        }
        CBAR();
        SB();
        PH(5);
#pragma unroll
        for (int pc = 0; pc < 4; ++pc) {
          u32x4 o;
#pragma unroll
          for (int k = 0; k < 4; ++k) {
            const int i = pc * 4 + k;
            const unsigned s = i < 8 ? sm0[i >> 1] : sm1[(i >> 1) & 3];
            o[k] = (((i & 1) ? __builtin_amdgcn_alignbit(s, s, 4) : s) & SM_MASK) | pick(w1[i], o1[i], T12[0], s1[i][0]);
          }
          *(LAS u32x4*)(blk + pc * 1024 + lane * 16) = o;
        }
      }
      CBAR();
      lds_st(ctl, a0 ? 16 : 20, kc + 1);
      PH(6);
      if (kc == 12) DBG(21);
      DBG(1 + kc);
    }
    DBG_FLUSH();
    return;
  }

  const int gw = local ? 4 : wave;
  const int task = local ? grp - 1 : wave;
  const int ch = task & 1;
  const int rg0 = (wg & 7) * 4 + (task >> 1) * 2;
  const float sc = p.sx[0] * p.sw[0];
  const int q = r16 >> 1;
  const int wl = RING_OFF + (g * 16 + q) * 128 + (((((q >> 1) & 3) | ((g & 1) << 2)) ^ (ch * 4)) << 4) + (r16 & 1) * 8;
  const __amdgpu_buffer_rsrc_t ors = __builtin_amdgcn_make_buffer_rsrc((void*)p.out, 0, NR * MS * NL * 4, 0x00020000);
  const int ooff = ((src * MS + rg0 * 16 + (r16 & 7)) * NL + cb * 128 + ch * 64 + (r16 >> 3) * 16 + g * 4) * 4;
  const unsigned char* xs = p.x + (size_t)(rg0 * 16 + r16) * KD + g * 16;

  auto gemm = [&](auto tag) {
    constexpr bool LOCAL = decltype(tag)::value;
    f32x4 acc[8];
#pragma unroll
    for (int i = 0; i < 8; ++i) acc[i] = f32x4{0.f, 0.f, 0.f, 0.f};
    u32x4 nx[8];
    if constexpr (LOCAL) {
#pragma unroll
      for (int i = 0; i < 8; ++i) nx[i] = *(const u32x4*)(xs + (size_t)(i >> 2) * (16 * KD) + (i & 3) * 64);
    }
    if constexpr (!LOCAL) build_lut(true);
    __builtin_amdgcn_s_barrier();
    CBAR();
    bool bad = false;
    int slot = 0, aslot = 0;
#pragma unroll 1
    for (int turn = 0; turn < NKC; ++turn) {
      const int kc = chunk_of(turn);
      while (lds_ld(lds, CTL_FULL) <= (unsigned)(turn * 2 + 1)) __builtin_amdgcn_s_sleep(1);
      CBAR();
      i32x8 wa[8];
#pragma unroll
      for (int f = 0; f < 8; ++f) {
        LAS unsigned char* a = lds + slot * (2 * SLOT_BYTES) + (wl ^ ((f & 3) << 4)) + (f >> 2) * SLOT_BYTES;
        const i32x2 t0 = __builtin_amdgcn_ds_read_tr8_b64_v2i32((LAS i32x2*)(a));
        const i32x2 t1 = __builtin_amdgcn_ds_read_tr8_b64_v2i32((LAS i32x2*)(a + 1024));
        const i32x2 t2 = __builtin_amdgcn_ds_read_tr8_b64_v2i32((LAS i32x2*)(a + 8192));
        const i32x2 t3 = __builtin_amdgcn_ds_read_tr8_b64_v2i32((LAS i32x2*)(a + 9216));
        wa[f][0] = t0[0]; wa[f][1] = t0[1]; wa[f][2] = t1[0]; wa[f][3] = t1[1];
        wa[f][4] = t2[0]; wa[f][5] = t2[1]; wa[f][6] = t3[0]; wa[f][7] = t3[1];
      }
      CBAR();
      lds_st(lds, CTL_DONE + gw * 4, turn * 2 + 2);
#pragma unroll
      for (int r = 0; r < 2; ++r) {
        u32x4 xv[4];
        if constexpr (LOCAL) {
#pragma unroll
          for (int pc = 0; pc < 4; ++pc) xv[pc] = nx[r * 4 + pc];
        } else {
          LAS unsigned char* ctl = lds + CTL_A + ((wave & 2) + r) * 32;
          u64 st;
          while ((unsigned)(st = *(volatile LAS u64*)(ctl + (kc < NP ? 0 : 8))) <= (unsigned)kc) __builtin_amdgcn_s_sleep(1);
          bad |= (int)__builtin_amdgcn_readfirstlane((unsigned)st) < 0;
          if ((((unsigned)__builtin_amdgcn_readfirstlane((unsigned)(st >> 32)) >> (2 * kc)) & 3u) != 3u) {
            while (lds_ld(ctl, kc < NP ? 16 : 20) <= (unsigned)kc) {
              if ((int)lds_ld(ctl, kc < NP ? 0 : 8) < 0) {
                bad = true;
                break;
              }
              __builtin_amdgcn_s_sleep(GSLP);
            }
          }
          CBAR();
          if (kc == 15 && r == 1) DBG(1);
          LAS unsigned char* blk = lds + ABUF_OFF + (((wave & 2) + r) * NAS + aslot) * 4096;
#pragma unroll
          for (int pc = 0; pc < 4; ++pc) xv[pc] = *(LAS u32x4*)(blk + pc * 1024 + lane * 16);
          CBAR();
          lds_st(ctl, 24 + ch * 4, turn + 1);
        }
#pragma unroll
        for (int f = 0; f < 8; ++f) {
          i32x8 xb;
#pragma unroll
          for (int i = 0; i < 8; ++i) xb[i] = xv[(f >> 2) * 2 + (i >> 2)][i & 3];
          acc[r * 4 + (f & 3)] = __builtin_amdgcn_mfma_scale_f32_16x16x128_f8f6f4(wa[f], xb, acc[r * 4 + (f & 3)], 1, 0, 0, 0, 0, 0);
        }
        if constexpr (LOCAL) {
          SB();
#pragma unroll
          for (int pc = 0; pc < 4; ++pc)
            nx[r * 4 + pc] = *(const u32x4*)(xs + (size_t)r * (16 * KD) + chunk_of((turn + 1) & (NKC - 1)) * 256 + pc * 64);
          SB();
        }
        if (turn == NKC - 1) {
          if (r == 1) DBG(17);
          const float s = bad ? __builtin_nanf("") : sc;
#pragma unroll
          for (int t = 0; t < 2; ++t) {
            const u32x4 a = __builtin_bit_cast(u32x4, acc[r * 4 + 2 * t] * s), b = __builtin_bit_cast(u32x4, acc[r * 4 + 2 * t + 1] * s);
            u32x4 lo, hi;
#pragma unroll
            for (int i = 0; i < 4; ++i) {
              lo[i] = __builtin_amdgcn_update_dpp(a[i], b[i], 0x128, 0xf, 0xc, false);
              hi[i] = __builtin_amdgcn_update_dpp(b[i], a[i], 0x128, 0xf, 0x3, false);
            }
            __builtin_amdgcn_raw_buffer_store_b128(lo, ors, ooff + t * 128, r * 16 * NL * 4, OUT_AUX);
            __builtin_amdgcn_raw_buffer_store_b128(hi, ors, ooff + t * 128, (r * 16 + 8) * NL * 4, OUT_AUX);
          }
        }
      }
      slot = slot == WCH - 1 ? 0 : slot + 1;
      aslot = aslot == NAS - 1 ? 0 : aslot + 1;
      if (turn < 15) DBG(2 + turn);
    }
  };
  if (local) gemm(std::true_type{}); else gemm(std::false_type{});
}

}

extern "C" void kernel_launch(int rank, int nranks, ncclComm_t comm, hipStream_t stream,
                              void* const* d_in, const int* in_sizes, int n_in, void* d_out,
                              int out_size, const int* in_dtypes, int out_dtype,
                              void* const* d_out_all, void* const* d_ws_all, size_t ws_bytes) {
  (void)comm; (void)in_dtypes; (void)out_dtype; (void)d_out_all;
  if (nranks != NR || n_in != 4 || (unsigned)rank >= (unsigned)NR || ws_bytes < WS_NEED ||
      in_sizes[0] != MS * KD || in_sizes[1] != KD * NL || out_size != NR * MS * NL) {
    fprintf(stderr, "kernel_launch: unexpected problem shape\n");
    return;
  }
  Params p;
  p.x = (const unsigned char*)d_in[0];
  p.w = (const unsigned char*)d_in[1];
  p.sx = (const float*)d_in[2];
  p.sw = (const float*)d_in[3];
  p.out = (float*)d_out;
  for (int i = 0; i < NR; ++i) p.ws[i] = (unsigned char*)d_ws_all[i];
  p.my = p.ws[rank];
  p.rank = rank;
  p.pad = 0;
  hipLaunchKernelGGL(ag_gemm_kernel, dim3(NWG), dim3(NWAVE * 64), 0, stream, p);
  hipError_t e = hipGetLastError();
  if (e != hipSuccess) fprintf(stderr, "launch error %s\n", hipGetErrorString(e));
}
```

```cpp
#include <hip/hip_runtime.h>
#include <rccl/rccl.h>
#include <cstdio>
#include <cstdint>
#include <type_traits>

namespace {

constexpr int NR = 8;
constexpr int MS = 512;
constexpr int KD = 4096;
constexpr int NL = 512;
constexpr int NKC = 16;
constexpr int NRG = 32;
constexpr int NWG = 256;
constexpr int NWAVE = 12;
#define NPUSH 4
constexpr int NP = NPUSH;
constexpr int NPART = (NKC - NP + 3) / 4;
constexpr size_t SHARD_BYTES = (size_t)MS * KD;
constexpr size_t PAR_BYTES = NR * SHARD_BYTES;
constexpr size_t FLAG_OFF = 2 * PAR_BYTES;
constexpr size_t FLAG_PAR = (size_t)NR * (NRG / 2) * 128;
constexpr size_t CNT_OFF = FLAG_OFF + 2 * FLAG_PAR;
constexpr size_t TMO_OFF = CNT_OFF + (size_t)NWG * 128;
constexpr size_t PTAIL_OFF = TMO_OFF + 4096;
constexpr size_t PTAIL_PAR = (size_t)NRG * NKC * 320;
constexpr size_t DBG_OFF = PTAIL_OFF + 2 * PTAIL_PAR;
constexpr size_t WS_NEED = DBG_OFF + (1u << 19);
constexpr unsigned long long DEADLINE_TICKS = 50000000ull;

constexpr int LUT_OFF = 0;
constexpr int CTL_OFF = 2048;
constexpr int SCR_OFF = CTL_OFF + 512;
constexpr int SCR_BYTES = 512;
constexpr int TAB_OFF = SCR_OFF + 2048;
constexpr int ELUT_OFF = TAB_OFF + 1536;
constexpr int ETAIL_OFF = ELUT_OFF + 2048;
constexpr int BIG_OFF = 16384;
#define WCH 2
constexpr int NWS = 2 * WCH;
constexpr int NAS = 9 - 2 * WCH;
constexpr int SLOT_BYTES = 16384;
constexpr int RING_OFF = BIG_OFF;
constexpr int ABUF_OFF = RING_OFF + NWS * SLOT_BYTES;
constexpr int LDS_BYTES = ABUF_OFF + 4 * NAS * 4096;
static_assert(ETAIL_OFF + 4096 <= 14336 && NPART <= 4 && NP >= 2 && LDS_BYTES <= 163840, "LDS map");
constexpr int CTL_FULL = CTL_OFF;
constexpr int CTL_DONE = CTL_OFF + 32;
constexpr int CTL_A = CTL_OFF + 64;
constexpr int CTL_T = CTL_OFF + 192;
constexpr int CTL_E = CTL_OFF + 224;
constexpr int CTL_META = CTL_OFF;
constexpr int CTL_CALL = CTL_OFF + 128;
constexpr int TAIL = 3072;

typedef unsigned u32x4 __attribute__((ext_vector_type(4)));
typedef unsigned u32x3 __attribute__((ext_vector_type(3)));
typedef unsigned u32x2 __attribute__((ext_vector_type(2)));
typedef int i32x2 __attribute__((ext_vector_type(2)));
typedef int i32x8 __attribute__((ext_vector_type(8)));
typedef float f32x4 __attribute__((ext_vector_type(4)));
typedef unsigned long long u64;
#define LAS __attribute__((address_space(3)))
#define GAS __attribute__((address_space(1)))

struct Params {
  const unsigned char* x;
  unsigned char* my;
  const unsigned char* w;
  const float* sx;
  const float* sw;
  float* out;
  unsigned char* ws[NR];
  int rank;
  int pad;
};

__device__ __forceinline__ void st16_sys(unsigned char* p, u32x4 v) {
  asm volatile("global_store_dwordx4 %0, %1, off sc0 sc1\n\ts_nop 1" ::"v"(p), "v"(v) : "memory");
}
__device__ __forceinline__ void st8_sys(unsigned char* p, u32x2 v) {
  asm volatile("global_store_dwordx2 %0, %1, off sc0 sc1" ::"v"(p), "v"(v) : "memory");
}
template <int N>
__device__ __forceinline__ void wait_vm() {
  asm volatile("s_waitcnt vmcnt(%0)" ::"i"(N) : "memory");
}
__device__ __forceinline__ unsigned lds_ld(LAS unsigned char* l, int off) {
  return *(volatile LAS unsigned*)(l + off);
}
__device__ __forceinline__ void lds_st(LAS unsigned char* l, int off, unsigned v) {
  *(volatile LAS unsigned*)(l + off) = v;
}
#define CBAR() asm volatile("" ::: "memory")

__device__ __forceinline__ unsigned wave_scan(unsigned v) {
  v += __builtin_amdgcn_update_dpp(0u, v, 0x111, 0xf, 0xf, true);
  v += __builtin_amdgcn_update_dpp(0u, v, 0x112, 0xf, 0xf, true);
  v += __builtin_amdgcn_update_dpp(0u, v, 0x114, 0xf, 0xf, true);
  v += __builtin_amdgcn_update_dpp(0u, v, 0x118, 0xf, 0xf, true);
  v += __builtin_amdgcn_update_dpp(0u, v, 0x142, 0xa, 0xf, false);
  v += __builtin_amdgcn_update_dpp(0u, v, 0x143, 0xc, 0xf, false);
  return v;
}
__device__ __forceinline__ unsigned esc_count(unsigned c) {
  return __builtin_popcount(c & (c >> 1) & 0x55555555u);
}

#define DBG(slot) do {} while (0)
#define DBG_FLUSH() do {} while (0)

#define ENC_DELAY 4
#define HWIN 12
#define XMAP 0
#define RAWR 60
#define UPRIO 2
#define GSLP 2
#define WWIN 16
#define PACE 214
#define PLEAD -30
#define SWIN 4
#define PSLP 1
#define NRAW 2
#define TSUM 8
#define EBLK 0
#define EWIN 7
#define PWIN 5
#define QUIET 40
#define OUT_AUX 18
__device__ __forceinline__ unsigned blk_off(int rg, int kc) { return (unsigned)(rg * NKC + kc) * 4096u; }
#define ORD 0x3120
__device__ __forceinline__ int chunk_of(int turn) {
  const unsigned d = turn - (NP - 2);
  return d < 4 ? NP - 2 + ((ORD >> (4 * d)) & 3) : turn;
}
__device__ __forceinline__ int turn_of(int kc) {
  constexpr unsigned INV = (0u << (4 * (ORD & 3))) | (1u << (4 * ((ORD >> 4) & 3))) | (2u << (4 * ((ORD >> 8) & 3))) | (3u << (4 * ((ORD >> 12) & 3)));
  const unsigned d = kc - (NP - 2);
  return d < 4 ? NP - 2 + ((INV >> (4 * d)) & 3) : kc;
}
__device__ __forceinline__ unsigned tail_lanes(unsigned c) { return c == 0 ? 20u : c == 1 ? 24u : c == 2 ? 40u : 64u; }
constexpr unsigned SM_MASK = 0x87878787u;

__global__ void __launch_bounds__(NWAVE * 64, 3) ag_gemm_kernel(Params p) {
  __shared__ __attribute__((aligned(1024))) unsigned char smem[LDS_BYTES];
  LAS unsigned char* lds = (LAS unsigned char*)smem;
  const int wg = blockIdx.x;
  const int lane = threadIdx.x & 63;
  const int wave = __builtin_amdgcn_readfirstlane(threadIdx.x >> 6);
  const int grp = wg >> 5;
  unsigned char* my = p.my;
  const int g = lane >> 4, r16 = lane & 15;

  DBG(0);

  if (__builtin_expect(grp == 0 && wave == 0, 1)) {
    unsigned* cnt = (unsigned*)(my + CNT_OFF + (size_t)wg * 128);
    unsigned call;
    asm volatile("global_load_dword %0, %1, off sc1" : "=v"(call) : "v"(cnt) : "memory");
    const unsigned char* xs = p.x + (size_t)((wg & 15) * 32 + r16) * KD + g * 16;
#pragma unroll
    for (int i = 0; i < 4 * NRAW; ++i)
      __builtin_amdgcn_global_load_lds((const GAS void*)(xs + (size_t)((i >> 2) & 1) * (16 * KD) + (i >> 3) * 256 + (i & 3) * 64),
                                       (LAS void*)(lds + BIG_OFF + i * 1024), 16, 0, 0);
    __builtin_amdgcn_s_barrier();
    asm volatile("s_waitcnt vmcnt(%1)" : "+v"(call) : "i"(4 * NRAW - 4) : "memory");
    call = __builtin_amdgcn_readfirstlane(call) + 1;
    lds_st(lds, CTL_CALL, call);
    lds_st(lds, CTL_META, 0x80030000u | 64u);
    wait_vm<0>();
    if (lane < NRAW) lds_st(lds, CTL_META + lane * 4, 0x80030000u | 64u);
    DBG(1);
    if (lane == 0) __hip_atomic_store(cnt, call, __ATOMIC_RELAXED, __HIP_MEMORY_SCOPE_AGENT);
    DBG(18);
    DBG_FLUSH();
    return;
  }
  if (grp == 0 ? wave > 8 : (wave == 9 && grp > 4)) return;

  auto build_lut = [&](bool pack) {
    const int idx = (wave & 3) * 64 + lane;
    unsigned v = 0, n = 0;
#pragma unroll
    for (int b = 0; b < 4; ++b) {
      const unsigned dg = (idx >> (2 * b)) & 3;
      v |= (pack ? (dg == 3 ? 1u << (2 * n) : 0u) : (dg == 3 ? 4u + n : dg)) << (8 * b);
      n += dg == 3 ? 1u : 0u;
    }
    *(LAS u32x2*)(lds + (pack ? ELUT_OFF : LUT_OFF) + idx * 8) = u32x2{v, pack ? 2 * n : n};
    asm volatile("s_waitcnt lgkmcnt(0)" ::: "memory");
  };

  const int part = 7 - grp, pkc = NP + part * 4 + wave - 5;
  const bool packer = grp == 0 ? wave < 5 : (wave > 4 && wave < 9 && pkc < NKC);
  if (!packer) {
    if (wave == (grp == 0 ? 5 : 4))
      lds_st(lds, CTL_OFF + lane * 4, (grp != 0 && lane >= (grp > 4 ? 12 : 13) && lane < 16) ? 1000u : 0u);
    if (wave < 9 && wave != 4) build_lut(grp == 0 || wave < 4);
    asm volatile("s_waitcnt lgkmcnt(0)" ::: "memory");
    __builtin_amdgcn_s_barrier();
    CBAR();
  } else {
    {
      const bool home = grp != 0;
      const int rg0 = home ? wg & 30 : (wg & 15) * 2;
      const int j0 = home ? pkc * 2 + (wg & 1) : NRAW - 1 + wave, jend = home ? j0 + 1 : 2 * NP;
      unsigned* cnt = (unsigned*)(my + CNT_OFF + (size_t)wg * 128) + wave;
      const unsigned char* xs = p.x + (size_t)(rg0 * 16 + r16) * KD + g * 16;
      u32x4 nx[4];
      auto load = [&](int j) {
        const unsigned char* a = xs + (size_t)(j & 1) * (16 * KD) + (j >> 1) * 256;
#pragma unroll
        for (int pc = 0; pc < 4; ++pc) nx[pc] = *(const u32x4*)(a + pc * 64);
      };
      if (wave == 3 || wave == 4) __builtin_amdgcn_s_sleep(ENC_DELAY);
      __builtin_amdgcn_s_setprio(2);
      const u32x3 smp = *(const u32x3*)(p.x + (size_t)(rg0 * 16 + (lane & 31)) * KD + (lane >> 5) * 16);
      load(j0);
      unsigned call = __hip_atomic_load(cnt, __ATOMIC_RELAXED, __HIP_MEMORY_SCOPE_AGENT);
      CBAR();
      __builtin_amdgcn_sched_barrier(0);
      if (home) build_lut(false);
      __builtin_amdgcn_s_barrier();
      __builtin_amdgcn_sched_barrier(0);
      CBAR();
      u32x4 C;
      {
        LAS unsigned char* tab = lds + TAB_OFF + wave * 128;
        u64 h = 0;
#pragma unroll
        for (int i = 0; i < 3; ++i) {
#pragma unroll
          for (int b = 0; b < 4; ++b) h += 1ull << (((smp[i] >> (8 * b + 3)) & 15u) * 4u);
        }
        u32x4 hb = {(unsigned)h & 0x0f0f0f0fu, (unsigned)(h >> 32) & 0x0f0f0f0fu,
                    ((unsigned)h >> 4) & 0x0f0f0f0fu, (unsigned)(h >> 36) & 0x0f0f0f0fu};
#pragma unroll
        for (int i = 0; i < 4; ++i) {
          hb[i] += __builtin_amdgcn_update_dpp(0u, hb[i], 0x121, 0xf, 0xf, false);
          hb[i] += __builtin_amdgcn_update_dpp(0u, hb[i], 0x122, 0xf, 0xf, false);
          hb[i] += __builtin_amdgcn_update_dpp(0u, hb[i], 0x124, 0xf, 0xf, false);
          hb[i] += __builtin_amdgcn_update_dpp(0u, hb[i], 0x128, 0xf, 0xf, false);
        }
        const unsigned w01 = (r16 & 8) ? hb[1] : hb[0], w23 = (r16 & 8) ? hb[3] : hb[2];
        unsigned cnt = (((r16 & 1) ? w23 : w01) >> ((r16 & 6) * 4)) & 255u;
        cnt += __shfl_xor(cnt, 16);
        cnt += __shfl_xor(cnt, 32);
        const unsigned nb = __builtin_amdgcn_update_dpp(0u, cnt, 0x111, 0xf, 0xf, true) +
                            __builtin_amdgcn_update_dpp(0u, cnt, 0x101, 0xf, 0xf, true);
        const unsigned key = (cnt << 16) | (nb << 4) | (unsigned)r16;
        unsigned rank = 0;
#define RK(t) rank += __builtin_amdgcn_update_dpp(0u, key, 0x120 + t, 0xf, 0xf, false) > key ? 1u : 0u;
        RK(1) RK(2) RK(3) RK(4) RK(5) RK(6) RK(7) RK(8) RK(9) RK(10) RK(11) RK(12) RK(13) RK(14) RK(15)
#undef RK
        const unsigned code = rank < 3 ? rank : rank < 6 ? (3u | ((rank - 3) << 2)) : rank < 10 ? (15u | ((rank - 6) << 4)) : 0x80u;
        const unsigned slot = rank < 3 ? rank : rank < 6 ? rank + 1 : rank + 2;
        *(volatile LAS unsigned char*)(tab + r16) = (unsigned char)code;
        *(volatile LAS unsigned char*)(tab + 16 + slot) = (unsigned char)(r16 << 3);
        CBAR();
        C = *(LAS u32x4*)tab;
        if (wave == 1) lds_st(lds, CTL_CALL + 4, 1u);
      }
      DBG(home ? 19 : 10);
#pragma unroll 1
      for (int j = j0; j < jend; j += 4) {
        unsigned d[16];
#pragma unroll
        for (int i = 0; i < 16; ++i) d[i] = nx[i >> 2][i & 3];
        if (j + 4 < jend) load(j + 4);
        LAS unsigned char* blk = lds + (home ? ETAIL_OFF + (wave - 5) * 1024 - TAIL : BIG_OFF + j * 4096);
        unsigned meta = 0x80030000u | 64u;
        u32x4 o0, o1, o2;
        unsigned c1[4], a2[4], n2[4], a3 = 0, n3 = 0, badv = 0;
#pragma unroll
        for (int hf = 0; hf < 2; ++hf) {
          unsigned d2[8], d3[8];
          u32x2 w1[8], w2[8];
          c1[2 * hf] = 0;
          c1[2 * hf + 1] = 0;
#pragma unroll
          for (int i = 0; i < 8; ++i) {
            const unsigned v = d[hf * 8 + i];
            const unsigned e7 = (v >> 3) & 0x07070707u;
            const unsigned lo = __builtin_amdgcn_perm(C[1], C[0], e7);
            const unsigned hi = __builtin_amdgcn_perm(C[3], C[2], e7);
            const unsigned code = __builtin_amdgcn_perm(hi, lo, ((v >> 4) & 0x04040404u) | 0x03020100u);
            badv |= code;
            const unsigned p1 = __builtin_amdgcn_udot4(code & 0x03030303u, 0x40100401u, 0u, false);
            c1[2 * hf + (i >> 2)] |= p1 << (8 * (i & 3));
            d2[i] = (code >> 2) & 0x03030303u;
            d3[i] = (code >> 4) & 0x03030303u;
            const unsigned p2 = __builtin_amdgcn_udot4(d2[i], 0x40100401u, 0u, false);
            w1[i] = *(LAS u32x2*)(lds + ELUT_OFF + p1 * 8);
            w2[i] = *(LAS u32x2*)(lds + ELUT_OFF + p2 * 8);
          }
          __builtin_amdgcn_sched_barrier(0);
#pragma unroll
          for (int i = 0; i < 8; ++i) {
            const int q = 2 * hf + (i >> 2);
            if ((i & 3) == 0) { a2[q] = 0; n2[q] = 0; }
            a2[q] |= __builtin_amdgcn_udot4(d2[i], w1[i][0], 0u, false) << n2[q];
            n2[q] += w1[i][1];
            a3 |= __builtin_amdgcn_udot4(d3[i], w2[i][0], 0u, false) << (n3 & 31);
            n3 += w2[i][1];
          }
          asm volatile("" : "+v"(a2[2 * hf]), "+v"(a2[2 * hf + 1]), "+v"(a3), "+v"(c1[2 * hf]), "+v"(c1[2 * hf + 1]));
        }
        const unsigned mine = (n2[0] + n2[1] + n2[2] + n2[3]) | (n3 << 16);
        const unsigned inc = wave_scan(mine);
        const unsigned tot = __builtin_amdgcn_readlane(inc, 63);
        const unsigned t2 = tot & 0xffff, t3 = tot >> 16;
        const bool raw = __any((badv & 0x80808080u) != 0 || n3 > 32) || t2 > 4096 || t3 > 1024 ||
                         (j >> 1) * 4 + (wg & 3) >= RAWR;
        if (!raw) {
          *(LAS u32x4*)(blk + TAIL + lane * 16) = u32x4{0u, 0u, 0u, 0u};
          CBAR();
          const unsigned exc = inc - mine;
          unsigned pos = exc & 0xffff;
#pragma unroll
          for (int q = 0; q < 4; ++q) {
            const u64 v = (u64)a2[q] << (pos & 31);
            LAS unsigned* o = (LAS unsigned*)(blk + TAIL + (pos >> 5) * 4);
            __hip_atomic_fetch_or(o, (unsigned)v, __ATOMIC_RELAXED, __HIP_MEMORY_SCOPE_WORKGROUP);
            __hip_atomic_fetch_or(o + 1, (unsigned)(v >> 32), __ATOMIC_RELAXED, __HIP_MEMORY_SCOPE_WORKGROUP);
            pos += n2[q];
          }
          const unsigned tw = ((t2 + 31) >> 5) + ((t3 + 31) >> 5);
          const unsigned cl = tw <= 80 ? 0u : tw <= 96 ? 1u : 2u;
          const unsigned nl = tail_lanes(cl);
          {
            const unsigned p3 = exc >> 16;
            const u64 v = (u64)a3 << (p3 & 31);
            LAS unsigned* o = (LAS unsigned*)(blk + TAIL + nl * 16 - 8 - (p3 >> 5) * 4);
            __hip_atomic_fetch_or(o + 1, (unsigned)v, __ATOMIC_RELAXED, __HIP_MEMORY_SCOPE_WORKGROUP);
            __hip_atomic_fetch_or(o, (unsigned)(v >> 32), __ATOMIC_RELAXED, __HIP_MEMORY_SCOPE_WORKGROUP);
          }
#pragma unroll
          for (int i = 0; i < 4; ++i) {
            o0[i] = (d[2 * i] & SM_MASK) | (__builtin_amdgcn_alignbit(d[2 * i + 1], d[2 * i + 1], 28) & ~SM_MASK);
            o1[i] = (d[8 + 2 * i] & SM_MASK) | (__builtin_amdgcn_alignbit(d[9 + 2 * i], d[9 + 2 * i], 28) & ~SM_MASK);
          }
          o2 = u32x4{c1[0], c1[1], c1[2], c1[3]};
          meta = 0x80000000u | (cl << 16) | nl;
        } else {
          o0 = u32x4{d[0], d[1], d[2], d[3]};
          o1 = u32x4{d[4], d[5], d[6], d[7]};
          o2 = u32x4{d[8], d[9], d[10], d[11]};
          *(LAS u32x4*)(blk + TAIL + lane * 16) = u32x4{d[12], d[13], d[14], d[15]};
        }
        if (!home) {
          *(LAS u32x4*)(blk + lane * 16) = o0;
          *(LAS u32x4*)(blk + 1024 + lane * 16) = o1;
          *(LAS u32x4*)(blk + 2048 + lane * 16) = o2;
          CBAR();
          lds_st(lds, CTL_META + j * 4, meta);
          DBG(1 + (j - 2) / 4);
        } else {
          CBAR();
          const u32x4 o3 = *(const LAS u32x4*)(blk + TAIL + lane * 16);
          DBG(23);
          call = __builtin_amdgcn_readfirstlane(call) + 1;
          unsigned char* dst = my + (size_t)(call & 1) * PAR_BYTES + (size_t)p.rank * SHARD_BYTES + blk_off(wg & 31, pkc) + lane * 16;
          st16_sys(dst, o0);
          st16_sys(dst + 1024, o1);
          st16_sys(dst + 2048, o2);
          if ((unsigned)lane < (meta & 0xff))
            st16_sys((meta & 0x30000u) ? dst + 3072 : my + PTAIL_OFF + (call & 1) * PTAIL_PAR + ((wg & 31) * NKC + pkc) * 320 + lane * 16, o3);
          if (lane == 0) __hip_atomic_store(cnt, call, __ATOMIC_RELAXED, __HIP_MEMORY_SCOPE_AGENT);
          unsigned char* pf = p.ws[lane & 7] + FLAG_OFF + (size_t)(call & 1) * FLAG_PAR +
                              ((size_t)p.rank * (NRG / 2) + ((wg & 31) >> 1)) * 128 + 64 + (wg & 1) * 32 + part * 8;
          wait_vm<0>();
          unsigned prev = 0;
          if (lane == 0) {
            __hip_atomic_fetch_or((LAS unsigned*)(lds + CTL_E + 4), ((meta >> 16) & 3u) << (2 * pkc), __ATOMIC_RELAXED, __HIP_MEMORY_SCOPE_WORKGROUP);
            prev = __hip_atomic_fetch_add((LAS unsigned*)(lds + CTL_E), 1u, __ATOMIC_RELAXED, __HIP_MEMORY_SCOPE_WORKGROUP);
          }
          const int left = NKC - NP - part * 4;
          if ((int)__builtin_amdgcn_readfirstlane(prev) + 1 == (left < 4 ? left : 4)) {
            const unsigned cls = lds_ld(lds, CTL_E + 4);
            if (lane < NR && lane != p.rank) st8_sys(pf, u32x2{call, cls});
          }
          DBG(22);
        }
      }
      if (!home) {
        DBG_FLUSH();
        return;
      }
    }
  }

  if (__builtin_expect(grp == 0, 1)) {
    const int pair = wg & 15, rg0 = pair * 2;
    const unsigned peer = (unsigned)(p.rank + 1 + (wg >> 4) * 4 + wave - 5) & 7u;
    unsigned char* pw = p.ws[peer];
    if (peer == (unsigned)p.rank) return;
    __builtin_amdgcn_s_setprio(3);
    unsigned call;
    while ((call = __builtin_amdgcn_readfirstlane(lds_ld(lds, CTL_CALL))) == 0) {}
    const unsigned fbase = (call - 1) * NKC;
    DBG(1);
    unsigned char* dst = pw + (size_t)(call & 1) * PAR_BYTES + (size_t)p.rank * SHARD_BYTES + lane * 16;
    const size_t foff = FLAG_OFF + (size_t)(call & 1) * FLAG_PAR + ((size_t)p.rank * (NRG / 2) + pair) * 128;
    unsigned char* flag = pw + foff + (lane & 3) * 8;
    u64 cls = 0;
    const int jend = 2 * NP;
#pragma unroll 1
    for (int j = 0; j <= jend; ++j) {
      if (j < jend) {
        unsigned meta;
        while ((meta = __builtin_amdgcn_readfirstlane(lds_ld(lds, CTL_META + j * 4))) == 0) __builtin_amdgcn_s_sleep(1);
        CBAR();
        cls |= (u64)((meta >> 16) & 3u) << ((j >> 1) * 2 + (j & 1) * 32);
        const LAS u32x4* sb = (const LAS u32x4*)(lds + BIG_OFF + j * 4096 + lane * 16);
        const u32x4 v0 = sb[0], v1 = sb[64], v2 = sb[128], v3 = sb[192];
        unsigned char* d = dst + blk_off(rg0 + (j & 1), j >> 1);
        st16_sys(d, v0);
        wait_vm<SWIN>();
        st16_sys(d + 1024, v1);
        wait_vm<SWIN>();
        st16_sys(d + 2048, v2);
        wait_vm<SWIN>();
        if ((unsigned)lane < (meta & 0xff)) st16_sys(d + 3072, v3);
        wait_vm<4>();
      } else {
        wait_vm<0>();
      }
      if (j >= 2 && !(j & 1)) {
        while (NRAW > 2 && lds_ld(lds, CTL_CALL + 4) == 0) {}
        CBAR();
        u32x2 fv = *(const LAS u32x2*)(lds + TAB_OFF + 128 + (lane & 3) * 8);
        if (lane < 2) fv = u32x2{fbase + (j >> 1), (unsigned)(cls >> (lane * 32))};
        if (lane < 4) st8_sys(flag, fv);
      }
      DBG(2 + (j >> 1 < 15 ? j >> 1 : 15));
    }
    DBG(18);
    DBG_FLUSH();
    return;
  }

  const int cb = (wg >> 3) & 3;
  const bool local = wave == 9;
  const int src = local ? p.rank : (p.rank + grp) & 7;

  if (wave == 4) {
    if (part < NPART) __builtin_amdgcn_s_sleep(QUIET);
    const int kk = lane >> 3, c = lane & 7;
    const unsigned char* wb = p.w + (size_t)kk * NL + cb * 128;
    const int f0 = (kk >> 1) & 3;
    const unsigned char* s0 = wb + ((c ^ f0) << 4);
    const unsigned char* s1 = wb + ((c ^ (f0 | 4)) << 4);
    int slot = 0, full = 0;
#pragma unroll 1
    for (int s = 0; s < 32; ++s) {
      if (s >= NWS) {
        const unsigned need = s - (NWS - 1);
        bool first = true;
        while (true) {
          unsigned d = lds_ld(lds, CTL_DONE + (lane & 7) * 4);
          if (__all(d >= need)) break;
          if (first) {
            if (WWIN > 16) {
              wait_vm<16>();
              lds_st(lds, CTL_FULL, full > s - 1 ? full : s - 1);
            }
            wait_vm<0>();
            full = s;
            lds_st(lds, CTL_FULL, s);
            first = false;
          }
          __builtin_amdgcn_s_sleep(2);
        }
        CBAR();
      }
      LAS unsigned char* dst = lds + RING_OFF + slot * SLOT_BYTES;
      slot = slot == NWS - 1 ? 0 : slot + 1;
      const size_t koff = (size_t)(chunk_of(s >> 1) * 2 + (s & 1)) * 128 * NL;
#pragma unroll
      for (int j = 0; j < 16; ++j) {
        const unsigned char* gp = (((j >> 1) & 1) ? s1 : s0) + koff + (size_t)j * 8 * NL;
        __builtin_amdgcn_global_load_lds((const GAS void*)gp, (LAS void*)(dst + j * 1024), 16, 0, 0);
      }
      wait_vm<WWIN>();
      full = full > s + 1 - WWIN / 16 ? full : s + 1 - WWIN / 16;
      lds_st(lds, CTL_FULL, full);
    }
    wait_vm<0>();
    lds_st(lds, CTL_FULL, 32);
    return;
  }

  if (wave > 9) {
    unsigned* cnt = (unsigned*)(my + CNT_OFF + (size_t)wg * 128) + wave;
    unsigned call = __hip_atomic_load(cnt, __ATOMIC_RELAXED, __HIP_MEMORY_SCOPE_AGENT);
    call = __builtin_amdgcn_readfirstlane(call) + 1;
    if (lane == 0) __hip_atomic_store(cnt, call, __ATOMIC_RELAXED, __HIP_MEMORY_SCOPE_AGENT);
    const unsigned fbase = (call - 1) * NKC;
    const int rg0 = (wg & 7) * 4;
    const size_t aoff = (size_t)(call & 1) * PAR_BYTES + (size_t)src * SHARD_BYTES + lane * 16;
    const unsigned char* lines = my + FLAG_OFF + (size_t)(call & 1) * FLAG_PAR + (size_t)src * (NRG / 2) * 128;
    LAS unsigned char* ctl = lds + CTL_A;
    auto slot_busy = [&](int r, int kc) { return __any(lds_ld(ctl, r * 32 + 24 + (lane & 1) * 4) + NAS <= (unsigned)turn_of(kc)); };
    bool bad = false;
    unsigned long long t0 = __builtin_amdgcn_s_memrealtime();
    if (wave == 10) {
      const u64* pollp = (const u64*)(lane < 8 ? lines + ((wg & 7) * 2 + (lane >> 2)) * 128 + (lane & 3) * 8 : my + TMO_OFF);
      int fk[2] = {0, 0};
      u64 pv = __hip_atomic_load(pollp, __ATOMIC_RELAXED, __HIP_MEMORY_SCOPE_SYSTEM);
      while (fk[0] < NP || fk[1] < NP) {
        const unsigned cls = (unsigned)(pv >> 32);
        bool any = false;
#pragma unroll
        for (int pr = 0; pr < 2; ++pr) {
          const int kc = fk[pr];
          if (kc < NP && __all((lane >> 1) != pr * 2 || (int)((unsigned)pv - fbase) > kc) && !slot_busy(pr * 2, kc) &&
              !slot_busy(pr * 2 + 1, kc)) {
            CBAR();
#pragma unroll
            for (int r = pr * 2; r < pr * 2 + 2; ++r) {
              const unsigned nl = tail_lanes(((unsigned)__builtin_amdgcn_readlane(cls, pr * 2 + r) >> (2 * kc)) & 3u);
              LAS unsigned char* dst = lds + ABUF_OFF + (r * NAS + turn_of(kc) % NAS) * 4096;
              const unsigned char* a = my + aoff + blk_off(rg0 + r, kc);
#pragma unroll
              for (int pc = 0; pc < 3; ++pc)
                __builtin_amdgcn_global_load_lds((const GAS void*)(a + pc * 1024), (LAS void*)(dst + pc * 1024), 16, 0, 17);
              if ((unsigned)lane < nl) __builtin_amdgcn_global_load_lds((const GAS void*)(a + 3072), (LAS void*)(dst + 3072), 16, 0, 17);
            }
            ++fk[pr];
            any = true;
          }
        }
        if (any) {
          t0 = __builtin_amdgcn_s_memrealtime();
        } else if ((unsigned)__builtin_amdgcn_readlane((unsigned)pv, 8) == call || __builtin_amdgcn_s_memrealtime() - t0 > DEADLINE_TICKS) {
          bad = true;
          break;
        }
        pv = __hip_atomic_load(pollp, __ATOMIC_RELAXED, __HIP_MEMORY_SCOPE_SYSTEM);
        wait_vm<0>();
        if (any) {
          if (lane < 8 && (lane & 2)) *(volatile LAS u64*)(lds + CTL_T + (lane >> 2) * 16 + (lane & 1) * 8) = pv;
          CBAR();
          if (lane < 8 && !(lane & 2))
            *(volatile LAS u64*)(ctl + ((lane >> 1) + (lane & 1)) * 32) = ((u64)cls << 32) | (unsigned)(lane < 4 ? fk[0] : fk[1]);
          DBG(fk[0] < fk[1] ? fk[0] : fk[1]);
        }
      }
    } else {
      const bool stg = lane >= 16 && lane < 16 + 4 * NPART;
      const u64* pollp = (const u64*)(lane < 16 ? lines + lane * 128
                                      : stg ? lines + ((wg & 7) * 2 + ((lane >> 1) & 1)) * 128 + 64 + (lane & 1) * 32 + ((lane >> 2) & 3) * 8
                                            : my + TMO_OFF);
      bool acquired = false;
      unsigned cls = 0;
      while (true) {
        u64 pv = fbase;
        if (acquired || lane >= 16) pv = __hip_atomic_load(pollp, __ATOMIC_RELAXED, __HIP_MEMORY_SCOPE_SYSTEM);
        if (acquired) {
          const int f = (int)((unsigned)pv - fbase);
          if ((int)__builtin_amdgcn_readlane(wave_scan(lane < 16 && f > 0 ? f : 0), 15) >= 16 * (NP - 2) + TSUM) break;
        } else if (__all(!stg || (int)((unsigned)pv - call) >= 0)) {
          DBG(1);
          cls = stg ? (unsigned)(pv >> 32) : 0u;
          cls |= __builtin_amdgcn_update_dpp(0u, cls, 0x124, 0xf, 0xf, false);
          cls |= __builtin_amdgcn_update_dpp(0u, cls, 0x128, 0xf, 0xf, false);
          __builtin_amdgcn_fence(__ATOMIC_ACQUIRE, "");
          wait_vm<0>();
          acquired = true;
          DBG(2);
          continue;
        }
        if ((unsigned)__builtin_amdgcn_readlane((unsigned)pv, 32) == call || __builtin_amdgcn_s_memrealtime() - t0 > DEADLINE_TICKS) {
          bad = true;
          break;
        }
        __builtin_amdgcn_s_sleep(PSLP);
      }
      CBAR();
      DBG(3);
      __builtin_amdgcn_s_setprio(3);
      constexpr int NB = 4 * (NKC - NP);
      const unsigned char* rp = p.ws[src] + aoff + blk_off(rg0, 0);
      const unsigned char* rt = p.ws[src] + PTAIL_OFF + (call & 1) * PTAIL_PAR + rg0 * NKC * 320 + lane * 16;
      int pub = 0;
      auto landed = [&](int n) {
        pub = n > pub ? n : pub;
        if ((lane >> 2) == 4)
          *(volatile LAS u64*)(ctl + (lane & 3) * 32 + 8) = ((u64)cls << 32) | (unsigned)(NP + ((pub + 3 - (lane & 3)) >> 2));
      };
      auto used = [&]() { return lds_ld(ctl, ((lane >> 1) & 3) * 32 + 24 + (lane & 1) * 4); };
      unsigned us = used();
      int slot = turn_of(NP) % NAS;
      const unsigned tp0 = (unsigned)__builtin_amdgcn_s_memrealtime() - PLEAD;
      unsigned due = 0;
      auto turn = [&](int i, auto win) {
        constexpr int W = decltype(win)::value;
        const int kc = NP + (i >> 2), r = i & 3;
        if (__any((lane >> 1) == r && us + NAS <= (unsigned)turn_of(kc)) && slot_busy(r, kc)) {
          wait_vm<0>();
          landed(i);
          while (slot_busy(r, kc)) __builtin_amdgcn_s_sleep(1);
        }
        CBAR();
        us = used();
        const unsigned nl = tail_lanes(((unsigned)__builtin_amdgcn_readlane(cls, 16 + r) >> (2 * kc)) & 3u);
        LAS unsigned char* dst = lds + ABUF_OFF + (r * NAS + slot) * 4096;
        const unsigned char* a = rp + blk_off(r, kc);
        due += (192 + nl) * PACE;
#pragma unroll
        for (int pc = 0; pc < 4; ++pc) {
          if (pc < 3 || (unsigned)lane < nl)
            __builtin_amdgcn_global_load_lds((const GAS void*)(pc == 3 && nl == 20 ? rt + (r * NKC + kc) * 320 : a + pc * 1024),
                                             (LAS void*)(dst + pc * 1024), 16, 0, 0);
          wait_vm<W>();
          if (pc == ((W - 1) & 3)) landed(i - (W - 1) / 4);
        }
        if (r == 3) {
          slot = turn_of(kc + 1) % NAS;
          DBG(kc);
        }
      };
      if (!bad) {
#pragma unroll 1
        for (int i = 0; i < NB; ++i) {
          if ((int)((unsigned)__builtin_amdgcn_s_memrealtime() - tp0) < (int)(due >> 10)) turn(i, std::integral_constant<int, PWIN>{});
          else turn(i, std::integral_constant<int, EWIN>{});
        }
        wait_vm<12>();
        landed(NB - 3);
        wait_vm<8>();
        landed(NB - 2);
        wait_vm<4>();
        landed(NB - 1);
        wait_vm<0>();
        landed(NB);
      }
    }
    if (bad) {
      if (lane == 0) __hip_atomic_store((unsigned*)(my + TMO_OFF), call, __ATOMIC_RELAXED, __HIP_MEMORY_SCOPE_AGENT);
      if (lane < 4) *(volatile LAS u64*)(ctl + lane * 32 + (wave - 10) * 8) = 0xffffffff80000000ull;
    }
    DBG(18);
    DBG_FLUSH();
    return;
  }

#define SB() __builtin_amdgcn_sched_barrier(0)
#define PH(i) do {} while (0)
  if (!local && wave > 4) {
    const int gw = wave - 5;
    CBAR();
    LAS unsigned char* ctl = lds + CTL_A + gw * 32;
    LAS unsigned char* tab = lds + CTL_T + (gw >> 1) * 16;
    const unsigned e3 = SCR_OFF + gw * SCR_BYTES;
    __builtin_amdgcn_s_setprio(UPRIO);
    int k0 = 0, k1 = NP;
#pragma unroll 1
    while (k0 < NP || k1 < NKC) {
      const u32x4 st = *(volatile LAS u32x4*)ctl;
      const bool a0 = k0 < NP && (unsigned)__builtin_amdgcn_readfirstlane(st[0]) > (unsigned)k0;
      if (!a0 && !(k1 < NKC && k0 >= 2 && (unsigned)__builtin_amdgcn_readfirstlane(st[2]) > (unsigned)k1)) {
        __builtin_amdgcn_s_sleep(1);
        continue;
      }
      CBAR();
      const int kc = a0 ? k0++ : k1++;
      if (kc == 12) DBG(20);
      const unsigned cl = ((unsigned)__builtin_amdgcn_readfirstlane(a0 ? st[1] : st[3]) >> (2 * kc)) & 3u;
      const unsigned boff = ABUF_OFF + (gw * NAS + turn_of(kc) % NAS) * 4096;
      LAS unsigned char* blk = lds + boff;
      if (cl == 3) continue;
      PH(0);
      {
        const unsigned l3 = *(LAS unsigned short*)(blk + TAIL + tail_lanes(cl) * 16 - 4 - (lane >> 1) * 4 + (lane & 1) * 2);
        const unsigned T3 = *(LAS unsigned*)(tab + 8);
        const unsigned l2a = *(LAS unsigned*)(blk + TAIL + lane * 4);
        const u32x4 c1 = *(LAS u32x4*)(blk + 2048 + lane * 16);
        const u32x2 T12 = *(LAS u32x2*)(tab);
        const u32x4 sm0 = *(LAS u32x4*)(blk + lane * 16), sm1 = *(LAS u32x4*)(blk + 1024 + lane * 16);
        const unsigned l2b = *(LAS unsigned*)(blk + TAIL + 256 + lane * 4);
        SB();
        PH(1);
        auto lut2 = [&](unsigned c, int i) { return *(LAS u32x2*)(lds + LUT_OFF + ((c >> (8 * i)) & 255u) * 8); };
        auto window = [&](unsigned off) {
          const LAS unsigned* w = (const LAS unsigned*)(lds + (off & ~3u));
          return u32x2{w[0], w[1]};
        };
        auto pick = [&](u32x2 w, unsigned off, unsigned T, unsigned sel) {
          return __builtin_amdgcn_perm(__builtin_amdgcn_alignbyte(w[1], w[0], off), T, sel);
        };
        {
          const unsigned t = l3 | (l3 << 12);
          const unsigned u = t & 0x000f000fu, v = (t >> 8) & 0x000f000fu;
          *(LAS u32x2*)(lds + e3 + lane * 8) = u32x2{__builtin_amdgcn_perm(0u, T3, (u | (u << 6)) & 0x03030303u),
                                                     __builtin_amdgcn_perm(0u, T3, (v | (v << 6)) & 0x03030303u)};
        }
        u32x2 s2[4], s1[16];
#pragma unroll
        for (int i = 0; i < 4; ++i) s2[i] = lut2(l2a, i);
        SB();
        const unsigned mine = (esc_count(c1[0]) + esc_count(c1[1]) + esc_count(c1[2]) + esc_count(c1[3])) | (esc_count(l2a) << 16);
#pragma unroll
        for (int i = 0; i < 8; ++i) s1[i] = lut2(c1[i >> 2], i & 3);
        const unsigned inc = wave_scan(mine);
        const unsigned exc = inc - mine;
        const unsigned tot = __builtin_amdgcn_readlane(inc, 63);
        CBAR();
        SB();
        PH(2);
        unsigned o2[4], o1[16];
        u32x2 w2[4], w1[16];
        o2[0] = e3 + (exc >> 16);
#pragma unroll
        for (int i = 1; i < 4; ++i) o2[i] = o2[i - 1] + s2[i - 1][1];
#pragma unroll
        for (int i = 0; i < 4; ++i) w2[i] = window(o2[i]);
        SB();
#pragma unroll
        for (int i = 8; i < 16; ++i) s1[i] = lut2(c1[i >> 2], i & 3);
        SB();
        PH(3);
        {
          auto put = [&](const u32x4& o, LAS unsigned char* to) {
            const unsigned nxt = __builtin_amdgcn_update_dpp(0u, o[0], 0x130, 0xf, 0xf, true);
            *(LAS u32x4*)(to + lane * 32) = u32x4{o[0], o[1], o[1], o[2]};
            *(LAS u32x4*)(to + lane * 32 + 16) = u32x4{o[2], o[3], o[3], nxt};
          };
          u32x4 o;
#pragma unroll
          for (int i = 0; i < 4; ++i) o[i] = pick(w2[i], o2[i], T12[1], s2[i][0]);
          put(o, blk);
          if ((tot & 0xffff) > 1024) {
            const unsigned m2 = esc_count(l2b);
            unsigned off = e3 + (tot >> 16) + wave_scan(m2) - m2;
#pragma unroll
            for (int i = 0; i < 4; ++i) s2[i] = lut2(l2b, i);
#pragma unroll
            for (int i = 0; i < 4; ++i) { o[i] = pick(window(off), off, T12[1], s2[i][0]); off += s2[i][1]; }
            put(o, blk + 2048);
            if (lane == 0) *(LAS unsigned*)(blk + 2044) = o[0];
          }
        }
        CBAR();
        PH(4);
        o1[0] = boff / 2 - 4 + (exc & 0xffff);
#pragma unroll
        for (int i = 0; i < 16; ++i) {
          if (i) o1[i] = o1[i - 1] + s1[i - 1][1];
          w1[i] = *(const LAS u32x2*)(lds + ((o1[i] << 1) | 7u) + 1);
        }
        CBAR();
        SB();
        PH(5);
#pragma unroll
        for (int pc = 0; pc < 4; ++pc) {
          u32x4 o;
#pragma unroll
          for (int k = 0; k < 4; ++k) {
            const int i = pc * 4 + k;
            const unsigned s = i < 8 ? sm0[i >> 1] : sm1[(i >> 1) & 3];
            o[k] = (((i & 1) ? __builtin_amdgcn_alignbit(s, s, 4) : s) & SM_MASK) | pick(w1[i], o1[i], T12[0], s1[i][0]);
          }
          *(LAS u32x4*)(blk + pc * 1024 + lane * 16) = o;
        }
      }
      CBAR();
      lds_st(ctl, a0 ? 16 : 20, kc + 1);
      PH(6);
      if (kc == 12) DBG(21);
      DBG(1 + kc);
    }
    DBG_FLUSH();
    return;
  }

  const int gw = local ? 4 : wave;
  const int task = local ? grp - 1 : wave;
  const int ch = task & 1;
  const int rg0 = (wg & 7) * 4 + (task >> 1) * 2;
  const int q = r16 >> 1;
  const int wl = RING_OFF + (g * 16 + q) * 128 + (((((q >> 1) & 3) | ((g & 1) << 2)) ^ (ch * 4)) << 4) + (r16 & 1) * 8;
  const __amdgpu_buffer_rsrc_t ors = __builtin_amdgcn_make_buffer_rsrc((void*)p.out, 0, NR * MS * NL * 4, 0x00020000);
  const int ooff = ((src * MS + rg0 * 16 + (r16 & 7)) * NL + cb * 128 + ch * 64 + (r16 >> 3) * 16 + g * 4) * 4;
  const unsigned char* xs = p.x + (size_t)(rg0 * 16 + r16) * KD + g * 16;

  auto gemm = [&](auto tag) {
    constexpr bool LOCAL = decltype(tag)::value;
    f32x4 acc[8];
#pragma unroll
    for (int i = 0; i < 8; ++i) acc[i] = f32x4{0.f, 0.f, 0.f, 0.f};
    u32x4 nx[8];
    if constexpr (LOCAL) {
#pragma unroll
      for (int i = 0; i < 8; ++i) nx[i] = *(const u32x4*)(xs + (size_t)(i >> 2) * (16 * KD) + (i & 3) * 64);
    }
    const float sc = p.sx[0] * p.sw[0];
    bool bad = false;
    int slot = 0, aslot = 0;
#pragma unroll 1
    for (int turn = 0; turn < NKC; ++turn) {
      const int kc = chunk_of(turn);
      while (lds_ld(lds, CTL_FULL) <= (unsigned)(turn * 2 + 1)) __builtin_amdgcn_s_sleep(1);
      CBAR();
      i32x8 wa[8];
#pragma unroll
      for (int f = 0; f < 8; ++f) {
        LAS unsigned char* a = lds + slot * (2 * SLOT_BYTES) + (wl ^ ((f & 3) << 4)) + (f >> 2) * SLOT_BYTES;
        const i32x2 t0 = __builtin_amdgcn_ds_read_tr8_b64_v2i32((LAS i32x2*)(a));
        const i32x2 t1 = __builtin_amdgcn_ds_read_tr8_b64_v2i32((LAS i32x2*)(a + 1024));
        const i32x2 t2 = __builtin_amdgcn_ds_read_tr8_b64_v2i32((LAS i32x2*)(a + 8192));
        const i32x2 t3 = __builtin_amdgcn_ds_read_tr8_b64_v2i32((LAS i32x2*)(a + 9216));
        wa[f][0] = t0[0]; wa[f][1] = t0[1]; wa[f][2] = t1[0]; wa[f][3] = t1[1];
        wa[f][4] = t2[0]; wa[f][5] = t2[1]; wa[f][6] = t3[0]; wa[f][7] = t3[1];
      }
      CBAR();
      lds_st(lds, CTL_DONE + gw * 4, turn * 2 + 2);
#pragma unroll
      for (int r = 0; r < 2; ++r) {
        u32x4 xv[4];
        if constexpr (LOCAL) {
#pragma unroll
          for (int pc = 0; pc < 4; ++pc) xv[pc] = nx[r * 4 + pc];
        } else {
          LAS unsigned char* ctl = lds + CTL_A + ((wave & 2) + r) * 32;
          u64 st;
          while ((unsigned)(st = *(volatile LAS u64*)(ctl + (kc < NP ? 0 : 8))) <= (unsigned)kc) __builtin_amdgcn_s_sleep(1);
          bad |= (int)__builtin_amdgcn_readfirstlane((unsigned)st) < 0;
          if ((((unsigned)__builtin_amdgcn_readfirstlane((unsigned)(st >> 32)) >> (2 * kc)) & 3u) != 3u) {
            while (lds_ld(ctl, kc < NP ? 16 : 20) <= (unsigned)kc) {
              if ((int)lds_ld(ctl, kc < NP ? 0 : 8) < 0) {
                bad = true;
                break;
              }
              __builtin_amdgcn_s_sleep(GSLP);
            }
          }
          CBAR();
          if (kc == 15 && r == 1) DBG(1);
          LAS unsigned char* blk = lds + ABUF_OFF + (((wave & 2) + r) * NAS + aslot) * 4096;
#pragma unroll
          for (int pc = 0; pc < 4; ++pc) xv[pc] = *(LAS u32x4*)(blk + pc * 1024 + lane * 16);
          CBAR();
          lds_st(ctl, 24 + ch * 4, turn + 1);
        }
#pragma unroll
        for (int f = 0; f < 8; ++f) {
          i32x8 xb;
#pragma unroll
          for (int i = 0; i < 8; ++i) xb[i] = xv[(f >> 2) * 2 + (i >> 2)][i & 3];
          acc[r * 4 + (f & 3)] = __builtin_amdgcn_mfma_scale_f32_16x16x128_f8f6f4(wa[f], xb, acc[r * 4 + (f & 3)], 1, 0, 0, 0, 0, 0);
        }
        if constexpr (LOCAL) {
          SB();
#pragma unroll
          for (int pc = 0; pc < 4; ++pc)
            nx[r * 4 + pc] = *(const u32x4*)(xs + (size_t)r * (16 * KD) + chunk_of((turn + 1) & (NKC - 1)) * 256 + pc * 64);
          SB();
        }
        if (turn == NKC - 1 && (!LOCAL || r == 1)) {
          if (r == 1) DBG(17);
          const float s = bad ? __builtin_nanf("") : sc;
#pragma unroll
          for (int t = LOCAL ? -2 : 0; t < 2; ++t) {
            const u32x4 a = __builtin_bit_cast(u32x4, acc[r * 4 + 2 * t] * s), b = __builtin_bit_cast(u32x4, acc[r * 4 + 2 * t + 1] * s);
            u32x4 lo, hi;
#pragma unroll
            for (int i = 0; i < 4; ++i) {
              lo[i] = __builtin_amdgcn_update_dpp(a[i], b[i], 0x128, 0xf, 0xc, false);
              hi[i] = __builtin_amdgcn_update_dpp(b[i], a[i], 0x128, 0xf, 0x3, false);
            }
            const int row = t < 0 ? 0 : r * 16;
            __builtin_amdgcn_raw_buffer_store_b128(lo, ors, ooff + (t & 1) * 128, row * NL * 4, OUT_AUX);
            __builtin_amdgcn_raw_buffer_store_b128(hi, ors, ooff + (t & 1) * 128, (row + 8) * NL * 4, OUT_AUX);
          }
        }
      }
      slot = slot == WCH - 1 ? 0 : slot + 1;
      aslot = aslot == NAS - 1 ? 0 : aslot + 1;
      if (turn < 15) DBG(2 + turn);
    }
  };
  if (local) gemm(std::true_type{}); else gemm(std::false_type{});
}

}

extern "C" void kernel_launch(int rank, int nranks, ncclComm_t comm, hipStream_t stream,
                              void* const* d_in, const int* in_sizes, int n_in, void* d_out,
                              int out_size, const int* in_dtypes, int out_dtype,
                              void* const* d_out_all, void* const* d_ws_all, size_t ws_bytes) {
  (void)comm; (void)in_dtypes; (void)out_dtype; (void)d_out_all;
  if (nranks != NR || n_in != 4 || (unsigned)rank >= (unsigned)NR || ws_bytes < WS_NEED ||
      in_sizes[0] != MS * KD || in_sizes[1] != KD * NL || out_size != NR * MS * NL) {
    fprintf(stderr, "kernel_launch: unexpected problem shape\n");
    return;
  }
  Params p;
  p.x = (const unsigned char*)d_in[0];
  p.w = (const unsigned char*)d_in[1];
  p.sx = (const float*)d_in[2];
  p.sw = (const float*)d_in[3];
  p.out = (float*)d_out;
  for (int i = 0; i < NR; ++i) p.ws[i] = (unsigned char*)d_ws_all[i];
  p.my = p.ws[rank];
  p.rank = rank;
  p.pad = 0;
  hipLaunchKernelGGL(ag_gemm_kernel, dim3(NWG), dim3(NWAVE * 64), 0, stream, p);
  hipError_t e = hipGetLastError();
  if (e != hipSuccess) fprintf(stderr, "launch error %s\n", hipGetErrorString(e));
}
```

```cpp
#include <hip/hip_runtime.h>
#include <rccl/rccl.h>
#include <cstdio>
#include <cstdint>
#include <type_traits>

namespace {

constexpr int NR = 8;
constexpr int MS = 512;
constexpr int KD = 4096;
constexpr int NL = 512;
constexpr int NKC = 16;
constexpr int NRG = 32;
constexpr int NWG = 256;
constexpr int NWAVE = 12;
#define NPUSH 4
constexpr int NP = NPUSH;
constexpr int NPART = (NKC - NP + 3) / 4;
constexpr size_t SHARD_BYTES = (size_t)MS * KD;
constexpr size_t PAR_BYTES = NR * SHARD_BYTES;
constexpr size_t FLAG_OFF = 2 * PAR_BYTES;
constexpr size_t FLAG_PAR = (size_t)NR * (NRG / 2) * 128;
constexpr size_t CNT_OFF = FLAG_OFF + 2 * FLAG_PAR;
constexpr size_t TMO_OFF = CNT_OFF + (size_t)NWG * 128;
constexpr size_t PTAIL_OFF = TMO_OFF + 4096;
constexpr size_t PTAIL_PAR = (size_t)NRG * NKC * 320;
constexpr size_t DBG_OFF = PTAIL_OFF + 2 * PTAIL_PAR;
constexpr size_t WS_NEED = DBG_OFF + (1u << 19);
constexpr unsigned long long DEADLINE_TICKS = 50000000ull;

constexpr int LUT_OFF = 0;
constexpr int CTL_OFF = 2048;
constexpr int SCR_OFF = CTL_OFF + 512;
constexpr int SCR_BYTES = 512;
constexpr int TAB_OFF = SCR_OFF + 2048;
constexpr int ELUT_OFF = TAB_OFF + 1536;
constexpr int ETAIL_OFF = ELUT_OFF + 2048;
constexpr int BIG_OFF = 16384;
#define WCH 2
constexpr int NWS = 2 * WCH;
constexpr int NAS = 9 - 2 * WCH;
constexpr int SLOT_BYTES = 16384;
constexpr int RING_OFF = BIG_OFF;
constexpr int ABUF_OFF = RING_OFF + NWS * SLOT_BYTES;
constexpr int LDS_BYTES = ABUF_OFF + 4 * NAS * 4096;
static_assert(ETAIL_OFF + 4096 <= 14336 && NP == 4 && LDS_BYTES <= 163840, "LDS map");
constexpr int CTL_FULL = CTL_OFF;
constexpr int CTL_DONE = CTL_OFF + 32;
constexpr int CTL_A = CTL_OFF + 64;
constexpr int CTL_T = CTL_OFF + 192;
constexpr int CTL_E = CTL_OFF + 224;
constexpr int CTL_META = CTL_OFF;
constexpr int CTL_CALL = CTL_OFF + 128;
constexpr int RAW_OFF = BIG_OFF + 2 * NP * 4096;
constexpr int TAIL = 3072;

typedef unsigned u32x4 __attribute__((ext_vector_type(4)));
typedef unsigned u32x3 __attribute__((ext_vector_type(3)));
typedef unsigned u32x2 __attribute__((ext_vector_type(2)));
typedef int i32x2 __attribute__((ext_vector_type(2)));
typedef int i32x8 __attribute__((ext_vector_type(8)));
typedef float f32x4 __attribute__((ext_vector_type(4)));
typedef unsigned long long u64;
#define LAS __attribute__((address_space(3)))
#define GAS __attribute__((address_space(1)))

struct Params {
  const unsigned char* x;
  unsigned char* my;
  const unsigned char* w;
  const float* sx;
  const float* sw;
  float* out;
  unsigned char* ws[NR];
  int rank;
  int pad;
};

__device__ __forceinline__ void st16_sys(unsigned char* p, u32x4 v) {
  asm volatile("global_store_dwordx4 %0, %1, off sc0 sc1\n\ts_nop 1" ::"v"(p), "v"(v) : "memory");
}
__device__ __forceinline__ void st16_sys_lanes(unsigned char* p, u32x4 v, unsigned n) {
  asm volatile("s_mov_b64 exec, %2\n\tglobal_store_dwordx4 %0, %1, off sc0 sc1\n\ts_mov_b64 exec, -1\n\ts_nop 0"
               ::"v"(p), "v"(v), "s"(~0ull >> (64 - n)) : "memory");
}
__device__ __forceinline__ void st8_sys(unsigned char* p, u32x2 v) {
  asm volatile("global_store_dwordx2 %0, %1, off sc0 sc1" ::"v"(p), "v"(v) : "memory");
}
template <int N>
__device__ __forceinline__ void wait_vm() {
  asm volatile("s_waitcnt vmcnt(%0)" ::"i"(N) : "memory");
}
__device__ __forceinline__ unsigned lds_ld(LAS unsigned char* l, int off) {
  return *(volatile LAS unsigned*)(l + off);
}
__device__ __forceinline__ void lds_st(LAS unsigned char* l, int off, unsigned v) {
  *(volatile LAS unsigned*)(l + off) = v;
}
#define CBAR() asm volatile("" ::: "memory")

__device__ __forceinline__ unsigned wave_scan(unsigned v) {
  v += __builtin_amdgcn_update_dpp(0u, v, 0x111, 0xf, 0xf, true);
  v += __builtin_amdgcn_update_dpp(0u, v, 0x112, 0xf, 0xf, true);
  v += __builtin_amdgcn_update_dpp(0u, v, 0x114, 0xf, 0xf, true);
  v += __builtin_amdgcn_update_dpp(0u, v, 0x118, 0xf, 0xf, true);
  v += __builtin_amdgcn_update_dpp(0u, v, 0x142, 0xa, 0xf, false);
  v += __builtin_amdgcn_update_dpp(0u, v, 0x143, 0xc, 0xf, false);
  return v;
}
__device__ __forceinline__ unsigned esc_count(unsigned c) {
  return __builtin_popcount(c & (c >> 1) & 0x55555555u);
}

#define DBG(slot) do {} while (0)
#define DBG_FLUSH() do {} while (0)

#define RAWR 60
#define UPRIO 2
#define GSLP 4
#define WT0 7
#define PACE 214
#define PLEAD -45
#define PSLP 1
#define NSM 4
#define NST 4
#define LIGHT 7
#define GATE 60
#define PAHEAD 40
#define GPRIO 0
#define USLP 2
#define WSLP 4
#define FARB 40
#define FWIN 9
#define TSUM 8
#define EWIN 7
#define PWIN 5
#define QUIET 40
#define OUT_AUX 18
__device__ __forceinline__ unsigned blk_off(int rg, int kc) { return (unsigned)(rg * NKC + kc) * 4096u; }
#define ORD 0x3120
__device__ __forceinline__ int chunk_of(int turn) {
  const unsigned d = turn - (NP - 2);
  return d < 4 ? NP - 2 + ((ORD >> (4 * d)) & 3) : turn;
}
__device__ __forceinline__ int turn_of(int kc) {
  constexpr unsigned INV = (0u << (4 * (ORD & 3))) | (1u << (4 * ((ORD >> 4) & 3))) | (2u << (4 * ((ORD >> 8) & 3))) | (3u << (4 * ((ORD >> 12) & 3)));
  const unsigned d = kc - (NP - 2);
  return d < 4 ? NP - 2 + ((INV >> (4 * d)) & 3) : kc;
}
__device__ __forceinline__ unsigned tail_lanes(unsigned c) { return c == 0 ? 20u : c == 1 ? 24u : c == 2 ? 40u : 64u; }
static_assert(EWIN <= 8, "the end of the pull");
constexpr unsigned SM_MASK = 0x87878787u;
__device__ __forceinline__ unsigned ex_pair(unsigned a, unsigned b) {
  return (a & ~SM_MASK) | (__builtin_amdgcn_alignbit(b, b, 28) & SM_MASK);
}
__device__ __forceinline__ unsigned sm_pair(unsigned a, unsigned b) {
  return (a & SM_MASK) | (__builtin_amdgcn_alignbit(b, b, 28) & ~SM_MASK);
}

__global__ void __launch_bounds__(NWAVE * 64, 3) ag_gemm_kernel(Params p) {
  __shared__ __attribute__((aligned(1024))) unsigned char smem[LDS_BYTES];
  LAS unsigned char* lds = (LAS unsigned char*)smem;
  const int wg = blockIdx.x;
  const int lane = threadIdx.x & 63;
  const int wave = __builtin_amdgcn_readfirstlane(threadIdx.x >> 6);
  const int grp = wg >> 5;
  unsigned char* my = p.my;
  const int g = lane >> 4, r16 = lane & 15;

  DBG(0);

  if (__builtin_expect(grp == 0 && wave == 0, 1)) {
    unsigned* cnt = (unsigned*)(my + CNT_OFF + (size_t)wg * 128);
    unsigned call;
    asm volatile("global_load_dword %0, %1, off sc1" : "=v"(call) : "v"(cnt) : "memory");
    const unsigned char* xs = p.x + (size_t)((wg & 15) * 32 + r16) * KD + g * 16;
    auto stage = [&](int i0, int i1) {
#pragma unroll
      for (int i = i0; i < i1; ++i)
        __builtin_amdgcn_global_load_lds((const GAS void*)(xs + (size_t)((i >> 2) & 1) * (16 * KD) + (i >> 3) * 256 + (i & 3) * 64),
                                         (LAS void*)(lds + RAW_OFF + i * 1024), 16, 0, 0);
    };
    stage(0, 4 * NST);
    __builtin_amdgcn_s_barrier();
    asm volatile("s_waitcnt vmcnt(%1)" : "+v"(call) : "i"(4 * NST - 4) : "memory");
    call = __builtin_amdgcn_readfirstlane(call) + 1;
    lds_st(lds, CTL_CALL, call);
    wait_vm<4 * NST - 8>();
    lds_st(lds, CTL_CALL + 4, 2u);
    wait_vm<4 * NST - 16>();
    lds_st(lds, CTL_CALL + 4, 4u);
    stage(4 * NST, 32);
    wait_vm<8>();
    lds_st(lds, CTL_CALL + 4, 6u);
    wait_vm<0>();
    lds_st(lds, CTL_CALL + 4, 8u);
    DBG(1);
    if (lane == 0) __hip_atomic_store(cnt, call, __ATOMIC_RELAXED, __HIP_MEMORY_SCOPE_AGENT);
    DBG(18);
    DBG_FLUSH();
    return;
  }
  if (grp == 0 ? wave > 8 : (wave == 9 && grp > 4)) return;

  auto build_lut = [&](bool pack) {
    const int idx = (wave & 3) * 64 + lane;
    unsigned v = 0, n = 0;
#pragma unroll
    for (int b = 0; b < 4; ++b) {
      const unsigned dg = (idx >> (2 * b)) & 3;
      v |= (pack ? (dg == 3 ? 1u << (2 * n) : 0u) : (dg == 3 ? 4u + n : dg)) << (8 * b);
      n += dg == 3 ? 1u : 0u;
    }
    *(LAS u32x2*)(lds + (pack ? ELUT_OFF : LUT_OFF) + idx * 8) = u32x2{v, pack ? 2 * n : n};
    asm volatile("s_waitcnt lgkmcnt(0)" ::: "memory");
  };

  const int part = 7 - grp, pkc = NP + part * 4 + wave - 5;
  const bool packer = grp == 0 ? wave < 5 : (wave > 4 && wave < 9 && pkc < NKC);
  if (!packer) {
    if (wave == (grp == 0 ? 5 : 4))
      lds_st(lds, CTL_OFF + lane * 4, (grp != 0 && lane >= (grp > 4 ? 12 : 13) && lane < 16) ? 1000u : 0u);
    if (wave < 9 && wave != 4) build_lut(grp == 0 || wave < 4);
    asm volatile("s_waitcnt lgkmcnt(0)" ::: "memory");
    __builtin_amdgcn_s_barrier();
    CBAR();
  } else {
    {
      const bool home = grp != 0;
      const int rg0 = home ? wg & 30 : (wg & 15) * 2;
      const int j0 = home ? pkc * 2 + (wg & 1) : wave - 1, jend = home ? j0 + 1 : 2 * NP;
      unsigned* cnt = (unsigned*)(my + CNT_OFF + (size_t)wg * 128) + wave;
      const unsigned char* xs = p.x + (size_t)(rg0 * 16 + r16) * KD + g * 16;
      __builtin_amdgcn_s_setprio(2);
      u32x3 smp;
      u32x4 nx[4];
      unsigned call = 0;
      if (home) {
        smp = *(const u32x3*)(p.x + (size_t)(rg0 * 16 + (lane >> 5) * 16 + r16) * KD + (g & 1) * 16);
#pragma unroll
        for (int pc = 0; pc < 4; ++pc) nx[pc] = *(const u32x4*)(xs + (size_t)(j0 & 1) * (16 * KD) + (j0 >> 1) * 256 + pc * 64);
        call = __hip_atomic_load(cnt, __ATOMIC_RELAXED, __HIP_MEMORY_SCOPE_AGENT);
        CBAR();
        __builtin_amdgcn_sched_barrier(0);
        build_lut(false);
        __builtin_amdgcn_s_barrier();
        __builtin_amdgcn_sched_barrier(0);
      } else {
        __builtin_amdgcn_s_barrier();
        while (lds_ld(lds, CTL_CALL + 4) < 2) {}
        CBAR();
        smp = *(const LAS u32x3*)(lds + RAW_OFF + (lane >> 5) * 4096 + (lane & 31) * 16);
      }
      CBAR();
      u32x4 C;
      {
        LAS unsigned char* tab = lds + TAB_OFF + wave * 128;
        u64 h = 0;
#pragma unroll
        for (int i = 0; i < 3; ++i) {
#pragma unroll
          for (int b = 0; b < 4; ++b) h += 1ull << (((smp[i] >> (8 * b + 3)) & 15u) * 4u);
        }
        u32x4 hb = {(unsigned)h & 0x0f0f0f0fu, (unsigned)(h >> 32) & 0x0f0f0f0fu,
                    ((unsigned)h >> 4) & 0x0f0f0f0fu, (unsigned)(h >> 36) & 0x0f0f0f0fu};
#pragma unroll
        for (int i = 0; i < 4; ++i) {
          hb[i] += __builtin_amdgcn_update_dpp(0u, hb[i], 0x121, 0xf, 0xf, false);
          hb[i] += __builtin_amdgcn_update_dpp(0u, hb[i], 0x122, 0xf, 0xf, false);
          hb[i] += __builtin_amdgcn_update_dpp(0u, hb[i], 0x124, 0xf, 0xf, false);
          hb[i] += __builtin_amdgcn_update_dpp(0u, hb[i], 0x128, 0xf, 0xf, false);
        }
        const unsigned w01 = (r16 & 8) ? hb[1] : hb[0], w23 = (r16 & 8) ? hb[3] : hb[2];
        unsigned cnt = (((r16 & 1) ? w23 : w01) >> ((r16 & 6) * 4)) & 255u;
        cnt += __shfl_xor(cnt, 16);
        cnt += __shfl_xor(cnt, 32);
        const unsigned nb = __builtin_amdgcn_update_dpp(0u, cnt, 0x111, 0xf, 0xf, true) +
                            __builtin_amdgcn_update_dpp(0u, cnt, 0x101, 0xf, 0xf, true);
        const unsigned key = (cnt << 16) | (nb << 4) | (unsigned)r16;
        unsigned rank = 0;
#define RK(t) rank += __builtin_amdgcn_update_dpp(0u, key, 0x120 + t, 0xf, 0xf, false) > key ? 1u : 0u;
        RK(1) RK(2) RK(3) RK(4) RK(5) RK(6) RK(7) RK(8) RK(9) RK(10) RK(11) RK(12) RK(13) RK(14) RK(15)
#undef RK
        const unsigned code = rank < 3 ? rank : rank < 6 ? (3u | ((rank - 3) << 2)) : rank < 10 ? (15u | ((rank - 6) << 4)) : 0x80u;
        const unsigned slot = rank < 3 ? rank : rank < 6 ? rank + 1 : rank + 2;
        *(volatile LAS unsigned char*)(tab + r16) = (unsigned char)code;
        *(volatile LAS unsigned char*)(tab + 16 + slot) = (unsigned char)(r16 << 3);
        CBAR();
        C = *(LAS u32x4*)tab;
      }
      DBG(home ? 19 : 10);
#pragma unroll 1
      for (int j = j0; j < jend; j += 4) {
        if (!home) {
          while (lds_ld(lds, CTL_CALL + 4) <= (unsigned)j) __builtin_amdgcn_s_sleep(1);
          CBAR();
#pragma unroll
          for (int pc = 0; pc < 4; ++pc) nx[pc] = *(const LAS u32x4*)(lds + RAW_OFF + j * 4096 + pc * 1024 + lane * 16);
        }
        unsigned d[16];
#pragma unroll
        for (int i = 0; i < 16; ++i) d[i] = nx[i >> 2][i & 3];
        LAS unsigned char* blk = lds + (home ? ETAIL_OFF + (wave - 5) * 1024 - TAIL : BIG_OFF + j * 4096);
        unsigned meta = 0x80030000u | 64u;
        u32x4 o0, o1, o2;
        unsigned c1[4], a2[4], n2[4], a3 = 0, n3 = 0, badv = 0;
#pragma unroll
        for (int hf = 0; hf < 2; ++hf) {
          unsigned d2[8], d3[8];
          u32x2 w1[8], w2[8];
          c1[2 * hf] = 0;
          c1[2 * hf + 1] = 0;
#pragma unroll
          for (int i = 0; i < 8; ++i) {
            const unsigned v = d[hf * 8 + i];
            const unsigned e7 = (v >> 3) & 0x07070707u;
            const unsigned lo = __builtin_amdgcn_perm(C[1], C[0], e7);
            const unsigned hi = __builtin_amdgcn_perm(C[3], C[2], e7);
            const unsigned code = __builtin_amdgcn_perm(hi, lo, ((v >> 4) & 0x04040404u) | 0x03020100u);
            badv |= code;
            const unsigned p1 = __builtin_amdgcn_udot4(code & 0x03030303u, 0x40100401u, 0u, false);
            c1[2 * hf + (i >> 2)] |= p1 << (8 * (i & 3));
            d2[i] = (code >> 2) & 0x03030303u;
            d3[i] = (code >> 4) & 0x03030303u;
            const unsigned p2 = __builtin_amdgcn_udot4(d2[i], 0x40100401u, 0u, false);
            w1[i] = *(LAS u32x2*)(lds + ELUT_OFF + p1 * 8);
            w2[i] = *(LAS u32x2*)(lds + ELUT_OFF + p2 * 8);
          }
          __builtin_amdgcn_sched_barrier(0);
#pragma unroll
          for (int i = 0; i < 8; ++i) {
            const int q = 2 * hf + (i >> 2);
            if ((i & 3) == 0) { a2[q] = 0; n2[q] = 0; }
            a2[q] |= __builtin_amdgcn_udot4(d2[i], w1[i][0], 0u, false) << n2[q];
            n2[q] += w1[i][1];
            a3 |= __builtin_amdgcn_udot4(d3[i], w2[i][0], 0u, false) << (n3 & 31);
            n3 += w2[i][1];
          }
          asm volatile("" : "+v"(a2[2 * hf]), "+v"(a2[2 * hf + 1]), "+v"(a3), "+v"(c1[2 * hf]), "+v"(c1[2 * hf + 1]));
        }
        const unsigned mine = (n2[0] + n2[1] + n2[2] + n2[3]) | (n3 << 16);
        const unsigned inc = wave_scan(mine);
        const unsigned tot = __builtin_amdgcn_readlane(inc, 63);
        const unsigned t2 = tot & 0xffff, t3 = tot >> 16;
        const bool raw = __any((badv & 0x80808080u) != 0 || n3 > 32) || t2 > 4096 || t3 > 1024 ||
                         (j >> 1) * 4 + (wg & 3) >= RAWR;
        if (!raw) {
          *(LAS u32x4*)(blk + TAIL + lane * 16) = u32x4{0u, 0u, 0u, 0u};
          CBAR();
          const unsigned exc = inc - mine;
          unsigned pos = exc & 0xffff;
#pragma unroll
          for (int q = 0; q < 4; ++q) {
            const u64 v = (u64)a2[q] << (pos & 31);
            LAS unsigned* o = (LAS unsigned*)(blk + TAIL + (pos >> 5) * 4);
            __hip_atomic_fetch_or(o, (unsigned)v, __ATOMIC_RELAXED, __HIP_MEMORY_SCOPE_WORKGROUP);
            __hip_atomic_fetch_or(o + 1, (unsigned)(v >> 32), __ATOMIC_RELAXED, __HIP_MEMORY_SCOPE_WORKGROUP);
            pos += n2[q];
          }
          const unsigned tw = ((t2 + 31) >> 5) + ((t3 + 31) >> 5);
          const unsigned cl = tw <= 80 ? 0u : tw <= 96 ? 1u : 2u;
          const unsigned nl = tail_lanes(cl);
          {
            const unsigned p3 = exc >> 16;
            const u64 v = (u64)a3 << (p3 & 31);
            LAS unsigned* o = (LAS unsigned*)(blk + TAIL + nl * 16 - 8 - (p3 >> 5) * 4);
            __hip_atomic_fetch_or(o + 1, (unsigned)v, __ATOMIC_RELAXED, __HIP_MEMORY_SCOPE_WORKGROUP);
            __hip_atomic_fetch_or(o, (unsigned)(v >> 32), __ATOMIC_RELAXED, __HIP_MEMORY_SCOPE_WORKGROUP);
          }
#pragma unroll
          for (int i = 0; i < 4; ++i) {
            o0[i] = sm_pair(d[2 * i], d[2 * i + 1]);
            o1[i] = sm_pair(d[8 + 2 * i], d[9 + 2 * i]);
          }
          o2 = u32x4{c1[0], c1[1], c1[2], c1[3]};
          meta = 0x80000000u | (cl << 16) | nl;
        } else {
          u32x4 o3;
#pragma unroll
          for (int i = 0; i < 4; ++i) {
            o0[i] = home ? d[i] : sm_pair(d[2 * i], d[2 * i + 1]);
            o1[i] = home ? d[4 + i] : sm_pair(d[8 + 2 * i], d[9 + 2 * i]);
            o2[i] = home ? d[8 + i] : ex_pair(d[2 * i], d[2 * i + 1]);
            o3[i] = home ? d[12 + i] : ex_pair(d[8 + 2 * i], d[9 + 2 * i]);
          }
          *(LAS u32x4*)(blk + TAIL + lane * 16) = o3;
        }
        if (!home) {
          if (j >= NSM) {
            *(LAS u32x4*)(blk + lane * 16) = o0;
            *(LAS u32x4*)(blk + 1024 + lane * 16) = o1;
          }
          *(LAS u32x4*)(blk + 2048 + lane * 16) = o2;
          CBAR();
          lds_st(lds, CTL_META + j * 4, meta);
          DBG(1 + j / 4);
        } else {
          CBAR();
          const u32x4 o3 = *(const LAS u32x4*)(blk + TAIL + lane * 16);
          DBG(23);
          call = __builtin_amdgcn_readfirstlane(call) + 1;
          unsigned char* dst = my + (size_t)(call & 1) * PAR_BYTES + (size_t)p.rank * SHARD_BYTES + blk_off(wg & 31, pkc) + lane * 16;
          st16_sys(dst, o0);
          st16_sys(dst + 1024, o1);
          st16_sys(dst + 2048, o2);
          if ((unsigned)lane < (meta & 0xff))
            st16_sys((meta & 0x30000u) ? dst + 3072 : my + PTAIL_OFF + (call & 1) * PTAIL_PAR + ((wg & 31) * NKC + pkc) * 320 + lane * 16, o3);
          if (lane == 0) __hip_atomic_store(cnt, call, __ATOMIC_RELAXED, __HIP_MEMORY_SCOPE_AGENT);
          unsigned char* pf = p.ws[lane & 7] + FLAG_OFF + (size_t)(call & 1) * FLAG_PAR +
                              ((size_t)p.rank * (NRG / 2) + ((wg & 31) >> 1)) * 128 + 64 + (wg & 1) * 32 + part * 8;
          wait_vm<0>();
          unsigned prev = 0;
          if (lane == 0) {
            __hip_atomic_fetch_or((LAS unsigned*)(lds + CTL_E + 4), ((meta >> 16) & 3u) << (2 * pkc), __ATOMIC_RELAXED, __HIP_MEMORY_SCOPE_WORKGROUP);
            prev = __hip_atomic_fetch_add((LAS unsigned*)(lds + CTL_E), 1u, __ATOMIC_RELAXED, __HIP_MEMORY_SCOPE_WORKGROUP);
          }
          const int left = NKC - NP - part * 4;
          if ((int)__builtin_amdgcn_readfirstlane(prev) + 1 == (left < 4 ? left : 4)) {
            const unsigned cls = lds_ld(lds, CTL_E + 4);
            if (lane < NR && lane != p.rank) st8_sys(pf, u32x2{call, cls});
          }
          DBG(22);
        }
      }
      if (!home) {
        DBG_FLUSH();
        return;
      }
    }
  }

  if (__builtin_expect(grp == 0, 1)) {
    const int pair = wg & 15, rg0 = pair * 2;
    const unsigned peer = (unsigned)(p.rank + 1 + (wg >> 4) * 4 + wave - 5) & 7u;
    unsigned char* pw = p.ws[peer];
    if (peer == (unsigned)p.rank) return;
    __builtin_amdgcn_s_setprio(3);
    unsigned call;
    while ((call = __builtin_amdgcn_readfirstlane(lds_ld(lds, CTL_CALL))) == 0) {}
    const unsigned fbase = (call - 1) * NKC;
    DBG(1);
    unsigned char* dst = pw + (size_t)(call & 1) * PAR_BYTES + (size_t)p.rank * SHARD_BYTES + lane * 16;
    const size_t foff = FLAG_OFF + (size_t)(call & 1) * FLAG_PAR + ((size_t)p.rank * (NRG / 2) + pair) * 128;
    unsigned char* flag = pw + foff + (lane & 3) * 8;
    u64 cls = 0;
#pragma unroll 1
    for (int j = 0; j < NSM; ++j) {
      while (j && lds_ld(lds, CTL_CALL + 4) <= (unsigned)j) {}
      CBAR();
      const LAS u32x4* sb = (const LAS u32x4*)(lds + RAW_OFF + j * 4096 + lane * 16);
      const u32x4 v0 = sb[0], v1 = sb[64], v2 = sb[128], v3 = sb[192];
      const u32x4 s0 = {sm_pair(v0[0], v0[1]), sm_pair(v0[2], v0[3]), sm_pair(v1[0], v1[1]), sm_pair(v1[2], v1[3])};
      u32x4 s1 = {sm_pair(v2[0], v2[1]), sm_pair(v2[2], v2[3]), sm_pair(v3[0], v3[1]), sm_pair(v3[2], v3[3])};
      unsigned char* d = dst + blk_off(rg0 + (j & 1), j >> 1);
      asm volatile("" : "+v"(s1));
      wait_vm<4>();
      st16_sys(d, s0);
      wait_vm<4>();
      st16_sys(d + 1024, s1);
    }
    DBG(17);
#pragma unroll 1
    for (int j = 0; j <= 2 * NP; ++j) {
      if (j < 2 * NP) {
        unsigned meta;
        while ((meta = __builtin_amdgcn_readfirstlane(lds_ld(lds, CTL_META + j * 4))) == 0) __builtin_amdgcn_s_sleep(1);
        CBAR();
        cls |= (u64)((meta >> 16) & 3u) << ((j >> 1) * 2 + (j & 1) * 32);
        const LAS u32x4* sb = (const LAS u32x4*)(lds + BIG_OFF + j * 4096 + lane * 16);
        const u32x4 v0 = sb[0], v1 = sb[64], v2 = sb[128], v3 = sb[192];
        unsigned char* d = dst + blk_off(rg0 + (j & 1), j >> 1);
        if (j >= NSM) {
          st16_sys(d, v0);
          wait_vm<4>();
          st16_sys(d + 1024, v1);
          wait_vm<4>();
        }
        st16_sys(d + 2048, v2);
        if (j < NSM) wait_vm<LIGHT>(); else wait_vm<4>();
        st16_sys_lanes(d + 3072, v3, meta & 0xff);
        if (j < NSM && (j < 3 || !(j & 1))) wait_vm<LIGHT>(); else wait_vm<4>();
      } else {
        wait_vm<0>();
      }
      if (j < NSM ? j == 0 || (j >= 3 && (j & 1)) : !(j & 1)) {
        u32x2 fv = *(const LAS u32x2*)(lds + TAB_OFF + 128 + (lane & 3) * 8);
        if (lane < 2) fv = u32x2{fbase + (j >> 1), (unsigned)(cls >> (lane * 32))};
        if (lane < 4) st8_sys(flag, fv);
      }
      DBG(2 + (j >> 1));
    }
    DBG(18);
    DBG_FLUSH();
    return;
  }

  const int cb = (wg >> 3) & 3;
  const bool local = wave == 9;
  const int src = local ? p.rank : (p.rank + grp) & 7;

  if (wave > 9) {
    unsigned* cnt = (unsigned*)(my + CNT_OFF + (size_t)wg * 128) + wave;
    unsigned call = __hip_atomic_load(cnt, __ATOMIC_RELAXED, __HIP_MEMORY_SCOPE_AGENT);
    call = __builtin_amdgcn_readfirstlane(call) + 1;
    if (lane == 0) __hip_atomic_store(cnt, call, __ATOMIC_RELAXED, __HIP_MEMORY_SCOPE_AGENT);
    const unsigned fbase = (call - 1) * NKC;
    const int rg0 = (wg & 7) * 4;
    const size_t aoff = (size_t)(call & 1) * PAR_BYTES + (size_t)src * SHARD_BYTES + lane * 16;
    const unsigned char* lines = my + FLAG_OFF + (size_t)(call & 1) * FLAG_PAR + (size_t)src * (NRG / 2) * 128;
    LAS unsigned char* ctl = lds + CTL_A;
    auto slot_busy = [&](int r, int kc) { return __any(lds_ld(ctl, r * 32 + 24 + (lane & 1) * 4) + NAS <= (unsigned)turn_of(kc)); };
    bool bad = false;
    unsigned long long t0 = __builtin_amdgcn_s_memrealtime();
    auto give_up = [&]() {
      const unsigned long long dt = __builtin_amdgcn_s_memrealtime() - t0;
      return dt > 10000 && (dt > DEADLINE_TICKS ||
                            (unsigned)__builtin_amdgcn_readfirstlane(__hip_atomic_load((const unsigned*)(my + TMO_OFF), __ATOMIC_RELAXED,
                                                                                      __HIP_MEMORY_SCOPE_AGENT)) == call);
    };
    if (wave == 10) {
      const u64* pollp = (const u64*)(lines + ((wg & 7) * 2 + ((lane >> 2) & 1)) * 128 + (lane & 3) * 8);
      int fk[2] = {0, 0};
      u64 pv = __hip_atomic_load(pollp, __ATOMIC_RELAXED, __HIP_MEMORY_SCOPE_SYSTEM);
      while (fk[0] < NP || fk[1] < NP) {
        const unsigned cls = (unsigned)(pv >> 32);
        bool any = false;
#pragma unroll
        for (int pr = 0; pr < 2; ++pr) {
          const int kc = fk[pr];
          if (kc < NP && __all((lane >> 1) != pr * 2 || (int)((unsigned)pv - fbase) > kc) && !slot_busy(pr * 2, kc) &&
              !slot_busy(pr * 2 + 1, kc)) {
            CBAR();
#pragma unroll
            for (int r = pr * 2; r < pr * 2 + 2; ++r) {
              const unsigned nl = tail_lanes(((unsigned)__builtin_amdgcn_readlane(cls, pr * 2 + r) >> (2 * kc)) & 3u);
              LAS unsigned char* dst = lds + ABUF_OFF + (r * NAS + turn_of(kc) % NAS) * 4096;
              const unsigned char* a = my + aoff + blk_off(rg0 + r, kc);
#pragma unroll
              for (int pc = 0; pc < 3; ++pc)
                __builtin_amdgcn_global_load_lds((const GAS void*)(a + pc * 1024), (LAS void*)(dst + pc * 1024), 16, 0, 17);
              if ((unsigned)lane < nl) __builtin_amdgcn_global_load_lds((const GAS void*)(a + 3072), (LAS void*)(dst + 3072), 16, 0, 17);
            }
            if (pr == 0) DBG(8 + kc);
            ++fk[pr];
            any = true;
          }
        }
        if (any) {
          t0 = __builtin_amdgcn_s_memrealtime();
        } else if (give_up()) {
          bad = true;
          break;
        }
        pv = __hip_atomic_load(pollp, __ATOMIC_RELAXED, __HIP_MEMORY_SCOPE_SYSTEM);
        wait_vm<0>();
        if (any) {
          if (lane < 8 && (lane & 2)) *(volatile LAS u64*)(lds + CTL_T + (lane >> 2) * 16 + (lane & 1) * 8) = pv;
          CBAR();
          if (lane < 8 && !(lane & 2))
            *(volatile LAS u64*)(ctl + ((lane >> 1) + (lane & 1)) * 32) = ((u64)cls << 32) | (unsigned)(lane < 4 ? fk[0] : fk[1]);
          DBG(fk[0] < fk[1] ? fk[0] : fk[1]);
        }
      }
    } else {
      const unsigned char* flags = my + FLAG_OFF + (size_t)(call & 1) * FLAG_PAR;
      const int sw = ((wg & 7) * 2 + ((lane >> 1) & 1)) * 128 + 64 + (lane & 1) * 32;
      int phase = 0;
      unsigned cls = 0, tgo = 0;
      constexpr int NB = 4 * (NKC - NP);
      unsigned d_src = 0, d_tail = 0, d_lds = 0, d_due = 0;
      while (true) {
        const bool on = phase == 0 ? lane >= 16 && lane < 16 + 4 * NPART : phase == 1 ? lane < 56 : lane < 16;
        const unsigned char* pp = phase == 0 ? lines + sw + ((lane >> 2) & 3) * 8
                                  : phase == 1 ? flags + ((p.rank + 1 + (lane >> 3)) & 7) * (NRG / 2) * 128 + sw + ((lane >> 2) & 1) * (NPART - 1) * 8
                                               : lines + lane * 128;
        u64 pv = 0;
        if (on) pv = __hip_atomic_load((const u64*)pp, __ATOMIC_RELAXED, __HIP_MEMORY_SCOPE_SYSTEM);
        if (phase < 2) {
          if (__all(!on || (int)((unsigned)pv - call) >= 0)) {
            if (phase == 0) {
              DBG(1);
              cls = on ? (unsigned)(pv >> 32) : 0u;
              cls |= __builtin_amdgcn_update_dpp(0u, cls, 0x124, 0xf, 0xf, false);
              cls |= __builtin_amdgcn_update_dpp(0u, cls, 0x128, 0xf, 0xf, false);
              __builtin_amdgcn_fence(__ATOMIC_ACQUIRE, "");
              wait_vm<0>();
              DBG(2);
              const int kc = NP + (lane >> 2), r = lane & 3;
              const unsigned nl = tail_lanes(((unsigned)__shfl(cls, 16 + r) >> (2 * kc)) & 3u);
              d_src = (unsigned)((call & 1) * PAR_BYTES + src * SHARD_BYTES) + blk_off(rg0 + r, kc);
              d_tail = nl == 20 ? (unsigned)(PTAIL_OFF + (call & 1) * PTAIL_PAR) + ((rg0 + r) * NKC + kc) * 320 - 3072 : d_src;
              d_lds = (ABUF_OFF + (r * NAS + turn_of(kc) % NAS) * 4096) | (nl - 1) | (turn_of(kc) << 24);
              const unsigned by = (192 + nl) * PACE;
              d_due = (wave_scan(by) - by) >> 10;
            }
            tgo = (unsigned)__builtin_amdgcn_s_memrealtime() + GATE;
            ++phase;
            continue;
          }
        } else {
          if (__builtin_popcountll(__ballot(on && (int)((unsigned)pv - fbase) >= NP - 1)) >= TSUM &&
              (int)((unsigned)__builtin_amdgcn_s_memrealtime() - tgo) >= 0)
            break;
        }
        if (give_up()) {
          bad = true;
          break;
        }
        __builtin_amdgcn_s_sleep(PSLP);
      }
      CBAR();
      DBG(3);
      __builtin_amdgcn_s_setprio(3);
      const u64 rbv = (u64)p.ws[src];
      const unsigned char* rb = (const unsigned char*)((u64)(unsigned)__builtin_amdgcn_readfirstlane((unsigned)rbv) |
                                                       ((u64)(unsigned)__builtin_amdgcn_readfirstlane((unsigned)(rbv >> 32)) << 32));
      const unsigned lo16 = lane * 16;
      int pub = 0;
      auto landed = [&](int n) {
        pub = n > pub ? n : pub;
        if ((lane >> 2) == 4)
          *(volatile LAS u64*)(ctl + (lane & 3) * 32 + 8) = ((u64)cls << 32) | (unsigned)(NP + ((pub + 3 - (lane & 3)) >> 2));
      };
      const unsigned tp0 = (unsigned)__builtin_amdgcn_s_memrealtime() - PLEAD;
      const unsigned char *na, *nt;
      unsigned nl, nmode;
      auto plan = [&](int i) {
        const unsigned us = lds_ld(ctl, (i & 3) * 32 + 24 + (lane & 1) * 4);
        na = rb + (unsigned)__builtin_amdgcn_readlane(d_src, i);
        nt = rb + (unsigned)__builtin_amdgcn_readlane(d_tail, i);
        nl = __builtin_amdgcn_readlane(d_lds, i);
        const unsigned slack = (unsigned)__builtin_amdgcn_readlane(d_due, i) - PAHEAD - ((unsigned)__builtin_amdgcn_s_memrealtime() - tp0);
        nmode = (slack >> 31) | (((slack + FARB) >> 31) << 1);
        nmode |= (unsigned)__ballot(us + NAS <= (nl >> 24)) << 2;
        nmode = __builtin_amdgcn_readfirstlane(nmode);
        asm volatile("" : "+s"(na), "+s"(nt), "+s"(nl), "+s"(nmode));
      };
      auto turn = [&](int i, auto win) {
        constexpr int W = decltype(win)::value;
        const unsigned char *a = na, *t = nt;
        const unsigned l = nl;
        LAS unsigned char* dst = lds + (l & 0xfff000u);
        __builtin_amdgcn_global_load_lds((const GAS void*)(a + lo16), (LAS void*)dst, 16, 0, 0);
        plan(i + 1);
        wait_vm<W>();
        __builtin_amdgcn_global_load_lds((const GAS void*)(a + lo16), (LAS void*)dst, 16, 1024, 0);
        if (0 == ((W - 1) & 3)) landed(i - (W - 1) / 4);
        wait_vm<W>();
        __builtin_amdgcn_global_load_lds((const GAS void*)(a + lo16), (LAS void*)dst, 16, 2048, 0);
        if (1 == ((W - 1) & 3)) landed(i - (W - 1) / 4);
        wait_vm<W>();
        if ((unsigned)lane <= (l & 63u)) __builtin_amdgcn_global_load_lds((const GAS void*)(t + lo16), (LAS void*)dst, 16, 3072, 0);
        if (2 == ((W - 1) & 3)) landed(i - (W - 1) / 4);
        wait_vm<W>();
        if (3 == ((W - 1) & 3)) landed(i - (W - 1) / 4);
        if ((i & 3) == 3) DBG(NP + (i >> 2));
      };
      if (!bad) {
        plan(0);
#pragma unroll 1
        for (int i = 0; i < NB; ++i) {
          if (nmode > 3) {
            const int kc = NP + (i >> 2);
            if (slot_busy(i & 3, kc)) {
              wait_vm<0>();
              landed(i);
              while (slot_busy(i & 3, kc)) __builtin_amdgcn_s_sleep(1);
            }
            CBAR();
          }
          if (nmode & 2) turn(i, std::integral_constant<int, FWIN>{});
          else if (nmode & 1) turn(i, std::integral_constant<int, EWIN>{});
          else turn(i, std::integral_constant<int, PWIN>{});
        }
        wait_vm<8>();
        landed(NB - 2);
        wait_vm<4>();
        landed(NB - 1);
        wait_vm<0>();
        landed(NB);
      }
    }
    if (bad) {
      if (lane == 0) __hip_atomic_store((unsigned*)(my + TMO_OFF), call, __ATOMIC_RELAXED, __HIP_MEMORY_SCOPE_AGENT);
      if (lane < 4) *(volatile LAS u64*)(ctl + lane * 32 + (wave - 10) * 8) = 0xffffffff80000000ull;
    }
    DBG(18);
    DBG_FLUSH();
    if (wave == 11) return;
  }

  if (wave == 4 || wave == 10) {
    if (wave == 4 && part < NPART) __builtin_amdgcn_s_sleep(QUIET);
    const int kk = lane >> 3, c = lane & 7;
    const unsigned char* wb = p.w + (size_t)kk * NL + cb * 128;
    const int f0 = (kk >> 1) & 3;
    const unsigned char* s0 = wb + ((c ^ f0) << 4);
    const unsigned char* s1 = wb + ((c ^ (f0 | 4)) << 4);
    int prev = -1;
    auto filled = [&]() {
      if (prev >= 0) lds_st(lds, CTL_FULL + (wave == 4 ? prev & 1 : 2) * 4, (prev >> 1) + 1);
    };
#pragma unroll 1
    for (int s = wave == 4 ? 0 : 2 * WT0 + 1; s < 2 * NKC; s += s < 2 * WT0 ? 1 : 2) {
      if (s >= NWS) {
        const unsigned need = (s >> 1) - WCH + 1;
        bool first = true;
        while (true) {
          unsigned d = lds_ld(lds, CTL_DONE + (lane & 7) * 4);
          if (__all(d >= need)) break;
          if (first) {
            wait_vm<0>();
            filled();
            first = false;
          }
          __builtin_amdgcn_s_sleep(WSLP);
        }
        CBAR();
      }
      LAS unsigned char* dst = lds + RING_OFF + (s % NWS) * SLOT_BYTES;
      const size_t koff = (size_t)(chunk_of(s >> 1) * 2 + (s & 1)) * 128 * NL;
#pragma unroll
      for (int j = 0; j < 16; ++j) {
        const unsigned char* gp = (((j >> 1) & 1) ? s1 : s0) + koff + (size_t)j * 8 * NL;
        __builtin_amdgcn_global_load_lds((const GAS void*)gp, (LAS void*)(dst + j * 1024), 16, 0, 0);
      }
      wait_vm<16>();
      filled();
      prev = s;
    }
    wait_vm<0>();
    filled();
    return;
  }

#define SB() __builtin_amdgcn_sched_barrier(0)
#define PH(i) do {} while (0)
  if (!local && wave > 4) {
    const int gw = wave - 5;
    CBAR();
    LAS unsigned char* ctl = lds + CTL_A + gw * 32;
    LAS unsigned char* tab = lds + CTL_T + (gw >> 1) * 16;
    const unsigned e3 = SCR_OFF + gw * SCR_BYTES;
    __builtin_amdgcn_s_setprio(UPRIO);
    int k0 = 0, k1 = NP;
#pragma unroll 1
    while (k0 < NP || k1 < NKC) {
      const u32x4 st = *(volatile LAS u32x4*)ctl;
      const bool a0 = k0 < NP && (unsigned)__builtin_amdgcn_readfirstlane(st[0]) > (unsigned)k0;
      if (!a0 && !(k1 < NKC && k0 >= 1 && (unsigned)__builtin_amdgcn_readfirstlane(st[2]) > (unsigned)k1)) {
        __builtin_amdgcn_s_setprio(0);
        __builtin_amdgcn_s_sleep(USLP);
        continue;
      }
      __builtin_amdgcn_s_setprio(UPRIO);
      CBAR();
      const int kc = a0 ? k0++ : k1++;
      if (kc == 12) DBG(20);
      const unsigned cl = ((unsigned)__builtin_amdgcn_readfirstlane(a0 ? st[1] : st[3]) >> (2 * kc)) & 3u;
      const unsigned boff = ABUF_OFF + (gw * NAS + turn_of(kc) % NAS) * 4096;
      LAS unsigned char* blk = lds + boff;
      if (cl == 3) {
        if (a0) {
          u32x4 v[4];
#pragma unroll
          for (int pc = 0; pc < 4; ++pc) v[pc] = *(LAS u32x4*)(blk + pc * 1024 + lane * 16);
          CBAR();
#pragma unroll
          for (int pc = 0; pc < 4; ++pc) {
            u32x4 o;
#pragma unroll
            for (int k = 0; k < 2; ++k) {
              const unsigned sm = v[pc >> 1][(pc & 1) * 2 + k], ex = v[2 + (pc >> 1)][(pc & 1) * 2 + k];
              const unsigned t = (ex & SM_MASK) | (sm & ~SM_MASK);
              o[2 * k] = (sm & SM_MASK) | (ex & ~SM_MASK);
              o[2 * k + 1] = __builtin_amdgcn_alignbit(t, t, 4);
            }
            *(LAS u32x4*)(blk + pc * 1024 + lane * 16) = o;
          }
          CBAR();
          lds_st(ctl, 16, kc + 1);
        }
        continue;
      }
      PH(0);
      {
        const unsigned l3 = *(LAS unsigned short*)(blk + TAIL + tail_lanes(cl) * 16 - 4 - (lane >> 1) * 4 + (lane & 1) * 2);
        const unsigned T3 = *(LAS unsigned*)(tab + 8);
        const unsigned l2a = *(LAS unsigned*)(blk + TAIL + lane * 4);
        const u32x4 c1 = *(LAS u32x4*)(blk + 2048 + lane * 16);
        const u32x2 T12 = *(LAS u32x2*)(tab);
        const u32x4 sm0 = *(LAS u32x4*)(blk + lane * 16), sm1 = *(LAS u32x4*)(blk + 1024 + lane * 16);
        const unsigned l2b = *(LAS unsigned*)(blk + TAIL + 256 + lane * 4);
        SB();
        PH(1);
        auto lut2 = [&](unsigned c, int i) { return *(LAS u32x2*)(lds + LUT_OFF + ((c >> (8 * i)) & 255u) * 8); };
        auto window = [&](unsigned off) {
          const LAS unsigned* w = (const LAS unsigned*)(lds + (off & ~3u));
          return u32x2{w[0], w[1]};
        };
        auto pick = [&](u32x2 w, unsigned off, unsigned T, unsigned sel) {
          return __builtin_amdgcn_perm(__builtin_amdgcn_alignbyte(w[1], w[0], off), T, sel);
        };
        {
          const unsigned t = l3 | (l3 << 12);
          const unsigned u = t & 0x000f000fu, v = (t >> 8) & 0x000f000fu;
          *(LAS u32x2*)(lds + e3 + lane * 8) = u32x2{__builtin_amdgcn_perm(0u, T3, (u | (u << 6)) & 0x03030303u),
                                                     __builtin_amdgcn_perm(0u, T3, (v | (v << 6)) & 0x03030303u)};
        }
        u32x2 s2[4], s1[16];
#pragma unroll
        for (int i = 0; i < 4; ++i) s2[i] = lut2(l2a, i);
        SB();
        const unsigned mine = (esc_count(c1[0]) + esc_count(c1[1]) + esc_count(c1[2]) + esc_count(c1[3])) | (esc_count(l2a) << 16);
#pragma unroll
        for (int i = 0; i < 8; ++i) s1[i] = lut2(c1[i >> 2], i & 3);
        const unsigned inc = wave_scan(mine);
        const unsigned exc = inc - mine;
        const unsigned tot = __builtin_amdgcn_readlane(inc, 63);
        CBAR();
        SB();
        PH(2);
        unsigned o2[4], o1[16];
        u32x2 w2[4], w1[16];
        o2[0] = e3 + (exc >> 16);
#pragma unroll
        for (int i = 1; i < 4; ++i) o2[i] = o2[i - 1] + s2[i - 1][1];
#pragma unroll
        for (int i = 0; i < 4; ++i) w2[i] = window(o2[i]);
        SB();
#pragma unroll
        for (int i = 8; i < 16; ++i) s1[i] = lut2(c1[i >> 2], i & 3);
        SB();
        PH(3);
        {
          auto put = [&](const u32x4& o, LAS unsigned char* to) {
            const unsigned nxt = __builtin_amdgcn_update_dpp(0u, o[0], 0x130, 0xf, 0xf, true);
            *(LAS u32x4*)(to + lane * 32) = u32x4{o[0], o[1], o[1], o[2]};
            *(LAS u32x4*)(to + lane * 32 + 16) = u32x4{o[2], o[3], o[3], nxt};
          };
          u32x4 o;
#pragma unroll
          for (int i = 0; i < 4; ++i) o[i] = pick(w2[i], o2[i], T12[1], s2[i][0]);
          put(o, blk);
          if ((tot & 0xffff) > 1024) {
            const unsigned m2 = esc_count(l2b);
            unsigned off = e3 + (tot >> 16) + wave_scan(m2) - m2;
#pragma unroll
            for (int i = 0; i < 4; ++i) s2[i] = lut2(l2b, i);
#pragma unroll
            for (int i = 0; i < 4; ++i) { o[i] = pick(window(off), off, T12[1], s2[i][0]); off += s2[i][1]; }
            put(o, blk + 2048);
            if (lane == 0) *(LAS unsigned*)(blk + 2044) = o[0];
          }
        }
        CBAR();
        PH(4);
        o1[0] = boff / 2 - 4 + (exc & 0xffff);
#pragma unroll
        for (int i = 0; i < 16; ++i) {
          if (i) o1[i] = o1[i - 1] + s1[i - 1][1];
          w1[i] = *(const LAS u32x2*)(lds + ((o1[i] << 1) | 7u) + 1);
        }
        CBAR();
        SB();
        PH(5);
#pragma unroll
        for (int pc = 0; pc < 4; ++pc) {
          u32x4 o;
#pragma unroll
          for (int k = 0; k < 4; ++k) {
            const int i = pc * 4 + k;
            const unsigned s = i < 8 ? sm0[i >> 1] : sm1[(i >> 1) & 3];
            o[k] = (((i & 1) ? __builtin_amdgcn_alignbit(s, s, 4) : s) & SM_MASK) | pick(w1[i], o1[i], T12[0], s1[i][0]);
          }
          *(LAS u32x4*)(blk + pc * 1024 + lane * 16) = o;
        }
      }
      CBAR();
      lds_st(ctl, a0 ? 16 : 20, kc + 1);
      PH(6);
      if (kc == 12) DBG(21);
      DBG(1 + kc);
    }
    DBG_FLUSH();
    return;
  }

  const int gw = local ? 4 : wave;
  const int task = local ? grp - 1 : wave;
  const int ch = task & 1;
  const int rg0 = (wg & 7) * 4 + (task >> 1) * 2;
  const int q = r16 >> 1;
  const int wl = RING_OFF + (g * 16 + q) * 128 + (((((q >> 1) & 3) | ((g & 1) << 2)) ^ (ch * 4)) << 4) + (r16 & 1) * 8;
  const __amdgpu_buffer_rsrc_t ors = __builtin_amdgcn_make_buffer_rsrc((void*)p.out, 0, NR * MS * NL * 4, 0x00020000);
  const int ooff = ((src * MS + rg0 * 16 + (r16 & 7)) * NL + cb * 128 + ch * 64 + (r16 >> 3) * 16 + g * 4) * 4;
  const unsigned char* xs = p.x + (size_t)(rg0 * 16 + r16) * KD + g * 16;

  auto gemm = [&](auto tag) {
    constexpr bool LOCAL = decltype(tag)::value;
    f32x4 acc[8];
#pragma unroll
    for (int i = 0; i < 8; ++i) acc[i] = f32x4{0.f, 0.f, 0.f, 0.f};
    u32x4 nx[8];
    if constexpr (LOCAL) {
#pragma unroll
      for (int i = 0; i < 8; ++i) nx[i] = *(const u32x4*)(xs + (size_t)(i >> 2) * (16 * KD) + (i & 3) * 64);
    }
    const float sc = p.sx[0] * p.sw[0];
    __builtin_amdgcn_s_setprio(GPRIO);
    unsigned badm = 0;
    int slot = 0, aslot = 0;
    LAS unsigned char* ctl = lds + CTL_A + (wave & 2) * 32;
    const int so = lane < 16 ? CTL_A + (wave & 2) * 32 + lane * 4 : CTL_FULL + (lane & 3) * 4;
    unsigned sv = lds_ld(lds, so);
#pragma unroll 1
    for (int turn = 0; turn < NKC; ++turn) {
      const int kc = chunk_of(turn);
      const unsigned thr = lane < 16 ? kc : turn;
      const int pl = kc < NP ? 0 : 1;
      auto ready = [&]() {
        const unsigned gt = (unsigned)__ballot(sv > thr);
        const unsigned to = (unsigned)__ballot((int)sv < 0) >> (2 * pl);
        const unsigned raw = pl ? (gt >> 2) & ((unsigned)__ballot(((sv >> (2 * kc)) & 3u) == 3u) >> 3) : 0u;
        badm |= to;
        return (((gt >> (4 + pl)) | to | raw) & 0x101u) | (gt & (gt >> (turn < WT0 ? 1 : 2)) & 0x10000u);
      };
      unsigned rd = ready();
      auto await = [&](unsigned bit) {
        while (!(rd & bit)) {
          if (turn < NKC - 2) __builtin_amdgcn_s_sleep(GSLP); else __builtin_amdgcn_s_sleep(1);
          sv = lds_ld(lds, so);
          rd = ready();
        }
      };
      auto x_read = [&](int r) {
        CBAR();
        LAS unsigned char* blk = lds + ABUF_OFF + (((wave & 2) + r) * NAS + aslot) * 4096;
#pragma unroll
        for (int pc = 0; pc < 4; ++pc) nx[r * 4 + pc] = *(LAS u32x4*)(blk + pc * 1024 + lane * 16);
        CBAR();
        lds_st(ctl, r * 32 + 24 + ch * 4, turn + 1);
      };
      auto store = [&](int r) {
        const float s = !LOCAL && (badm & 0x101u) ? __builtin_nanf("") : sc;
#pragma unroll
        for (int t = 0; t < 2; ++t) {
          const u32x4 a = __builtin_bit_cast(u32x4, acc[r * 4 + 2 * t] * s), b = __builtin_bit_cast(u32x4, acc[r * 4 + 2 * t + 1] * s);
          u32x4 lo, hi;
#pragma unroll
          for (int i = 0; i < 4; ++i) {
            lo[i] = __builtin_amdgcn_update_dpp(a[i], b[i], 0x128, 0xf, 0xc, false);
            hi[i] = __builtin_amdgcn_update_dpp(b[i], a[i], 0x128, 0xf, 0x3, false);
          }
          __builtin_amdgcn_raw_buffer_store_b128(lo, ors, ooff + t * 128, r * 16 * NL * 4, OUT_AUX);
          __builtin_amdgcn_raw_buffer_store_b128(hi, ors, ooff + t * 128, (r * 16 + 8) * NL * 4, OUT_AUX);
        }
      };
      await(0x10000u);
      CBAR();
      if (turn == 9) DBG(19);
      i32x8 wa[8];
#pragma unroll
      for (int f = 0; f < 8; ++f) {
        LAS unsigned char* a = lds + slot * (2 * SLOT_BYTES) + (wl ^ ((f & 3) << 4)) + (f >> 2) * SLOT_BYTES;
        const i32x2 t0 = __builtin_amdgcn_ds_read_tr8_b64_v2i32((LAS i32x2*)(a));
        const i32x2 t1 = __builtin_amdgcn_ds_read_tr8_b64_v2i32((LAS i32x2*)(a + 1024));
        const i32x2 t2 = __builtin_amdgcn_ds_read_tr8_b64_v2i32((LAS i32x2*)(a + 8192));
        const i32x2 t3 = __builtin_amdgcn_ds_read_tr8_b64_v2i32((LAS i32x2*)(a + 9216));
        wa[f][0] = t0[0]; wa[f][1] = t0[1]; wa[f][2] = t1[0]; wa[f][3] = t1[1];
        wa[f][4] = t2[0]; wa[f][5] = t2[1]; wa[f][6] = t3[0]; wa[f][7] = t3[1];
      }
      CBAR();
      lds_st(lds, CTL_DONE + gw * 4, turn + 1);
      auto mma = [&](int r) {
#pragma unroll
        for (int f = 0; f < 8; ++f) {
          i32x8 xb;
#pragma unroll
          for (int i = 0; i < 8; ++i) xb[i] = nx[r * 4 + (f >> 2) * 2 + (i >> 2)][i & 3];
          acc[r * 4 + (f & 3)] = __builtin_amdgcn_mfma_scale_f32_16x16x128_f8f6f4(wa[f], xb, acc[r * 4 + (f & 3)], 1, 0, 0, 0, 0, 0);
        }
      };
      if constexpr (LOCAL) {
        sv = lds_ld(lds, so);
#pragma unroll
        for (int r = 0; r < 2; ++r) {
          mma(r);
          SB();
#pragma unroll
          for (int pc = 0; pc < 4; ++pc)
            nx[r * 4 + pc] = *(const u32x4*)(xs + (size_t)r * (16 * KD) + chunk_of((turn + 1) & (NKC - 1)) * 256 + pc * 64);
          SB();
        }
        if (turn == NKC - 1) {
          DBG(17);
          store(0);
          store(1);
        }
      } else {
        await(1u);
        x_read(0);
        const bool both = rd & 0x100u;
        if (both) x_read(1);
        sv = lds_ld(lds, so);
        SB();
        mma(0);
        if (turn == NKC - 1) store(0);
        if (!both) {
          rd = ready();
          await(0x100u);
          if (turn == 15) DBG(1);
          x_read(1);
        }
        mma(1);
        if (turn == NKC - 1) {
          DBG(17);
          store(1);
        }
      }
      slot = slot == WCH - 1 ? 0 : slot + 1;
      aslot = aslot == NAS - 1 ? 0 : aslot + 1;
      if (turn < 15) DBG(2 + turn);
    }
  };
  if (local) gemm(std::true_type{}); else gemm(std::false_type{});
}

}

extern "C" void kernel_launch(int rank, int nranks, ncclComm_t comm, hipStream_t stream,
                              void* const* d_in, const int* in_sizes, int n_in, void* d_out,
                              int out_size, const int* in_dtypes, int out_dtype,
                              void* const* d_out_all, void* const* d_ws_all, size_t ws_bytes) {
  (void)comm; (void)in_dtypes; (void)out_dtype; (void)d_out_all;
  if (nranks != NR || n_in != 4 || (unsigned)rank >= (unsigned)NR || ws_bytes < WS_NEED ||
      in_sizes[0] != MS * KD || in_sizes[1] != KD * NL || out_size != NR * MS * NL) {
    fprintf(stderr, "kernel_launch: unexpected problem shape\n");
    return;
  }
  Params p;
  p.x = (const unsigned char*)d_in[0];
  p.w = (const unsigned char*)d_in[1];
  p.sx = (const float*)d_in[2];
  p.sw = (const float*)d_in[3];
  p.out = (float*)d_out;
  for (int i = 0; i < NR; ++i) p.ws[i] = (unsigned char*)d_ws_all[i];
  p.my = p.ws[rank];
  p.rank = rank;
  p.pad = 0;
  hipLaunchKernelGGL(ag_gemm_kernel, dim3(NWG), dim3(NWAVE * 64), 0, stream, p);
  hipError_t e = hipGetLastError();
  if (e != hipSuccess) fprintf(stderr, "launch error %s\n", hipGetErrorString(e));
}
```

```cpp
#include <hip/hip_runtime.h>
#include <rccl/rccl.h>
#include <cstdio>
#include <cstdint>
#include <type_traits>

namespace {

constexpr int NR = 8;
constexpr int MS = 512;
constexpr int KD = 4096;
constexpr int NL = 512;
constexpr int NKC = 16;
constexpr int NRG = 32;
constexpr int NWG = 256;
constexpr int NWAVE = 12;
constexpr int NP = 4;
constexpr int NPART = (NKC - NP + 3) / 4;
constexpr size_t SHARD_BYTES = (size_t)MS * KD;
constexpr size_t PAR_BYTES = NR * SHARD_BYTES;
constexpr size_t FLAG_OFF = 2 * PAR_BYTES;
constexpr size_t FLAG_PAR = (size_t)NR * (NRG / 2) * 128;
constexpr size_t CNT_OFF = FLAG_OFF + 2 * FLAG_PAR;
constexpr size_t TMO_OFF = CNT_OFF + (size_t)NWG * 128;
constexpr size_t PTAIL_OFF = TMO_OFF + 4096;
constexpr size_t PTAIL_PAR = (size_t)NRG * NKC * 320;
constexpr size_t WS_NEED = PTAIL_OFF + 2 * PTAIL_PAR;
constexpr unsigned long long DEADLINE_TICKS = 50000000ull;

constexpr int LUT_OFF = 0;
constexpr int CTL_OFF = 2048;
constexpr int SCR_OFF = CTL_OFF + 512;
constexpr int SCR_BYTES = 512;
constexpr int TAB_OFF = SCR_OFF + 2048;
constexpr int ELUT_OFF = TAB_OFF + 1536;
constexpr int ETAIL_OFF = ELUT_OFF + 2048;
constexpr int HSMP_OFF = ETAIL_OFF + 4096;
constexpr int BIG_OFF = 16384;
constexpr int WCH = 2, NWS = 2 * WCH;
constexpr int NAS = 5;
constexpr int SLOT_BYTES = 16384;
constexpr int RING_OFF = BIG_OFF;
constexpr int ABUF_OFF = RING_OFF + NWS * SLOT_BYTES;
constexpr int LDS_BYTES = ABUF_OFF + 4 * NAS * 4096;
static_assert(HSMP_OFF + 1024 <= BIG_OFF && LDS_BYTES <= 163840, "LDS map");
constexpr int CTL_FULL = CTL_OFF;
constexpr int CTL_DONE = CTL_OFF + 32;
constexpr int CTL_A = CTL_OFF + 64;
constexpr int CTL_T = CTL_OFF + 192;
constexpr int CTL_E = CTL_OFF + 224;
constexpr int CTL_META = CTL_OFF;
constexpr int CTL_CALL = CTL_OFF + 128;
constexpr int RAW_OFF = BIG_OFF + 2 * NP * 4096;
constexpr int TAIL = 3072;

typedef unsigned u32x4 __attribute__((ext_vector_type(4)));
typedef unsigned u32x3 __attribute__((ext_vector_type(3)));
typedef unsigned u32x2 __attribute__((ext_vector_type(2)));
typedef int i32x2 __attribute__((ext_vector_type(2)));
typedef int i32x8 __attribute__((ext_vector_type(8)));
typedef float f32x4 __attribute__((ext_vector_type(4)));
typedef unsigned long long u64;
#define LAS __attribute__((address_space(3)))
#define GAS __attribute__((address_space(1)))

struct Params {
  const unsigned char* x;
  unsigned char* ws[NR];
  int rank;
  int pad;
  const unsigned char* w;
  const float* sx;
  const float* sw;
  float* out;
};

__device__ __forceinline__ void st16_sys(unsigned char* p, u32x4 v) {
  asm volatile("global_store_dwordx4 %0, %1, off sc0 sc1\n\ts_nop 1" ::"v"(p), "v"(v) : "memory");
}
__device__ __forceinline__ void st16_sys_lanes(unsigned char* p, u32x4 v, unsigned n) {
  asm volatile("s_mov_b64 exec, %2\n\tglobal_store_dwordx4 %0, %1, off sc0 sc1\n\ts_mov_b64 exec, -1\n\ts_nop 0"
               ::"v"(p), "v"(v), "s"(~0ull >> (64 - n)) : "memory");
}
__device__ __forceinline__ void st8_sys(unsigned char* p, u32x2 v) {
  asm volatile("global_store_dwordx2 %0, %1, off sc0 sc1" ::"v"(p), "v"(v) : "memory");
}
template <int N>
__device__ __forceinline__ void wait_vm() {
  asm volatile("s_waitcnt vmcnt(%0)" ::"i"(N) : "memory");
}
__device__ __forceinline__ unsigned lds_ld(LAS unsigned char* l, int off) {
  return *(volatile LAS unsigned*)(l + off);
}
__device__ __forceinline__ void lds_st(LAS unsigned char* l, int off, unsigned v) {
  *(volatile LAS unsigned*)(l + off) = v;
}
#define CBAR() asm volatile("" ::: "memory")

__device__ __forceinline__ unsigned wave_scan(unsigned v) {
  v += __builtin_amdgcn_update_dpp(0u, v, 0x111, 0xf, 0xf, true);
  v += __builtin_amdgcn_update_dpp(0u, v, 0x112, 0xf, 0xf, true);
  v += __builtin_amdgcn_update_dpp(0u, v, 0x114, 0xf, 0xf, true);
  v += __builtin_amdgcn_update_dpp(0u, v, 0x118, 0xf, 0xf, true);
  v += __builtin_amdgcn_update_dpp(0u, v, 0x142, 0xa, 0xf, false);
  v += __builtin_amdgcn_update_dpp(0u, v, 0x143, 0xc, 0xf, false);
  return v;
}
__device__ __forceinline__ unsigned esc_count(unsigned c) {
  return __builtin_popcount(c & (c >> 1) & 0x55555555u);
}

constexpr int NSM = 5;
constexpr int LIGHT = 7;
constexpr int TSUM = 8;
constexpr int GATE = 60;
constexpr int PWIN = 5, EWIN = 7, FWIN = 9;
constexpr int PACE = 214;
constexpr int PSLACK = 5, FARB = 40;
constexpr int WT0 = 7;

__device__ __forceinline__ unsigned blk_off(int rg, int kc) { return (unsigned)(rg * NKC + kc) * 4096u; }
__device__ __forceinline__ int chunk_of(int turn) { return turn == NP - 1 ? NP : turn == NP ? NP - 1 : turn; }
__device__ __forceinline__ int turn_of(int kc) { return chunk_of(kc); }
__device__ __forceinline__ unsigned tail_lanes(unsigned c) { return c == 0 ? 20u : c == 1 ? 24u : c == 2 ? 40u : 64u; }
constexpr unsigned SM_MASK = 0x87878787u;
__device__ __forceinline__ unsigned ex_pair(unsigned a, unsigned b) {
  return (a & ~SM_MASK) | (__builtin_amdgcn_alignbit(b, b, 28) & SM_MASK);
}
__device__ __forceinline__ unsigned sm_pair(unsigned a, unsigned b) {
  return (a & SM_MASK) | (__builtin_amdgcn_alignbit(b, b, 28) & ~SM_MASK);
}

__global__ void __launch_bounds__(NWAVE * 64, 3) ag_gemm_kernel(Params p) {
  __shared__ __attribute__((aligned(1024))) unsigned char smem[LDS_BYTES];
  LAS unsigned char* lds = (LAS unsigned char*)smem;
  const int wg = blockIdx.x;
  const int lane = threadIdx.x & 63;
  const int wave = __builtin_amdgcn_readfirstlane(threadIdx.x >> 6);
  const int grp = wg >> 5;
  u64 pc0;
  asm volatile("s_getpc_b64 %0" : "=s"(pc0));
  if (__builtin_expect(grp == 0 && wave > 8, 1)) {
    const unsigned char* pc = (const unsigned char*)(pc0 & ~255ull);
    const int i0 = ((wg >> 3) * 3 + wave - 9) * 2;
#pragma unroll
    for (int i = 0; i < 2; ++i)
      __builtin_amdgcn_global_load_lds((const GAS void*)(pc + (i0 + i) * 1024 + lane * 16), (LAS void*)(lds + LDS_BYTES - 1024), 16, 0, 0);
    __builtin_amdgcn_s_barrier();
    return;
  }
  unsigned char* my = p.ws[0];
  asm volatile("" ::"s"(p.x), "s"(my));
  const int g = lane >> 4, r16 = lane & 15;

  if (__builtin_expect(grp == 0 && wave == 0, 1)) {
    unsigned* cnt = (unsigned*)(my + CNT_OFF + (size_t)wg * 128);
    unsigned call;
    asm volatile("global_load_dword %0, %1, off sc1" : "=v"(call) : "v"(cnt) : "memory");
    const unsigned char* xs = p.x + (size_t)((wg & 15) * 32 + r16) * KD + g * 16;
    auto stage = [&](int i0, int i1) {
#pragma unroll 1
      for (int j = i0 / 4; j < i1 / 4; ++j) {
        const GAS void* a = (const GAS void*)(xs + (j & 1) * (16 * KD) + (j >> 1) * 256);
        LAS unsigned char* d = lds + RAW_OFF + j * 4096;
        __builtin_amdgcn_global_load_lds(a, (LAS void*)d, 16, 0, 0);
        __builtin_amdgcn_global_load_lds(a, (LAS void*)(d + 960), 16, 64, 0);
        __builtin_amdgcn_global_load_lds(a, (LAS void*)(d + 1920), 16, 128, 0);
        __builtin_amdgcn_global_load_lds(a, (LAS void*)(d + 2880), 16, 192, 0);
      }
    };
    stage(0, 4 * NSM);
    __builtin_amdgcn_s_barrier();
    asm volatile("s_waitcnt vmcnt(%1)" : "+v"(call) : "i"(4 * NSM - 4) : "memory");
    call = __builtin_amdgcn_readfirstlane(call) + 1;
    lds_st(lds, CTL_CALL, call);
    wait_vm<4 * NSM - 8>();
    lds_st(lds, CTL_CALL + 4, 2u);
    wait_vm<4 * NSM - 16>();
    lds_st(lds, CTL_CALL + 4, 4u);
    stage(4 * NSM, 32);
    wait_vm<32 - 4 * NSM>();
    lds_st(lds, CTL_CALL + 4, NSM);
    wait_vm<8>();
    lds_st(lds, CTL_CALL + 4, 6u);
    wait_vm<0>();
    lds_st(lds, CTL_CALL + 4, 8u);
    if (lane == 0) __hip_atomic_store(cnt, call, __ATOMIC_RELAXED, __HIP_MEMORY_SCOPE_AGENT);
    return;
  }
  static_assert(NKC - NP == 12, "three groups of four packers");
  const int part = grp - 1, pkc = NP + part * 4 + wave - 5;
  const bool packer = grp == 0 ? wave < 5 : (grp < 4 && wave > 4 && wave < 9);
  if (grp != 0 && packer) {
    const GAS void* a = (const GAS void*)(p.x + (size_t)((wg & 31) * 16 + r16) * KD + pkc * 256 + g * 16);
    LAS unsigned char* d = lds + RING_OFF + (NWS - 1) * SLOT_BYTES + (wave - 5) * 4096;
    __builtin_amdgcn_global_load_lds((const GAS void*)(p.x + (size_t)((wg & 30) * 16 + (lane >> 5) * 16 + r16) * KD + (g & 1) * 16),
                                     (LAS void*)(lds + HSMP_OFF), 16, 0, 0);
    __builtin_amdgcn_global_load_lds(a, (LAS void*)d, 16, 0, 0);
    __builtin_amdgcn_global_load_lds(a, (LAS void*)(d + 960), 16, 64, 0);
    __builtin_amdgcn_global_load_lds(a, (LAS void*)(d + 1920), 16, 128, 0);
    __builtin_amdgcn_global_load_lds(a, (LAS void*)(d + 2880), 16, 192, 0);
    __builtin_amdgcn_global_load_lds((const GAS void*)(my + CNT_OFF + wg * 128 + wave * 4), (LAS void*)(lds + ETAIL_OFF + (wave - 5) * 1024), 4, 0, 16);
  }
  if (wave == 9 && grp < 4) return;

  auto build_lut = [&](bool pack) {
    const int idx = (wave & 3) * 64 + lane;
    unsigned v = 0, n = 0;
#pragma unroll
    for (int b = 0; b < 4; ++b) {
      const unsigned dg = (idx >> (2 * b)) & 3;
      v |= (pack ? (dg == 3 ? 1u << (2 * n) : 0u) : (dg == 3 ? 4u + n : dg)) << (8 * b);
      n += dg == 3 ? 1u : 0u;
    }
    *(LAS u32x2*)(lds + (pack ? ELUT_OFF : LUT_OFF) + idx * 8) = u32x2{v, pack ? 2 * n : n};
    asm volatile("s_waitcnt lgkmcnt(0)" ::: "memory");
  };

  const unsigned peer = (unsigned)(1 + (wg >> 4) * 4 + wave - 5) & 7u;
  unsigned char* pw = p.ws[peer];
  const int rank = p.rank;
  {
    if (wave == 4) lds_st(lds, CTL_OFF + lane * 4, (grp != 0 && lane >= (grp < 4 ? 12 : 13) && lane < 16) ? 1000u : 0u);
    if (grp == 0 ? wave < 5 : wave < 9 && wave != 4) build_lut(wave < 5);
    asm volatile("s_waitcnt lgkmcnt(0)" ::"s"(pw), "s"(rank) : "memory");
    __builtin_amdgcn_s_barrier();
    CBAR();
  }

  if (__builtin_expect(grp == 0 && !packer, 1)) {
    const int pair = wg & 15, rg0 = pair * 2;
    if (peer == 0) return;
    __builtin_amdgcn_s_setprio(3);
    unsigned call;
    while ((call = __builtin_amdgcn_readfirstlane(lds_ld(lds, CTL_CALL))) == 0) {}
    const unsigned fbase = (call - 1) * NKC;
    unsigned char* dst = pw + (size_t)(call & 1) * PAR_BYTES + (size_t)rank * SHARD_BYTES + lane * 16;
    const size_t foff = FLAG_OFF + (size_t)(call & 1) * FLAG_PAR + ((size_t)rank * (NRG / 2) + pair) * 128;
    unsigned char* flag = pw + foff + (lane & 3) * 8;
    u64 cls = 0;
#pragma unroll 1
    for (int j = 0; j < NSM; ++j) {
      while (j && lds_ld(lds, CTL_CALL + 4) <= (unsigned)j) {}
      CBAR();
      const LAS u32x4* sb = (const LAS u32x4*)(lds + RAW_OFF + j * 4096 + lane * 16);
      const u32x4 v0 = sb[0], v1 = sb[64], v2 = sb[128], v3 = sb[192];
      const u32x4 s0 = {sm_pair(v0[0], v0[1]), sm_pair(v0[2], v0[3]), sm_pair(v1[0], v1[1]), sm_pair(v1[2], v1[3])};
      u32x4 s1 = {sm_pair(v2[0], v2[1]), sm_pair(v2[2], v2[3]), sm_pair(v3[0], v3[1]), sm_pair(v3[2], v3[3])};
      unsigned char* d = dst + blk_off(rg0 + (j & 1), j >> 1);
      asm volatile("" : "+v"(s1));
      wait_vm<4>();
      st16_sys(d, s0);
      wait_vm<4>();
      st16_sys(d + 1024, s1);
    }
#pragma unroll
    for (int j = 0; j <= 2 * NP; ++j) {
      if (j < 2 * NP) {
        unsigned meta;
        while ((meta = __builtin_amdgcn_readfirstlane(lds_ld(lds, CTL_META + j * 4))) == 0) __builtin_amdgcn_s_sleep(1);
        CBAR();
        cls |= (u64)((meta >> 16) & 3u) << ((j >> 1) * 2 + (j & 1) * 32);
        const LAS u32x4* sb = (const LAS u32x4*)(lds + BIG_OFF + j * 4096 + lane * 16);
        const u32x4 v0 = sb[0], v1 = sb[64], v2 = sb[128], v3 = sb[192];
        unsigned char* d = dst + blk_off(rg0 + (j & 1), j >> 1);
        if (j >= NSM) {
          st16_sys(d, v0);
          wait_vm<4>();
          st16_sys(d + 1024, v1);
          wait_vm<4>();
        }
        st16_sys(d + 2048, v2);
        if (j < NSM) wait_vm<LIGHT>(); else wait_vm<4>();
        st16_sys_lanes(d + 3072, v3, meta & 0xff);
        if (j < NSM && (j < 3 || j == 4)) wait_vm<LIGHT>(); else wait_vm<4>();
      } else {
        wait_vm<0>();
      }
      if (j == 0 || j == 3 || j == (NSM > 4 ? 5 : 4) || j == 6 || j == 8) {
        u32x2 fv = *(const LAS u32x2*)(lds + TAB_OFF + 128 + (lane & 3) * 8);
        if (lane < 2) fv = u32x2{fbase + (j >> 1), (unsigned)(cls >> (lane * 32))};
        if (lane < 4) st8_sys(flag, fv);
      }
    }
    return;
  }

  if (packer) {
    {
      const bool home = grp != 0;
      const int j0 = home ? pkc * 2 + (wg & 1) : wave - 1, jend = home ? j0 + 1 : 2 * NP;
      unsigned* cnt = (unsigned*)(my + CNT_OFF + (size_t)wg * 128) + wave;
      __builtin_amdgcn_s_setprio(2);
      if (home) wait_vm<5>();
      else while (lds_ld(lds, CTL_CALL + 4) < 2) {}
      CBAR();
      const u32x3 smp = *(const LAS u32x3*)(lds + (home ? HSMP_OFF + lane * 16 : RAW_OFF + (lane >> 5) * 4096 + (lane & 31) * 16));
      u32x4 C;
      {
        LAS unsigned char* tab = lds + TAB_OFF + wave * 128;
        u64 h = 0;
#pragma unroll
        for (int i = 0; i < 3; ++i) {
#pragma unroll
          for (int b = 0; b < 4; ++b) h += 1ull << (((smp[i] >> (8 * b + 3)) & 15u) * 4u);
        }
        u32x4 hb = {(unsigned)h & 0x0f0f0f0fu, (unsigned)(h >> 32) & 0x0f0f0f0fu,
                    ((unsigned)h >> 4) & 0x0f0f0f0fu, (unsigned)(h >> 36) & 0x0f0f0f0fu};
#pragma unroll
        for (int i = 0; i < 4; ++i) {
          hb[i] += __builtin_amdgcn_update_dpp(0u, hb[i], 0x121, 0xf, 0xf, false);
          hb[i] += __builtin_amdgcn_update_dpp(0u, hb[i], 0x122, 0xf, 0xf, false);
          hb[i] += __builtin_amdgcn_update_dpp(0u, hb[i], 0x124, 0xf, 0xf, false);
          hb[i] += __builtin_amdgcn_update_dpp(0u, hb[i], 0x128, 0xf, 0xf, false);
        }
        const unsigned w01 = (r16 & 8) ? hb[1] : hb[0], w23 = (r16 & 8) ? hb[3] : hb[2];
        unsigned cnt = (((r16 & 1) ? w23 : w01) >> ((r16 & 6) * 4)) & 255u;
        cnt += __shfl_xor(cnt, 16);
        cnt += __shfl_xor(cnt, 32);
        const unsigned nb = __builtin_amdgcn_update_dpp(0u, cnt, 0x111, 0xf, 0xf, true) +
                            __builtin_amdgcn_update_dpp(0u, cnt, 0x101, 0xf, 0xf, true);
        const unsigned key = (cnt << 16) | (nb << 4) | (unsigned)r16;
        unsigned rank = 0;
#define RK(t) rank += __builtin_amdgcn_update_dpp(0u, key, 0x120 + t, 0xf, 0xf, false) > key ? 1u : 0u;
        RK(1) RK(2) RK(3) RK(4) RK(5) RK(6) RK(7) RK(8) RK(9) RK(10) RK(11) RK(12) RK(13) RK(14) RK(15)
#undef RK
        const unsigned code = rank < 3 ? rank : rank < 6 ? (3u | ((rank - 3) << 2)) : rank < 10 ? (15u | ((rank - 6) << 4)) : 0x80u;
        const unsigned slot = rank < 3 ? rank : rank < 6 ? rank + 1 : rank + 2;
        *(volatile LAS unsigned char*)(tab + r16) = (unsigned char)code;
        *(volatile LAS unsigned char*)(tab + 16 + slot) = (unsigned char)(r16 << 3);
        CBAR();
        C = *(LAS u32x4*)tab;
      }
#pragma unroll 1
      for (int j = j0; j < jend; j += 4) {
        if (home) wait_vm<0>();
        else while (lds_ld(lds, CTL_CALL + 4) <= (unsigned)j) __builtin_amdgcn_s_sleep(1);
        CBAR();
        unsigned call = lds_ld(lds, ETAIL_OFF + ((wave - 5) & 3) * 1024);
        u32x4 nx[4];
#pragma unroll
        for (int pc = 0; pc < 4; ++pc)
          nx[pc] = *(const LAS u32x4*)(lds + (home ? RING_OFF + (NWS - 1) * SLOT_BYTES + (wave - 5) * 4096 : RAW_OFF + j * 4096) +
                                       pc * 1024 + lane * 16);
        CBAR();
        if (home && lane == 0)
          __hip_atomic_fetch_add((LAS unsigned*)(lds + CTL_E + 8), 1u, __ATOMIC_RELAXED, __HIP_MEMORY_SCOPE_WORKGROUP);
        unsigned d[16];
#pragma unroll
        for (int i = 0; i < 16; ++i) d[i] = nx[i >> 2][i & 3];
        LAS unsigned char* blk = lds + (home ? ETAIL_OFF + (wave - 5) * 1024 - TAIL : BIG_OFF + j * 4096);
        unsigned meta = 0x80030000u | 64u;
        u32x4 o0, o1, o2;
        unsigned c1[4], a2[4], n2[4], a3 = 0, n3 = 0, badv = 0;
#pragma unroll
        for (int hf = 0; hf < 2; ++hf) {
          unsigned d2[8], d3[8];
          u32x2 w1[8], w2[8];
          c1[2 * hf] = 0;
          c1[2 * hf + 1] = 0;
#pragma unroll
          for (int i = 0; i < 8; ++i) {
            const unsigned v = d[hf * 8 + i];
            const unsigned e7 = (v >> 3) & 0x07070707u;
            const unsigned lo = __builtin_amdgcn_perm(C[1], C[0], e7);
            const unsigned hi = __builtin_amdgcn_perm(C[3], C[2], e7);
            const unsigned code = __builtin_amdgcn_perm(hi, lo, ((v >> 4) & 0x04040404u) | 0x03020100u);
            badv |= code;
            const unsigned p1 = __builtin_amdgcn_udot4(code & 0x03030303u, 0x40100401u, 0u, false);
            c1[2 * hf + (i >> 2)] |= p1 << (8 * (i & 3));
            d2[i] = (code >> 2) & 0x03030303u;
            d3[i] = (code >> 4) & 0x03030303u;
            const unsigned p2 = __builtin_amdgcn_udot4(d2[i], 0x40100401u, 0u, false);
            w1[i] = *(LAS u32x2*)(lds + ELUT_OFF + p1 * 8);
            w2[i] = *(LAS u32x2*)(lds + ELUT_OFF + p2 * 8);
          }
          __builtin_amdgcn_sched_barrier(0);
#pragma unroll
          for (int i = 0; i < 8; ++i) {
            const int q = 2 * hf + (i >> 2);
            if ((i & 3) == 0) { a2[q] = 0; n2[q] = 0; }
            a2[q] |= __builtin_amdgcn_udot4(d2[i], w1[i][0], 0u, false) << n2[q];
            n2[q] += w1[i][1];
            a3 |= __builtin_amdgcn_udot4(d3[i], w2[i][0], 0u, false) << (n3 & 31);
            n3 += w2[i][1];
          }
          asm volatile("" : "+v"(a2[2 * hf]), "+v"(a2[2 * hf + 1]), "+v"(a3), "+v"(c1[2 * hf]), "+v"(c1[2 * hf + 1]));
        }
        const unsigned mine = (n2[0] + n2[1] + n2[2] + n2[3]) | (n3 << 16);
        const unsigned inc = wave_scan(mine);
        const unsigned tot = __builtin_amdgcn_readlane(inc, 63);
        const unsigned t2 = tot & 0xffff, t3 = tot >> 16;
        const bool raw = __any((badv & 0x80808080u) != 0 || n3 > 32) || t2 > 4096 || t3 > 1024 ||
                         (j >> 1) == NKC - 1;
        if (!raw) {
          *(LAS u32x4*)(blk + TAIL + lane * 16) = u32x4{0u, 0u, 0u, 0u};
          CBAR();
          const unsigned exc = inc - mine;
          unsigned pos = exc & 0xffff;
#pragma unroll
          for (int q = 0; q < 4; ++q) {
            const u64 v = (u64)a2[q] << (pos & 31);
            LAS unsigned* o = (LAS unsigned*)(blk + TAIL + (pos >> 5) * 4);
            __hip_atomic_fetch_or(o, (unsigned)v, __ATOMIC_RELAXED, __HIP_MEMORY_SCOPE_WORKGROUP);
            __hip_atomic_fetch_or(o + 1, (unsigned)(v >> 32), __ATOMIC_RELAXED, __HIP_MEMORY_SCOPE_WORKGROUP);
            pos += n2[q];
          }
          const unsigned tw = ((t2 + 31) >> 5) + ((t3 + 31) >> 5);
          const unsigned cl = tw <= 80 ? 0u : tw <= 96 ? 1u : 2u;
          const unsigned nl = tail_lanes(cl);
          {
            const unsigned p3 = exc >> 16;
            const u64 v = (u64)a3 << (p3 & 31);
            LAS unsigned* o = (LAS unsigned*)(blk + TAIL + nl * 16 - 8 - (p3 >> 5) * 4);
            __hip_atomic_fetch_or(o + 1, (unsigned)v, __ATOMIC_RELAXED, __HIP_MEMORY_SCOPE_WORKGROUP);
            __hip_atomic_fetch_or(o, (unsigned)(v >> 32), __ATOMIC_RELAXED, __HIP_MEMORY_SCOPE_WORKGROUP);
          }
#pragma unroll
          for (int i = 0; i < 4; ++i) {
            o0[i] = sm_pair(d[2 * i], d[2 * i + 1]);
            o1[i] = sm_pair(d[8 + 2 * i], d[9 + 2 * i]);
          }
          o2 = u32x4{c1[0], c1[1], c1[2], c1[3]};
          meta = 0x80000000u | (cl << 16) | nl;
        } else {
          u32x4 o3;
#pragma unroll
          for (int i = 0; i < 4; ++i) {
            o0[i] = home ? d[i] : sm_pair(d[2 * i], d[2 * i + 1]);
            o1[i] = home ? d[4 + i] : sm_pair(d[8 + 2 * i], d[9 + 2 * i]);
            o2[i] = home ? d[8 + i] : ex_pair(d[2 * i], d[2 * i + 1]);
            o3[i] = home ? d[12 + i] : ex_pair(d[8 + 2 * i], d[9 + 2 * i]);
          }
          *(LAS u32x4*)(blk + TAIL + lane * 16) = o3;
        }
        if (!home) {
          if (j >= NSM) {
            *(LAS u32x4*)(blk + lane * 16) = o0;
            *(LAS u32x4*)(blk + 1024 + lane * 16) = o1;
          }
          *(LAS u32x4*)(blk + 2048 + lane * 16) = o2;
          CBAR();
          lds_st(lds, CTL_META + j * 4, meta);
        } else {
          CBAR();
          const u32x4 o3 = *(const LAS u32x4*)(blk + TAIL + lane * 16);
          call = __builtin_amdgcn_readfirstlane(call) + 1;
          unsigned char* dst = my + (size_t)(call & 1) * PAR_BYTES + (size_t)rank * SHARD_BYTES + blk_off(wg & 31, pkc) + lane * 16;
          st16_sys(dst, o0);
          st16_sys(dst + 1024, o1);
          st16_sys(dst + 2048, o2);
          if ((unsigned)lane < (meta & 0xff))
            st16_sys((meta & 0x30000u) ? dst + 3072 : my + PTAIL_OFF + (call & 1) * PTAIL_PAR + ((wg & 31) * NKC + pkc) * 320 + lane * 16, o3);
          if (lane == 0) __hip_atomic_store(cnt, call, __ATOMIC_RELAXED, __HIP_MEMORY_SCOPE_AGENT);
          unsigned char* pf = p.ws[lane & 7] + FLAG_OFF + (size_t)(call & 1) * FLAG_PAR +
                              ((size_t)rank * (NRG / 2) + ((wg & 31) >> 1)) * 128 + 64 + (wg & 1) * 32 + part * 8;
          wait_vm<0>();
          unsigned prev = 0;
          if (lane == 0) {
            __hip_atomic_fetch_or((LAS unsigned*)(lds + CTL_E + 4), ((meta >> 16) & 3u) << (2 * pkc), __ATOMIC_RELAXED, __HIP_MEMORY_SCOPE_WORKGROUP);
            CBAR();
            prev = __hip_atomic_fetch_add((LAS unsigned*)(lds + CTL_E), 1u, __ATOMIC_RELAXED, __HIP_MEMORY_SCOPE_WORKGROUP);
          }
          CBAR();
          if (__builtin_amdgcn_readfirstlane(prev) == 3) {
            const unsigned cls = lds_ld(lds, CTL_E + 4);
            if (lane > 0 && lane < NR) st8_sys(pf, u32x2{call, cls});
          }
        }
      }
      if (!home) return;
    }
  }

  const int cb = (wg >> 3) & 3;
  const bool local = wave == 9;
  const int src = local ? rank : (rank + grp) & 7;

  if (wave > 9) {
    unsigned* cnt = (unsigned*)(my + CNT_OFF + (size_t)wg * 128) + wave;
    unsigned call = __hip_atomic_load(cnt, __ATOMIC_RELAXED, __HIP_MEMORY_SCOPE_AGENT);
    call = __builtin_amdgcn_readfirstlane(call) + 1;
    if (lane == 0) __hip_atomic_store(cnt, call, __ATOMIC_RELAXED, __HIP_MEMORY_SCOPE_AGENT);
    const unsigned fbase = (call - 1) * NKC;
    const int rg0 = (wg & 7) * 4;
    const size_t aoff = (size_t)(call & 1) * PAR_BYTES + (size_t)src * SHARD_BYTES + lane * 16;
    const unsigned char* lines = my + FLAG_OFF + (size_t)(call & 1) * FLAG_PAR + (size_t)src * (NRG / 2) * 128;
    LAS unsigned char* ctl = lds + CTL_A;
    auto slot_busy = [&](int r, int kc) { return __any(lds_ld(ctl, r * 32 + 24 + (lane & 1) * 4) + NAS <= (unsigned)turn_of(kc)); };
    bool bad = false;
    unsigned long long t0 = __builtin_amdgcn_s_memrealtime();
    auto give_up = [&]() {
      const unsigned long long dt = __builtin_amdgcn_s_memrealtime() - t0;
      return dt > 10000 && (dt > DEADLINE_TICKS ||
                            (unsigned)__builtin_amdgcn_readfirstlane(__hip_atomic_load((const unsigned*)(my + TMO_OFF), __ATOMIC_RELAXED,
                                                                                      __HIP_MEMORY_SCOPE_AGENT)) == call);
    };
    if (wave == 10) {
      const u64* pollp = (const u64*)(lines + ((wg & 7) * 2 + ((lane >> 2) & 1)) * 128 + (lane & 3) * 8);
      static_assert(NP < NAS, "the pushed chunks are the first to use their slots");
      int fk[2] = {0, 0};
      u64 pv = __hip_atomic_load(pollp, __ATOMIC_RELAXED, __HIP_MEMORY_SCOPE_SYSTEM);
      while (fk[0] < NP || fk[1] < NP) {
        const unsigned cls = (unsigned)(pv >> 32);
        bool any = false;
#pragma unroll
        for (int pr = 0; pr < 2; ++pr) {
#pragma unroll 1
          for (int n = 0; n < 3 && fk[pr] < NP && __all((lane >> 1) != pr * 2 || (int)((unsigned)pv - fbase) > fk[pr]); ++n) {
            CBAR();
            const int kc = fk[pr]++;
#pragma unroll
            for (int r = pr * 2; r < pr * 2 + 2; ++r) {
              const unsigned nl = tail_lanes(((unsigned)__builtin_amdgcn_readlane(cls, pr * 2 + r) >> (2 * kc)) & 3u);
              LAS unsigned char* dst = lds + ABUF_OFF + (r * NAS + turn_of(kc)) * 4096;
              const unsigned char* a = my + aoff + blk_off(rg0 + r, kc);
              __builtin_amdgcn_global_load_lds((const GAS void*)a, (LAS void*)dst, 16, 0, 17);
              __builtin_amdgcn_global_load_lds((const GAS void*)a, (LAS void*)dst, 16, 1024, 17);
              __builtin_amdgcn_global_load_lds((const GAS void*)a, (LAS void*)dst, 16, 2048, 17);
              if ((unsigned)lane < nl) __builtin_amdgcn_global_load_lds((const GAS void*)a, (LAS void*)dst, 16, 3072, 17);
            }
            any = true;
          }
        }
        if (any) {
          t0 = __builtin_amdgcn_s_memrealtime();
        } else if (give_up()) {
          bad = true;
          break;
        } else {
          __builtin_amdgcn_s_sleep(2);
        }
        pv = __hip_atomic_load(pollp, __ATOMIC_RELAXED, __HIP_MEMORY_SCOPE_SYSTEM);
        wait_vm<0>();
        if (any) {
          if (lane < 8 && (lane & 2)) *(volatile LAS u64*)(lds + CTL_T + (lane >> 2) * 16 + (lane & 1) * 8) = pv;
          CBAR();
          if (lane < 8 && !(lane & 2))
            *(volatile LAS u64*)(ctl + ((lane >> 1) + (lane & 1)) * 32) = ((u64)cls << 32) | (unsigned)(lane < 4 ? fk[0] : fk[1]);
        }
      }
    } else {
      const unsigned char* flags = my + FLAG_OFF + (size_t)(call & 1) * FLAG_PAR;
      const int sw = ((wg & 7) * 2 + ((lane >> 1) & 1)) * 128 + 64 + (lane & 1) * 32;
      int phase = 0;
      unsigned cls = 0, tgo = 0;
      constexpr int NB = 4 * (NKC - NP);
      unsigned d_src = 0, d_tail = 0, d_lds = 0, d_due = 0;
      while (true) {
        const bool on = phase == 0 ? lane >= 16 && lane < 16 + 4 * NPART : phase == 1 ? lane < 56 : lane < 16;
        const unsigned char* pp = phase == 0 ? lines + sw + ((lane >> 2) & 3) * 8
                                  : phase == 1 ? flags + ((rank + 1 + (lane >> 3)) & 7) * (NRG / 2) * 128 + sw + ((lane >> 2) & 1) * (NPART - 1) * 8
                                               : lines + lane * 128;
        u64 pv = 0;
        if (on) pv = __hip_atomic_load((const u64*)pp, __ATOMIC_RELAXED, __HIP_MEMORY_SCOPE_SYSTEM);
        if (phase < 2) {
          if (__all(!on || (int)((unsigned)pv - call) >= 0)) {
            if (phase == 0) {
              cls = on ? (unsigned)(pv >> 32) : 0u;
              cls |= __builtin_amdgcn_update_dpp(0u, cls, 0x124, 0xf, 0xf, false);
              cls |= __builtin_amdgcn_update_dpp(0u, cls, 0x128, 0xf, 0xf, false);
              __builtin_amdgcn_fence(__ATOMIC_ACQUIRE, "");
              wait_vm<0>();
              const int kc = NP + (lane >> 2), r = lane & 3;
              const unsigned nl = tail_lanes(((unsigned)__shfl(cls, 16 + r) >> (2 * kc)) & 3u);
              d_src = (unsigned)((call & 1) * PAR_BYTES + src * SHARD_BYTES) + blk_off(rg0 + r, kc);
              d_tail = nl == 20 ? (unsigned)(PTAIL_OFF + (call & 1) * PTAIL_PAR) + ((rg0 + r) * NKC + kc) * 320 - 3072 : d_src;
              d_lds = (ABUF_OFF + (r * NAS + turn_of(kc) % NAS) * 4096) | (nl - 1) | (turn_of(kc) << 24);
              const unsigned by = (192 + nl) * PACE;
              d_due = (wave_scan(by) - by) >> 10;
            }
            tgo = (unsigned)__builtin_amdgcn_s_memrealtime() + GATE;
            ++phase;
            continue;
          }
        } else {
          if (__builtin_popcountll(__ballot(on && (int)((unsigned)pv - fbase) >= NP - 1)) >= TSUM &&
              (int)((unsigned)__builtin_amdgcn_s_memrealtime() - tgo) >= 0)
            break;
        }
        if (give_up()) {
          bad = true;
          break;
        }
        __builtin_amdgcn_s_sleep(1);
      }
      CBAR();
      __builtin_amdgcn_s_setprio(3);
      const u64 rbv = (u64)p.ws[grp];
      const unsigned char* rb = (const unsigned char*)((u64)(unsigned)__builtin_amdgcn_readfirstlane((unsigned)rbv) |
                                                       ((u64)(unsigned)__builtin_amdgcn_readfirstlane((unsigned)(rbv >> 32)) << 32));
      const unsigned lo16 = lane * 16;
      int pub = 0;
      auto landed = [&](int n) {
        pub = n > pub ? n : pub;
        if ((lane >> 2) == 4)
          *(volatile LAS u64*)(ctl + (lane & 3) * 32 + 8) = ((u64)cls << 32) | (unsigned)(NP + ((pub + 3 - (lane & 3)) >> 2));
      };
      const unsigned tp0 = (unsigned)__builtin_amdgcn_s_memrealtime() + PSLACK;
      const unsigned char *na, *nt;
      unsigned nl, nmode;
      auto plan = [&](int i) {
        const unsigned us = lds_ld(ctl, (i & 3) * 32 + 24 + (lane & 1) * 4);
        na = rb + (unsigned)__builtin_amdgcn_readlane(d_src, i);
        nt = rb + (unsigned)__builtin_amdgcn_readlane(d_tail, i);
        nl = __builtin_amdgcn_readlane(d_lds, i);
        const unsigned slack = (unsigned)__builtin_amdgcn_readlane(d_due, i) - ((unsigned)__builtin_amdgcn_s_memrealtime() - tp0);
        nmode = (slack >> 31) | (((slack + FARB) >> 31) << 1);
        nmode |= (unsigned)__ballot(us + NAS <= (nl >> 24)) << 2;
        nmode = __builtin_amdgcn_readfirstlane(nmode);
        asm volatile("" : "+s"(na), "+s"(nt), "+s"(nl), "+s"(nmode));
      };
      auto turn = [&](int i, auto win) {
        constexpr int W = decltype(win)::value;
        const unsigned char *a = na, *t = nt;
        const unsigned l = nl;
        LAS unsigned char* dst = lds + (l & 0xfff000u);
        __builtin_amdgcn_global_load_lds((const GAS void*)(a + lo16), (LAS void*)dst, 16, 0, 0);
        plan(i + 1);
        wait_vm<W>();
        __builtin_amdgcn_global_load_lds((const GAS void*)(a + lo16), (LAS void*)dst, 16, 1024, 0);
        if (0 == ((W - 1) & 3)) landed(i - (W - 1) / 4);
        wait_vm<W>();
        __builtin_amdgcn_global_load_lds((const GAS void*)(a + lo16), (LAS void*)dst, 16, 2048, 0);
        if (1 == ((W - 1) & 3)) landed(i - (W - 1) / 4);
        wait_vm<W>();
        if ((unsigned)lane <= (l & 63u)) __builtin_amdgcn_global_load_lds((const GAS void*)(t + lo16), (LAS void*)dst, 16, 3072, 0);
        if (2 == ((W - 1) & 3)) landed(i - (W - 1) / 4);
        wait_vm<W>();
        if (3 == ((W - 1) & 3)) landed(i - (W - 1) / 4);
      };
      if (!bad) {
        plan(0);
#pragma unroll 1
        for (int i = 0; i < NB; ++i) {
          if (nmode > 3) {
            const int kc = NP + (i >> 2);
            if (slot_busy(i & 3, kc)) {
              wait_vm<0>();
              landed(i);
              while (slot_busy(i & 3, kc)) __builtin_amdgcn_s_sleep(1);
            }
            CBAR();
          }
          if (nmode & 2) turn(i, std::integral_constant<int, FWIN>{});
          else if (nmode & 1) turn(i, std::integral_constant<int, EWIN>{});
          else turn(i, std::integral_constant<int, PWIN>{});
        }
        wait_vm<8>();
        landed(NB - 2);
        wait_vm<4>();
        landed(NB - 1);
        wait_vm<0>();
        landed(NB);
      }
    }
    if (bad) {
      if (lane == 0) __hip_atomic_store((unsigned*)(my + TMO_OFF), call, __ATOMIC_RELAXED, __HIP_MEMORY_SCOPE_AGENT);
      if (lane < 4) *(volatile LAS u64*)(ctl + lane * 32 + (wave - 10) * 8) = 0xffffffff80000000ull;
    }
    if (wave == 11) return;
  }

  if (wave == 4 || wave == 10) {
    if (grp < 4) {
      while (lds_ld(lds, CTL_E + 8) < 4) __builtin_amdgcn_s_sleep(4);
      CBAR();
    }
    const int kk = lane >> 3, c = lane & 7;
    const unsigned char* wb = p.w + (size_t)kk * NL + cb * 128;
    const int f0 = (kk >> 1) & 3;
    const unsigned char* s0 = wb + ((c ^ f0) << 4);
    const unsigned char* s1 = wb + ((c ^ (f0 | 4)) << 4);
    int prev = -1;
    auto filled = [&]() {
      if (prev >= 0) lds_st(lds, CTL_FULL + (wave == 4 ? prev & 1 : 2) * 4, (prev >> 1) + 1);
    };
#pragma unroll 1
    for (int s = wave == 4 ? 0 : 2 * WT0 + 1; s < 2 * NKC; s += s < 2 * WT0 ? 1 : 2) {
      if (s >= NWS) {
        const unsigned need = (s >> 1) - WCH + 1;
        bool first = true;
        while (true) {
          unsigned d = lds_ld(lds, CTL_DONE + (lane & 7) * 4);
          if (__all(d >= need)) break;
          if (first) {
            wait_vm<0>();
            filled();
            first = false;
          }
          __builtin_amdgcn_s_sleep(4);
        }
        CBAR();
      }
      LAS unsigned char* dst = lds + RING_OFF + (s % NWS) * SLOT_BYTES;
      const size_t koff = (size_t)(chunk_of(s >> 1) * 2 + (s & 1)) * 128 * NL;
#pragma unroll
      for (int j = 0; j < 16; ++j) {
        const unsigned char* gp = (((j >> 1) & 1) ? s1 : s0) + koff + (size_t)j * 8 * NL;
        __builtin_amdgcn_global_load_lds((const GAS void*)gp, (LAS void*)(dst + j * 1024), 16, 0, 0);
      }
      wait_vm<16>();
      filled();
      prev = s;
    }
    wait_vm<0>();
    filled();
    return;
  }

#define SB() __builtin_amdgcn_sched_barrier(0)
  if (!local && wave > 4) {
    const int gw = wave - 5;
    CBAR();
    LAS unsigned char* ctl = lds + CTL_A + gw * 32;
    LAS unsigned char* tab = lds + CTL_T + (gw >> 1) * 16;
    const unsigned e3 = SCR_OFF + gw * SCR_BYTES;
    __builtin_amdgcn_s_setprio(2);
    int k0 = 0, k1 = NP;
#pragma unroll 1
    while (k0 < NP || k1 < NKC) {
      const u32x4 st = *(volatile LAS u32x4*)ctl;
      const bool a0 = k0 < NP && (unsigned)__builtin_amdgcn_readfirstlane(st[0]) > (unsigned)k0;
      if (!a0 && !(k1 < NKC && k0 >= 1 && (unsigned)__builtin_amdgcn_readfirstlane(st[2]) > (unsigned)k1)) {
        __builtin_amdgcn_s_setprio(0);
        __builtin_amdgcn_s_sleep(2);
        continue;
      }
      __builtin_amdgcn_s_setprio(2);
      CBAR();
      const int kc = a0 ? k0++ : k1++;
      const unsigned cl = ((unsigned)__builtin_amdgcn_readfirstlane(a0 ? st[1] : st[3]) >> (2 * kc)) & 3u;
      const unsigned boff = ABUF_OFF + (gw * NAS + turn_of(kc) % NAS) * 4096;
      LAS unsigned char* blk = lds + boff;
      if (cl == 3) {
        if (a0) {
          u32x4 v[4];
#pragma unroll
          for (int pc = 0; pc < 4; ++pc) v[pc] = *(LAS u32x4*)(blk + pc * 1024 + lane * 16);
          CBAR();
#pragma unroll
          for (int pc = 0; pc < 4; ++pc) {
            u32x4 o;
#pragma unroll
            for (int k = 0; k < 2; ++k) {
              const unsigned sm = v[pc >> 1][(pc & 1) * 2 + k], ex = v[2 + (pc >> 1)][(pc & 1) * 2 + k];
              const unsigned t = (ex & SM_MASK) | (sm & ~SM_MASK);
              o[2 * k] = (sm & SM_MASK) | (ex & ~SM_MASK);
              o[2 * k + 1] = __builtin_amdgcn_alignbit(t, t, 4);
            }
            *(LAS u32x4*)(blk + pc * 1024 + lane * 16) = o;
          }
          CBAR();
          lds_st(ctl, 16, kc + 1);
        }
        continue;
      }
      {
        const unsigned l3 = *(LAS unsigned short*)(blk + TAIL + tail_lanes(cl) * 16 - 4 - (lane >> 1) * 4 + (lane & 1) * 2);
        const unsigned T3 = *(LAS unsigned*)(tab + 8);
        const unsigned l2a = *(LAS unsigned*)(blk + TAIL + lane * 4);
        const u32x4 c1 = *(LAS u32x4*)(blk + 2048 + lane * 16);
        const u32x2 T12 = *(LAS u32x2*)(tab);
        const u32x4 sm0 = *(LAS u32x4*)(blk + lane * 16), sm1 = *(LAS u32x4*)(blk + 1024 + lane * 16);
        const unsigned l2b = *(LAS unsigned*)(blk + TAIL + 256 + lane * 4);
        SB();
        auto lut2 = [&](unsigned c, int i) { return *(LAS u32x2*)(lds + LUT_OFF + ((c >> (8 * i)) & 255u) * 8); };
        auto window = [&](unsigned off) {
          const LAS unsigned* w = (const LAS unsigned*)(lds + (off & ~3u));
          return u32x2{w[0], w[1]};
        };
        auto pick = [&](u32x2 w, unsigned off, unsigned T, unsigned sel) {
          return __builtin_amdgcn_perm(__builtin_amdgcn_alignbyte(w[1], w[0], off), T, sel);
        };
        {
          const unsigned t = l3 | (l3 << 12);
          const unsigned u = t & 0x000f000fu, v = (t >> 8) & 0x000f000fu;
          *(LAS u32x2*)(lds + e3 + lane * 8) = u32x2{__builtin_amdgcn_perm(0u, T3, (u | (u << 6)) & 0x03030303u),
                                                     __builtin_amdgcn_perm(0u, T3, (v | (v << 6)) & 0x03030303u)};
        }
        u32x2 s2[4], s1[16];
#pragma unroll
        for (int i = 0; i < 4; ++i) s2[i] = lut2(l2a, i);
        SB();
        const unsigned mine = (esc_count(c1[0]) + esc_count(c1[1]) + esc_count(c1[2]) + esc_count(c1[3])) | (esc_count(l2a) << 16);
#pragma unroll
        for (int i = 0; i < 8; ++i) s1[i] = lut2(c1[i >> 2], i & 3);
        const unsigned inc = wave_scan(mine);
        const unsigned exc = inc - mine;
        const unsigned tot = __builtin_amdgcn_readlane(inc, 63);
        CBAR();
        SB();
        unsigned o2[4], o1[16];
        u32x2 w2[4], w1[16];
        o2[0] = e3 + (exc >> 16);
#pragma unroll
        for (int i = 1; i < 4; ++i) o2[i] = o2[i - 1] + s2[i - 1][1];
#pragma unroll
        for (int i = 0; i < 4; ++i) w2[i] = window(o2[i]);
        SB();
#pragma unroll
        for (int i = 8; i < 16; ++i) s1[i] = lut2(c1[i >> 2], i & 3);
        SB();
        {
          auto put = [&](const u32x4& o, LAS unsigned char* to) {
            const unsigned nxt = __builtin_amdgcn_update_dpp(0u, o[0], 0x130, 0xf, 0xf, true);
            *(LAS u32x4*)(to + lane * 32) = u32x4{o[0], o[1], o[1], o[2]};
            *(LAS u32x4*)(to + lane * 32 + 16) = u32x4{o[2], o[3], o[3], nxt};
          };
          u32x4 o;
#pragma unroll
          for (int i = 0; i < 4; ++i) o[i] = pick(w2[i], o2[i], T12[1], s2[i][0]);
          put(o, blk);
          if ((tot & 0xffff) > 1024) {
            const unsigned m2 = esc_count(l2b);
            unsigned off = e3 + (tot >> 16) + wave_scan(m2) - m2;
#pragma unroll
            for (int i = 0; i < 4; ++i) s2[i] = lut2(l2b, i);
#pragma unroll
            for (int i = 0; i < 4; ++i) { o[i] = pick(window(off), off, T12[1], s2[i][0]); off += s2[i][1]; }
            put(o, blk + 2048);
            if (lane == 0) *(LAS unsigned*)(blk + 2044) = o[0];
          }
        }
        CBAR();
        o1[0] = boff / 2 - 4 + (exc & 0xffff);
#pragma unroll
        for (int i = 0; i < 16; ++i) {
          if (i) o1[i] = o1[i - 1] + s1[i - 1][1];
          w1[i] = *(const LAS u32x2*)(lds + ((o1[i] << 1) | 7u) + 1);
        }
        CBAR();
        SB();
#pragma unroll
        for (int pc = 0; pc < 4; ++pc) {
          u32x4 o;
#pragma unroll
          for (int k = 0; k < 4; ++k) {
            const int i = pc * 4 + k;
            const unsigned s = i < 8 ? sm0[i >> 1] : sm1[(i >> 1) & 3];
            o[k] = (((i & 1) ? __builtin_amdgcn_alignbit(s, s, 4) : s) & SM_MASK) | pick(w1[i], o1[i], T12[0], s1[i][0]);
          }
          *(LAS u32x4*)(blk + pc * 1024 + lane * 16) = o;
        }
      }
      CBAR();
      lds_st(ctl, a0 ? 16 : 20, kc + 1);
    }
    return;
  }

  const int gw = local ? 4 : wave;
  const int task = local ? grp - 4 : wave;
  const int ch = task & 1;
  const int rg0 = (wg & 7) * 4 + (task >> 1) * 2;
  const int q = r16 >> 1;
  const int wl = RING_OFF + (g * 16 + q) * 128 + (((((q >> 1) & 3) | ((g & 1) << 2)) ^ (ch * 4)) << 4) + (r16 & 1) * 8;
  const __amdgpu_buffer_rsrc_t ors = __builtin_amdgcn_make_buffer_rsrc((void*)p.out, 0, NR * MS * NL * 4, 0x00020000);
  const int ooff = ((src * MS + rg0 * 16 + (r16 & 7)) * NL + cb * 128 + ch * 64 + (r16 >> 3) * 16 + g * 4) * 4;
  const unsigned char* xs = p.x + (size_t)(rg0 * 16 + r16) * KD + g * 16;

  auto gemm = [&](auto tag) {
    constexpr bool LOCAL = decltype(tag)::value;
    f32x4 acc[8];
#pragma unroll
    for (int i = 0; i < 8; ++i) acc[i] = f32x4{0.f, 0.f, 0.f, 0.f};
    u32x4 nx[8];
    if constexpr (LOCAL) {
#pragma unroll
      for (int i = 0; i < 8; ++i) nx[i] = *(const u32x4*)(xs + (size_t)(i >> 2) * (16 * KD) + (i & 3) * 64);
    }
    const float sc = p.sx[0] * p.sw[0];
    unsigned badm = 0;
    int slot = 0, aslot = 0;
    LAS unsigned char* ctl = lds + CTL_A + (wave & 2) * 32;
    const int so = lane < 16 ? CTL_A + (wave & 2) * 32 + lane * 4 : CTL_FULL + (lane & 3) * 4;
    unsigned sv = lds_ld(lds, so);
#pragma unroll 1
    for (int turn = 0; turn < NKC; ++turn) {
      const int kc = chunk_of(turn);
      const unsigned thr = lane < 16 ? kc : turn;
      const int pl = kc < NP ? 0 : 1;
      auto ready = [&]() {
        const unsigned gt = (unsigned)__ballot(sv > thr);
        const unsigned to = (unsigned)__ballot((int)sv < 0) >> (2 * pl);
        const unsigned raw = pl ? (gt >> 2) & ((unsigned)__ballot(((sv >> (2 * kc)) & 3u) == 3u) >> 3) : 0u;
        badm |= to;
        return (((gt >> (4 + pl)) | to | raw) & 0x101u) | (gt & (gt >> (turn < WT0 ? 1 : 2)) & 0x10000u);
      };
      unsigned rd = ready();
      auto await = [&](unsigned bit) {
        while (!(rd & bit)) {
          if (turn < NKC - 2) __builtin_amdgcn_s_sleep(4); else __builtin_amdgcn_s_sleep(1);
          sv = lds_ld(lds, so);
          rd = ready();
        }
      };
      auto x_read = [&](int r) {
        CBAR();
        LAS unsigned char* blk = lds + ABUF_OFF + (((wave & 2) + r) * NAS + aslot) * 4096;
#pragma unroll
        for (int pc = 0; pc < 4; ++pc) nx[r * 4 + pc] = *(LAS u32x4*)(blk + pc * 1024 + lane * 16);
        CBAR();
        lds_st(ctl, r * 32 + 24 + ch * 4, turn + 1);
      };
      auto store = [&](int r) {
        const float s = !LOCAL && (badm & 0x101u) ? __builtin_nanf("") : sc;
#pragma unroll
        for (int t = 0; t < 2; ++t) {
          const u32x4 a = __builtin_bit_cast(u32x4, acc[r * 4 + 2 * t] * s), b = __builtin_bit_cast(u32x4, acc[r * 4 + 2 * t + 1] * s);
          u32x4 lo, hi;
#pragma unroll
          for (int i = 0; i < 4; ++i) {
            lo[i] = __builtin_amdgcn_update_dpp(a[i], b[i], 0x128, 0xf, 0xc, false);
            hi[i] = __builtin_amdgcn_update_dpp(b[i], a[i], 0x128, 0xf, 0x3, false);
          }
          __builtin_amdgcn_raw_buffer_store_b128(lo, ors, ooff + t * 128, r * 16 * NL * 4, 18  );
          __builtin_amdgcn_raw_buffer_store_b128(hi, ors, ooff + t * 128, (r * 16 + 8) * NL * 4, 18  );
        }
      };
      await(0x10000u);
      CBAR();
      i32x8 wa[8];
#pragma unroll
      for (int f = 0; f < 8; ++f) {
        LAS unsigned char* a = lds + slot * (2 * SLOT_BYTES) + (wl ^ ((f & 3) << 4)) + (f >> 2) * SLOT_BYTES;
        const i32x2 t0 = __builtin_amdgcn_ds_read_tr8_b64_v2i32((LAS i32x2*)(a));
        const i32x2 t1 = __builtin_amdgcn_ds_read_tr8_b64_v2i32((LAS i32x2*)(a + 1024));
        const i32x2 t2 = __builtin_amdgcn_ds_read_tr8_b64_v2i32((LAS i32x2*)(a + 8192));
        const i32x2 t3 = __builtin_amdgcn_ds_read_tr8_b64_v2i32((LAS i32x2*)(a + 9216));
        wa[f][0] = t0[0]; wa[f][1] = t0[1]; wa[f][2] = t1[0]; wa[f][3] = t1[1];
        wa[f][4] = t2[0]; wa[f][5] = t2[1]; wa[f][6] = t3[0]; wa[f][7] = t3[1];
      }
      CBAR();
      lds_st(lds, CTL_DONE + gw * 4, turn + 1);
      auto mma = [&](int r) {
#pragma unroll
        for (int f = 0; f < 8; ++f) {
          i32x8 xb;
#pragma unroll
          for (int i = 0; i < 8; ++i) xb[i] = nx[r * 4 + (f >> 2) * 2 + (i >> 2)][i & 3];
          acc[r * 4 + (f & 3)] = __builtin_amdgcn_mfma_scale_f32_16x16x128_f8f6f4(wa[f], xb, acc[r * 4 + (f & 3)], 1, 0, 0, 0, 0, 0);
        }
      };
      if constexpr (LOCAL) {
        sv = lds_ld(lds, so);
#pragma unroll
        for (int r = 0; r < 2; ++r) {
          mma(r);
          SB();
#pragma unroll
          for (int pc = 0; pc < 4; ++pc)
            nx[r * 4 + pc] = *(const u32x4*)(xs + (size_t)r * (16 * KD) + chunk_of((turn + 1) & (NKC - 1)) * 256 + pc * 64);
          SB();
        }
        if (turn == NKC - 1) {
          store(0);
          store(1);
        }
      } else {
        await(1u);
        x_read(0);
        const bool both = rd & 0x100u;
        if (both) x_read(1);
        sv = lds_ld(lds, so);
        SB();
        mma(0);
        if (turn == NKC - 1) store(0);
        if (!both) {
          rd = ready();
          await(0x100u);
          x_read(1);
        }
        mma(1);
        if (turn == NKC - 1) store(1);
      }
      slot = slot == WCH - 1 ? 0 : slot + 1;
      aslot = aslot == NAS - 1 ? 0 : aslot + 1;
    }
  };
  if (local) gemm(std::true_type{}); else gemm(std::false_type{});
}

}

extern "C" void kernel_launch(int rank, int nranks, ncclComm_t comm, hipStream_t stream,
                              void* const* d_in, const int* in_sizes, int n_in, void* d_out,
                              int out_size, const int* in_dtypes, int out_dtype,
                              void* const* d_out_all, void* const* d_ws_all, size_t ws_bytes) {
  (void)comm; (void)in_dtypes; (void)out_dtype; (void)d_out_all;
  if (nranks != NR || n_in != 4 || (unsigned)rank >= (unsigned)NR || ws_bytes < WS_NEED ||
      in_sizes[0] != MS * KD || in_sizes[1] != KD * NL || out_size != NR * MS * NL) {
    fprintf(stderr, "kernel_launch: unexpected problem shape\n");
    return;
  }
  Params p;
  p.x = (const unsigned char*)d_in[0];
  p.w = (const unsigned char*)d_in[1];
  p.sx = (const float*)d_in[2];
  p.sw = (const float*)d_in[3];
  p.out = (float*)d_out;
  for (int i = 0; i < NR; ++i) p.ws[i] = (unsigned char*)d_ws_all[(rank + i) % NR];
  p.rank = rank;
  p.pad = 0;
  hipLaunchKernelGGL(ag_gemm_kernel, dim3(NWG), dim3(NWAVE * 64), 0, stream, p);
  hipError_t e = hipGetLastError();
  if (e != hipSuccess) fprintf(stderr, "launch error %s\n", hipGetErrorString(e));
}
```
